```python
import numpy as np

kernel_name = "gpu_gemm_rs_m4096_k4096_n32768_bf16_gelu"


ABSMAX_RTOL = 0.008

COMPUTE_DTYPE = "bfloat16"
EPILOGUE = "gelu"
IN_QUANT = None
OUT_QUANT = None
IN_DTYPES = {"x": "bfloat16", "w_mat": "bfloat16"}
WORLD_SIZE = 8

def _to_bf16(x):
    u = np.ascontiguousarray(x, dtype=np.float32).view(np.uint32)
    return ((u + 0x7FFF + ((u >> 16) & 1)) & 0xFFFF0000).view(np.float32)

def setup_inputs():

    x = _to_bf16(np.random.randn(4096, 4096).astype(np.float32))
    w_mat = _to_bf16((np.random.randn(4096, 32768) / np.sqrt(4096)).astype(np.float32))
    return {"x": x, "w_mat": w_mat}

def _epilogue(y):
    c = 0.7978845608028654
    return (0.5 * y * (1.0 + np.tanh(c * (y + 0.044715 * y ** 3)))).astype(np.float32)

def reference(x, w_mat):
    y = (x @ w_mat).astype(np.float32)
    return _epilogue(y)
```

```cpp
#include <hip/hip_runtime.h>
#include <rccl/rccl.h>
#include <cstdio>
#include <cstdint>

#define HIP_CHECK(expr) do { hipError_t _e = (expr); if (_e != hipSuccess) { \
    fprintf(stderr, "HIP error %s at %s:%d\n", hipGetErrorString(_e), __FILE__, __LINE__); \
    return; } } while (0)

namespace {
typedef unsigned u32x4 __attribute__((ext_vector_type(4)));
typedef short bf16x8 __attribute__((ext_vector_type(8)));
typedef float f32x4 __attribute__((ext_vector_type(4)));
#define LAS __attribute__((address_space(3)))

constexpr int NR = 8;
constexpr int MM = 4096, KS = 512, NN = 32768, NS = 4096, MS = 512;
constexpr unsigned IMG = 16384;
constexpr unsigned SLAB = 32 * IMG;
constexpr unsigned OFF_A = 1u << 20;
constexpr unsigned OFF_B = OFF_A + 64 * SLAB;
constexpr unsigned OFF_W = OFF_B + 64 * SLAB;
constexpr unsigned OFF_O = OFF_W + 64 * SLAB;
constexpr unsigned WS_NEED = OFF_O + 256u * 131072u;
constexpr unsigned OUT_BYTES = (unsigned)MS * NN * 2u;
constexpr int F_TICKET = 0, F_TMO = 1, F_SCNT = 2, F_SFLAG = 16, F_INCNT = 80, F_INB = 144, F_OUT = 160;
constexpr int NPULL = 28, NCOMM = 32, NWG = 256, NGEMM = NWG - NCOMM;
constexpr int NSTAGER = NWG - NPULL;
constexpr int UNITS_PER_CHUNK = 72, NUNITS = 8 * UNITS_PER_CHUNK;
constexpr unsigned long long DEADLINE = 50000000ull;
constexpr int HTB = 16384, STAGE_BYTES = 8 * HTB, LDS_BYTES = STAGE_BYTES + 256;
constexpr int AUX_SYS = 17, AUX_AGENT = 16;

struct Params { const unsigned short* x; const unsigned short* w; unsigned char* ws[NR]; unsigned short* out; int rank; };

__device__ __forceinline__ int lds_byte(int r, int c) { const int st = (r >> 4) * 2 + (c >> 5), rr = r & 15, cc = c & 31, ob = rr * 64 + cc * 2; return st * 1024 + (ob ^ (((ob >> 9) & 1) << 5)); }
__device__ __forceinline__ void stage_rc(int b, int& R, int& C) { const int st = b / 1024, sb = b % 1024, swz = sb ^ (((sb >> 9) & 1) << 5); R = (st >> 1) * 16 + swz / 64; C = (st & 1) * 32 + (swz % 64) / 2; }

__device__ __forceinline__ unsigned ld_sys(const unsigned* p) { return __hip_atomic_load(p, __ATOMIC_RELAXED, __HIP_MEMORY_SCOPE_SYSTEM); }
__device__ __forceinline__ void st_sys(unsigned* p, unsigned v) { __hip_atomic_store(p, v, __ATOMIC_RELAXED, __HIP_MEMORY_SCOPE_SYSTEM); }
__device__ __forceinline__ void wait_ge(const unsigned* flag, unsigned val, unsigned* tmo, unsigned call) {
    const unsigned long long t0 = __builtin_amdgcn_s_memrealtime();
    while (ld_sys(flag) < val) {
        if (__hip_atomic_load(tmo, __ATOMIC_RELAXED, __HIP_MEMORY_SCOPE_AGENT) == call || __builtin_amdgcn_s_memrealtime() - t0 > DEADLINE) {
            __hip_atomic_store(tmo, call, __ATOMIC_RELAXED, __HIP_MEMORY_SCOPE_AGENT); break; }
        __builtin_amdgcn_s_sleep(2);
    }
}
__device__ __forceinline__ unsigned cvt_pk_bf16(float lo, float hi) { unsigned r; asm volatile("v_cvt_pk_bf16_f32 %0, %1, %2" : "=v"(r) : "v"(lo), "v"(hi)); return r; }
__device__ __forceinline__ float gelu_tanh(float y) {
    const float u = y * (0.7978845608028654f + 0.035677408136300125f * y * y);
    const float e = __builtin_amdgcn_exp2f(u * -2.8853900817779268f);
    return y * __builtin_amdgcn_rcpf(1.0f + e);
}
#define WAIT_V(n) asm volatile("s_waitcnt vmcnt(" #n ")" ::: "memory")
#define WAIT_L(n) asm volatile("s_waitcnt lgkmcnt(" #n ")" ::: "memory")
#define BAR __builtin_amdgcn_s_barrier()
#define SCHED __builtin_amdgcn_sched_barrier(0)
#define CFENCE asm volatile("" ::: "memory")

__device__ __forceinline__ void stage_units(const Params& p, LAS unsigned char* lds, unsigned call, int first) {
    const int tid = threadIdx.x, rank = p.rank;
    unsigned char* wsl = p.ws[rank]; unsigned* fl = (unsigned*)wsl;
    const auto rs = __builtin_amdgcn_make_buffer_rsrc(wsl, 0, (int)WS_NEED, 0x00020000);
    for (int u = first; u < NUNITS; u += NSTAGER) {
        const int c = u / UNITS_PER_CHUNK, v = u % UNITS_PER_CHUNK;
        if (v >= 64) {
            const int vv = v - 64;
            const unsigned dbase = OFF_A + (unsigned)(rank * 8 + c) * SLAB + (unsigned)vv * 4u * IMG;
            u32x4 t[8];
#pragma unroll
            for (int i = 0; i < 8; ++i) { const int pc = tid + 512 * i, img = pc >> 10; int R, C; stage_rc((pc & 1023) * 16, R, C);
                t[i] = *(const u32x4*)(p.x + (size_t)(vv * 512 + img * 128 + R) * KS + c * 64 + C); }
#pragma unroll
            for (int i = 0; i < 8; ++i) __builtin_amdgcn_raw_buffer_store_b128(t[i], rs, (int)(dbase + (unsigned)(tid + 512 * i) * 16u), 0, AUX_SYS);
        } else {
            const int d = v >> 3, kb = tid >> 6, nb = tid & 63;
            const unsigned dbase = (d == rank ? OFF_B + (unsigned)(rank * 8 + c) * SLAB : OFF_W + (unsigned)(d * 8 + c) * SLAB) + (unsigned)(v & 7) * 4u * IMG;
            u32x4 r[8];
#pragma unroll
            for (int i = 0; i < 8; ++i) r[i] = *(const u32x4*)(p.w + (size_t)(c * 64 + kb * 8 + i) * NN + v * 512 + nb * 8);
#pragma unroll
            for (int j = 0; j < 8; ++j) {
                u32x4 o;
#pragma unroll
                for (int e = 0; e < 4; ++e) o[e] = __builtin_amdgcn_perm(r[2 * e + 1][j >> 1], r[2 * e][j >> 1], (j & 1) ? 0x07060302u : 0x05040100u);
                const int nl = nb * 8 + j, img = nl >> 7, rowi = nl & 127, cc = rowi & 31;
                const int rho = (rowi & ~31) + 16 * ((cc >> 2) & 1) + 4 * (cc >> 3) + (cc & 3);
                *(LAS u32x4*)(lds + img * IMG + lds_byte(rho, kb * 8)) = o;
            }
            __syncthreads();
#pragma unroll
            for (int i = 0; i < 8; ++i) { const unsigned b = (unsigned)(tid + 512 * i) * 16u; const u32x4 t = *(const LAS u32x4*)(lds + b);
                __builtin_amdgcn_raw_buffer_store_b128(t, rs, (int)(dbase + b), 0, AUX_SYS); }
        }
        WAIT_V(0);
        __syncthreads();
        if (tid == 0) {
            const unsigned old = __hip_atomic_fetch_add(fl + (F_SCNT + c) * 32, 1u, __ATOMIC_RELAXED, __HIP_MEMORY_SCOPE_AGENT);
            if (old + 1u == call * (unsigned)UNITS_PER_CHUNK) {
                __builtin_amdgcn_fence(__ATOMIC_RELEASE, "");
                WAIT_V(0);
#pragma unroll
                for (int d = 0; d < NR; ++d) st_sys((unsigned*)p.ws[d] + (F_SFLAG + rank * 8 + c) * 32, call);
            }
        }
    }
}

struct Desc { unsigned soff, doff; int kind; int flag; int sig; };
constexpr int NIT = 192;
__device__ __forceinline__ Desc puller_desc(int it, int rank, int s, int q) {
    Desc d; d.flag = -1; d.sig = -1; d.kind = 0;
    if (it < 120) {
        const int c = it / 15; int j = it - c * 15;
        if (j == 0) d.flag = F_SFLAG + s * 8 + c;
        if (j == 14) d.sig = F_INCNT + s * 8 + c;
        if (j < 8) { const unsigned off = (unsigned)q * 131072u + (unsigned)j * 16384u; d.soff = OFF_A + (unsigned)(8 * s + c) * SLAB + off; d.doff = d.soff; }
        else { j -= 8; const unsigned off = (unsigned)q * 114688u + (unsigned)j * 16384u; d.soff = OFF_W + (unsigned)(rank * 8 + c) * SLAB + off; d.doff = OFF_B + (unsigned)(8 * s + c) * SLAB + off; }
    } else if (it < 128) {
        const int c = it - 120; const unsigned off = 458752u + (unsigned)q * 16384u;
        d.soff = OFF_W + (unsigned)(rank * 8 + c) * SLAB + off; d.doff = OFF_B + (unsigned)(8 * s + c) * SLAB + off;
        if (it == 127) d.sig = F_INB + s;
    } else if (it < 184) {
        const int k = (it - 128) >> 3, j = (it - 128) & 7, idx = q + 4 * k, pml = idx / 14, pn = idx - pml * 14;
        d.kind = 1; if (j == 0) d.flag = F_OUT + s * 48 + idx;
        d.soff = OFF_O + (unsigned)((2 * rank + pml) * 16 + pn) * 131072u + (unsigned)j * 16384u;
        d.doff = (unsigned)(pml * 256 + j * 32) * 65536u + (unsigned)(s * 4096 + pn * 256) * 2u;
    } else {
        const int k = (it - 184) >> 1, j = (it - 184) & 1, idx = q + 4 * k, al = idx >> 2, b = 28 + (idx & 3);
        d.kind = 2; if (j == 0) d.flag = F_OUT + s * 48 + 28 + idx;
        d.soff = OFF_O + (unsigned)((2 * rank + (al >> 1)) * 16 + (b >> 1)) * 131072u + (unsigned)((al & 1) * 2 + (b & 1)) * 32768u + (unsigned)j * 16384u;
        d.doff = (unsigned)(al * 128 + j * 64) * 65536u + (unsigned)(s * 4096 + b * 128) * 2u;
    }
    return d;
}
__device__ __forceinline__ void puller(const Params& p, unsigned call, int l, int q) {
    const int tid = threadIdx.x, rank = p.rank, s = (rank + 1 + l) & 7;
    unsigned* fl = (unsigned*)p.ws[rank]; unsigned* tmo = fl + F_TMO * 32;
    const auto rsrc = __builtin_amdgcn_make_buffer_rsrc(p.ws[s], 0, (int)WS_NEED, 0x00020000);
    const auto rdst = __builtin_amdgcn_make_buffer_rsrc(p.ws[rank], 0, (int)WS_NEED, 0x00020000);
    const auto rout = __builtin_amdgcn_make_buffer_rsrc(p.out, 0, (int)OUT_BYTES, 0x00020000);
    const unsigned b0 = (unsigned)tid * 16u, b1 = b0 + 8192u;
    u32x4 c0, c1, n0, n1;
    Desc dc = puller_desc(0, rank, s, q);
    if (tid == 0) wait_ge(fl + dc.flag * 32, call, tmo, call);
    WAIT_V(0); BAR; CFENCE;
    c0 = __builtin_amdgcn_raw_buffer_load_b128(rsrc, (int)(dc.soff + b0), 0, AUX_SYS);
    c1 = __builtin_amdgcn_raw_buffer_load_b128(rsrc, (int)(dc.soff + b1), 0, AUX_SYS);
    int sig1 = -1, sig2 = -1;
    bool bad = false;
    for (int it = 0; it < NIT; ++it) {
        Desc dn = dc;
        if (it + 1 < NIT) {
            dn = puller_desc(it + 1, rank, s, q);
            if (dn.flag >= 0) {
                if (tid == 0) wait_ge(fl + dn.flag * 32, call, tmo, call);
                CFENCE; BAR; CFENCE;
                if (dn.kind != 0) bad = bad || (__hip_atomic_load(tmo, __ATOMIC_RELAXED, __HIP_MEMORY_SCOPE_AGENT) == call);
            }
            n0 = __builtin_amdgcn_raw_buffer_load_b128(rsrc, (int)(dn.soff + b0), 0, AUX_SYS);
            n1 = __builtin_amdgcn_raw_buffer_load_b128(rsrc, (int)(dn.soff + b1), 0, AUX_SYS);
        }
        CFENCE;
        if (sig2 >= 0) {
            WAIT_V(4); BAR; CFENCE;
            if (tid == 0) __hip_atomic_fetch_add(fl + sig2 * 32, 1u, __ATOMIC_RELAXED, __HIP_MEMORY_SCOPE_AGENT);
        }
        if (dc.kind == 0) {
            __builtin_amdgcn_raw_buffer_store_b128(c0, rdst, (int)(dc.doff + b0), 0, AUX_AGENT);
            __builtin_amdgcn_raw_buffer_store_b128(c1, rdst, (int)(dc.doff + b1), 0, AUX_AGENT);
        } else {
            const int sh = dc.kind == 1 ? 9 : 8; const unsigned m = (1u << sh) - 1u;
            if (bad) { c0 = (u32x4){0x7fc07fc0u, 0x7fc07fc0u, 0x7fc07fc0u, 0x7fc07fc0u}; c1 = c0; }
            __builtin_amdgcn_raw_buffer_store_b128(c0, rout, (int)(dc.doff + (b0 >> sh) * 65536u + (b0 & m)), 0, 0);
            __builtin_amdgcn_raw_buffer_store_b128(c1, rout, (int)(dc.doff + (b1 >> sh) * 65536u + (b1 & m)), 0, 0);
        }
        CFENCE;
        sig2 = sig1; sig1 = dc.sig; dc = dn; c0 = n0; c1 = n1;
    }
}

__device__ __forceinline__ int kt_of(int t, int rank) {
    if (t < 8) return rank * 8 + t;
    const int tt = t - 8, c = tt / 7, l = tt - c * 7; return ((rank + 1 + l) & 7) * 8 + c;
}
__device__ __forceinline__ void store_tile_rows(const f32x4& v0, const f32x4& v1, bool bad, u32x4& w) {
    w.x = cvt_pk_bf16(gelu_tanh(v0[0]), gelu_tanh(v0[1])); w.y = cvt_pk_bf16(gelu_tanh(v0[2]), gelu_tanh(v0[3]));
    w.z = cvt_pk_bf16(gelu_tanh(v1[0]), gelu_tanh(v1[1])); w.w = cvt_pk_bf16(gelu_tanh(v1[2]), gelu_tanh(v1[3]));
    if (bad) w = (u32x4){0x7fc07fc0u, 0x7fc07fc0u, 0x7fc07fc0u, 0x7fc07fc0u};
}

__device__ __forceinline__ void gemm_r1(const Params& p, LAS unsigned char* lds, unsigned call, int pm, int pn) {
    const int tid = threadIdx.x, wid = __builtin_amdgcn_readfirstlane(tid >> 6), lane = tid & 63, wr = wid >> 2, wc = wid & 3, fr = lane & 15, fq = lane >> 4;
    const int rank = p.rank;
    unsigned char* wsl = p.ws[rank]; unsigned* fl = (unsigned*)wsl; unsigned* tmo = fl + F_TMO * 32;
    volatile LAS unsigned* lready = (volatile LAS unsigned*)(lds + STAGE_BYTES);
    const char* gA = (const char*)wsl + OFF_A + (size_t)pm * 2 * IMG + tid * 16;
    const char* gB = (const char*)wsl + OFF_B + (size_t)pn * 2 * IMG + tid * 16;
    const unsigned ldsw = (unsigned)wid * 1024u;
    const int aoff = lds_byte(wr * 64 + fr, fq * 8), boff = lds_byte(wc * 32 + fr, fq * 8);
    const unsigned* myflag; unsigned mytarget;
    { const int t = lane; if (t < 8) { myflag = fl + (F_SFLAG + rank * 8 + t) * 32; mytarget = call; }
      else { const int tt = t - 8, c = tt / 7, l = tt - c * 7, s = (rank + 1 + l) & 7; myflag = fl + (F_INCNT + s * 8 + c) * 32; mytarget = 4u * call; } }
    int ready = 0;
#define SA(b, h) (((b) * 2 + (h)) * HTB)
#define SB(b, h) ((4 + (b) * 2 + (h)) * HTB)
#define STAGE(bufoff, gptr) do { _Pragma("unroll") for (int _i = 0; _i < 2; ++_i) \
        __builtin_amdgcn_global_load_lds((const unsigned*)((gptr) + _i * 8192), (LAS unsigned*)(lds + (bufoff) + ldsw + _i * 8192), 16, 0, 0); } while (0)
#define LDA(dst, b, h) do { _Pragma("unroll") for (int m = 0; m < 4; ++m) _Pragma("unroll") for (int k = 0; k < 2; ++k) dst[m][k] = *(const LAS bf16x8*)(lds + SA(b, h) + aoff + m * 2048 + k * 1024); } while (0)
#define LDB(dst, b, h) do { _Pragma("unroll") for (int n = 0; n < 2; ++n) _Pragma("unroll") for (int k = 0; k < 2; ++k) dst[n][k] = *(const LAS bf16x8*)(lds + SB(b, h) + boff + n * 2048 + k * 1024); } while (0)
#define MMA(ai, bj, At, Bt) do { __builtin_amdgcn_s_setprio(1); _Pragma("unroll") for (int m = 0; m < 4; ++m) _Pragma("unroll") for (int n = 0; n < 2; ++n) _Pragma("unroll") for (int k = 0; k < 2; ++k) \
        acc[ai][bj][m][n] = __builtin_amdgcn_mfma_f32_16x16x32_bf16(Bt[n][k], At[m][k], acc[ai][bj][m][n], 0, 0, 0); __builtin_amdgcn_s_setprio(0); } while (0)
#define NEED(nd) do { if ((nd) > ready) { \
        if (wid == 0) { const unsigned long long t0 = __builtin_amdgcn_s_memrealtime(); int pre; \
            for (;;) { const unsigned long long bal = __ballot(ld_sys(myflag) >= mytarget); pre = (bal == ~0ull) ? 64 : (int)__builtin_ctzll(~bal); if (pre >= (nd)) break; \
                if (__builtin_amdgcn_readfirstlane(__hip_atomic_load(tmo, __ATOMIC_RELAXED, __HIP_MEMORY_SCOPE_AGENT)) == call || __builtin_amdgcn_s_memrealtime() - t0 > DEADLINE) { \
                    if (lane == 0) __hip_atomic_store(tmo, call, __ATOMIC_RELAXED, __HIP_MEMORY_SCOPE_AGENT); pre = 64; break; } \
                __builtin_amdgcn_s_sleep(2); } \
            __builtin_amdgcn_fence(__ATOMIC_ACQUIRE, "agent"); \
            if (lane == 0) *lready = (unsigned)pre; \
            asm volatile("s_waitcnt vmcnt(0) lgkmcnt(0)" ::: "memory"); } \
        CFENCE; BAR; CFENCE; \
        ready = __builtin_amdgcn_readfirstlane((int)*lready); } } while (0)

    f32x4 acc[2][2][4][2];
#pragma unroll
    for (int a = 0; a < 2; ++a)
#pragma unroll
        for (int b = 0; b < 2; ++b)
#pragma unroll
            for (int m = 0; m < 4; ++m)
#pragma unroll
                for (int n = 0; n < 2; ++n) acc[a][b][m][n] = (f32x4){0.f, 0.f, 0.f, 0.f};
    bf16x8 At[4][2], B0[2][2], B1[2][2];
    NEED(2);
    {
        const size_t k0 = (size_t)kt_of(0, rank) * SLAB, k1 = (size_t)kt_of(1, rank) * SLAB;
        STAGE(SB(0, 0), gB + k0); STAGE(SA(0, 0), gA + k0); STAGE(SB(0, 1), gB + k0 + IMG); STAGE(SA(0, 1), gA + k0 + IMG);
        if (wr == 1) BAR;
        WAIT_V(4); BAR;
        STAGE(SB(1, 0), gB + k1); STAGE(SA(1, 0), gA + k1); STAGE(SB(1, 1), gB + k1 + IMG);
        WAIT_V(6); BAR;
    }
    for (int t = 0; t < 64; t += 2) {
        const bool last = (t == 62);
        NEED(last ? 64 : t + 4);
        const size_t o1 = (size_t)kt_of(t + 1, rank) * SLAB;
        const size_t o2 = (size_t)kt_of(last ? 0 : t + 2, rank) * SLAB, o3 = (size_t)kt_of(last ? 1 : t + 3, rank) * SLAB;
        LDB(B0, 0, 0); SCHED; LDA(At, 0, 0); STAGE(SA(1, 1), gA + o1 + IMG);
        WAIT_L(8); BAR; WAIT_L(0); MMA(0, 0, At, B0); BAR; SCHED;
        LDB(B1, 0, 1); STAGE(SB(0, 0), gB + o2);
        BAR; WAIT_L(0); MMA(0, 1, At, B1); BAR;
        LDA(At, 0, 1); STAGE(SA(0, 0), gA + o2);
        BAR; WAIT_L(0); MMA(1, 0, At, B0); BAR; SCHED;
        STAGE(SB(0, 1), gB + o2 + IMG);
        WAIT_V(6); BAR; MMA(1, 1, At, B1); BAR;
        LDB(B0, 1, 0); SCHED; LDA(At, 1, 0); STAGE(SA(0, 1), gA + o2 + IMG);
        WAIT_L(8); BAR; WAIT_L(0); MMA(0, 0, At, B0); BAR; SCHED;
        LDB(B1, 1, 1); STAGE(SB(1, 0), gB + o3);
        BAR; WAIT_L(0); MMA(0, 1, At, B1); BAR;
        LDA(At, 1, 1); STAGE(SA(1, 0), gA + o3);
        BAR; WAIT_L(0); MMA(1, 0, At, B0); BAR; SCHED;
        STAGE(SB(1, 1), gB + o3 + IMG);
        WAIT_V(6); BAR; MMA(1, 1, At, B1); BAR;
    }
    {
        int tz = threadIdx.x; asm volatile("" : "+v"(tz));
        const int lane2 = tz & 63, fr2 = lane2 & 15, fq2 = lane2 >> 4;
        const bool bad = __hip_atomic_load(tmo, __ATOMIC_RELAXED, __HIP_MEMORY_SCOPE_AGENT) == call;
        const int owner = pm >> 1;
        if (owner == rank) {
            const auto rout = __builtin_amdgcn_make_buffer_rsrc(p.out, 0, (int)OUT_BYTES, 0x00020000);
            const unsigned base = (unsigned)((pm & 1) * 256 + wr * 64 + fr2) * 65536u + (unsigned)(rank * 4096 + pn * 256 + wc * 32 + fq2 * 8) * 2u;
#pragma unroll
            for (int ai = 0; ai < 2; ++ai)
#pragma unroll
                for (int m = 0; m < 4; ++m)
#pragma unroll
                    for (int bj = 0; bj < 2; ++bj) { u32x4 w; store_tile_rows(acc[ai][bj][m][0], acc[ai][bj][m][1], bad, w);
                        __builtin_amdgcn_raw_buffer_store_b128(w, rout, (int)(base + (unsigned)(ai * 128 + m * 16) * 65536u + (unsigned)bj * 256u), 0, 0); }
        } else {
            const auto rs = __builtin_amdgcn_make_buffer_rsrc(wsl, 0, (int)WS_NEED, 0x00020000);
            const unsigned base = OFF_O + (unsigned)(pm * 16 + pn) * 131072u + (unsigned)(wr * 64 + fr2) * 512u + (unsigned)(wc * 32 + fq2 * 8) * 2u;
#pragma unroll
            for (int ai = 0; ai < 2; ++ai)
#pragma unroll
                for (int m = 0; m < 4; ++m)
#pragma unroll
                    for (int bj = 0; bj < 2; ++bj) { u32x4 w; store_tile_rows(acc[ai][bj][m][0], acc[ai][bj][m][1], bad, w);
                        __builtin_amdgcn_raw_buffer_store_b128(w, rs, (int)(base + (unsigned)(ai * 128 + m * 16) * 512u + (unsigned)bj * 256u), 0, AUX_SYS); }
        }
        WAIT_V(0);
        if (wr == 0) BAR;
        BAR; CFENCE;
        if (owner != rank && tz == 0) {
            __builtin_amdgcn_fence(__ATOMIC_RELEASE, "");
            WAIT_V(0);
            st_sys((unsigned*)p.ws[owner] + (F_OUT + rank * 48 + (pm & 1) * 14 + pn) * 32, call);
        }
    }
#undef NEED
}

__device__ __forceinline__ void gemm_r2(const Params& p, LAS unsigned char* lds, unsigned call, int a, int b) {
    const int tid = threadIdx.x, wid = __builtin_amdgcn_readfirstlane(tid >> 6), lane = tid & 63, wr = wid >> 2, wc = wid & 3, fr = lane & 15, fq = lane >> 4;
    const int rank = p.rank;
    unsigned char* wsl = p.ws[rank]; unsigned* fl = (unsigned*)wsl; unsigned* tmo = fl + F_TMO * 32;
    if (tid < 7) wait_ge(fl + (F_INB + ((rank + 1 + tid) & 7)) * 32, 4u * call, tmo, call);
    if (wid == 0) { __builtin_amdgcn_fence(__ATOMIC_ACQUIRE, "agent"); WAIT_V(0); }
    CFENCE; BAR; CFENCE;
    const char* gA = (const char*)wsl + OFF_A + (size_t)a * IMG + tid * 16;
    const char* gB = (const char*)wsl + OFF_B + (size_t)b * IMG + tid * 16;
    const unsigned ldsw = (unsigned)wid * 1024u;
    const int aoff = lds_byte(wr * 64 + fr, fq * 8), boff = lds_byte(wc * 32 + fr, fq * 8);
    f32x4 acc[4][2];
#pragma unroll
    for (int m = 0; m < 4; ++m)
#pragma unroll
        for (int n = 0; n < 2; ++n) acc[m][n] = (f32x4){0.f, 0.f, 0.f, 0.f};
#define STAGE2(slot, kt) do { STAGE((slot) * 32768, gA + (size_t)(kt) * SLAB); STAGE((slot) * 32768 + 16384, gB + (size_t)(kt) * SLAB); } while (0)
    STAGE2(0, 0); STAGE2(1, 1); STAGE2(2, 2);
    for (int t = 0; t < 64; ++t) {
        WAIT_V(8); BAR; CFENCE;
        STAGE2((t + 3) & 3, (t + 3) & 63);
        const int so = (t & 3) * 32768;
        bf16x8 Af[4][2], Bf[2][2];
#pragma unroll
        for (int n = 0; n < 2; ++n)
#pragma unroll
            for (int k = 0; k < 2; ++k) Bf[n][k] = *(const LAS bf16x8*)(lds + so + 16384 + boff + n * 2048 + k * 1024);
#pragma unroll
        for (int m = 0; m < 4; ++m)
#pragma unroll
            for (int k = 0; k < 2; ++k) Af[m][k] = *(const LAS bf16x8*)(lds + so + aoff + m * 2048 + k * 1024);
#pragma unroll
        for (int m = 0; m < 4; ++m)
#pragma unroll
            for (int n = 0; n < 2; ++n)
#pragma unroll
                for (int k = 0; k < 2; ++k) acc[m][n] = __builtin_amdgcn_mfma_f32_16x16x32_bf16(Bf[n][k], Af[m][k], acc[m][n], 0, 0, 0);
        WAIT_L(0);
    }
    const bool bad = __hip_atomic_load(tmo, __ATOMIC_RELAXED, __HIP_MEMORY_SCOPE_AGENT) == call;
    const int owner = a >> 2;
    if (owner == rank) {
        const auto rout = __builtin_amdgcn_make_buffer_rsrc(p.out, 0, (int)OUT_BYTES, 0x00020000);
        const unsigned base = (unsigned)((a & 3) * 128 + wr * 64 + fr) * 65536u + (unsigned)(rank * 4096 + b * 128 + wc * 32 + fq * 8) * 2u;
#pragma unroll
        for (int m = 0; m < 4; ++m) { u32x4 w; store_tile_rows(acc[m][0], acc[m][1], bad, w);
            __builtin_amdgcn_raw_buffer_store_b128(w, rout, (int)(base + (unsigned)(m * 16) * 65536u), 0, 0); }
    } else {
        const auto rs = __builtin_amdgcn_make_buffer_rsrc(wsl, 0, (int)WS_NEED, 0x00020000);
        const unsigned base = OFF_O + (unsigned)((a >> 1) * 16 + (b >> 1)) * 131072u + (unsigned)((a & 1) * 2 + (b & 1)) * 32768u + (unsigned)(wr * 64 + fr) * 256u + (unsigned)(wc * 32 + fq * 8) * 2u;
#pragma unroll
        for (int m = 0; m < 4; ++m) { u32x4 w; store_tile_rows(acc[m][0], acc[m][1], bad, w);
            __builtin_amdgcn_raw_buffer_store_b128(w, rs, (int)(base + (unsigned)(m * 16) * 256u), 0, AUX_SYS); }
    }
    WAIT_V(0); BAR; CFENCE;
    if (owner != rank && tid == 0) {
        __builtin_amdgcn_fence(__ATOMIC_RELEASE, "");
        WAIT_V(0);
        st_sys((unsigned*)p.ws[owner] + (F_OUT + rank * 48 + 28 + (a & 3) * 4 + (b & 3)) * 32, call);
    }
}

__global__ __launch_bounds__(512, 2) void fused_gemm_a2a(Params p) {
    extern __shared__ __attribute__((aligned(16))) unsigned char shm[];
    LAS unsigned char* lds = (LAS unsigned char*)shm;
    const int tid = threadIdx.x, bid = blockIdx.x;
    unsigned* fl = (unsigned*)p.ws[p.rank];
    volatile LAS unsigned* lcall = (volatile LAS unsigned*)(lds + STAGE_BYTES + 16);
    if (tid == 0) *lcall = __hip_atomic_fetch_add(fl + F_TICKET * 32, 1u, __ATOMIC_RELAXED, __HIP_MEMORY_SCOPE_AGENT) / (unsigned)NWG + 1u;
    __syncthreads();
    const unsigned call = __builtin_amdgcn_readfirstlane(*lcall);
    if (bid < NPULL) { puller(p, call, bid % 7, bid / 7); return; }
    stage_units(p, lds, call, bid - NPULL);
    if (bid < NCOMM) return;
    __syncthreads();
    const int g = bid - NCOMM, xcd = g & 7, idx = g >> 3;
    gemm_r1(p, lds, call, (xcd >> 1) * 4 + (idx & 3), (xcd & 1) * 7 + (idx >> 2));
    if (idx < 16) { __syncthreads(); gemm_r2(p, lds, call, (xcd >> 1) * 8 + (idx & 7), 28 + (xcd & 1) * 2 + (idx >> 3)); }
}
}

extern "C" void kernel_launch(int rank, int nranks, ncclComm_t comm,
                              hipStream_t stream,
                              void* const* d_in, const int* in_sizes, int n_in,
                              void* d_out, int out_size,
                              const int* in_dtypes, int out_dtype,
                              void* const* d_out_all,
                              void* const* d_ws_all, size_t ws_bytes) {
    if (nranks != NR || n_in != 2 || ws_bytes < (size_t)WS_NEED || (unsigned)rank >= (unsigned)NR) { fprintf(stderr, "kernel_launch: unexpected configuration\n"); return; }
    Params p;
    p.x = (const unsigned short*)d_in[0]; p.w = (const unsigned short*)d_in[1]; p.out = (unsigned short*)d_out; p.rank = rank;
    for (int i = 0; i < NR; ++i) p.ws[i] = (unsigned char*)d_ws_all[i];
    HIP_CHECK(hipFuncSetAttribute((const void*)fused_gemm_a2a, hipFuncAttributeMaxDynamicSharedMemorySize, LDS_BYTES));
    hipLaunchKernelGGL(fused_gemm_a2a, dim3(NWG), dim3(512), LDS_BYTES, stream, p);
    HIP_CHECK(hipGetLastError());
}
```

```cpp
#include <hip/hip_runtime.h>
#include <rccl/rccl.h>
#include <cstdio>
#include <cstdint>

#define HIP_CHECK(expr) do { hipError_t _e = (expr); if (_e != hipSuccess) { \
    fprintf(stderr, "HIP error %s at %s:%d\n", hipGetErrorString(_e), __FILE__, __LINE__); \
    return; } } while (0)

namespace {
typedef unsigned u32x4 __attribute__((ext_vector_type(4)));
typedef short bf16x8 __attribute__((ext_vector_type(8)));
typedef float f32x4 __attribute__((ext_vector_type(4)));
#define LAS __attribute__((address_space(3)))

constexpr int NR = 8;
constexpr int MM = 4096, KS = 512, NN = 32768, NS = 4096, MS = 512;
constexpr unsigned IMG = 16384;
constexpr unsigned SLAB = 32 * IMG;
constexpr unsigned OFF_A = 1u << 20;
constexpr unsigned OFF_B = OFF_A + 64 * SLAB;
constexpr unsigned OFF_W = OFF_B + 64 * SLAB;
constexpr unsigned OFF_O = OFF_W + 64 * SLAB;
constexpr unsigned WS_NEED = OFF_O + 256u * 131072u;
constexpr unsigned OUT_BYTES = (unsigned)MS * NN * 2u;
constexpr int F_TICKET = 0, F_TMO = 1, F_SCNT = 2, F_SFLAG = 16, F_INCNT = 80, F_INB = 144, F_Q = 152, F_OUT = 160  , F_INO = 672, F_Q2 = 736, F_SUM = 744;
constexpr int NPULL = 28, NWG = 256;
constexpr int NSTAGER = NWG - NPULL;
constexpr int UNITS_PER_CHUNK = 72, NUNITS = 8 * UNITS_PER_CHUNK;
constexpr unsigned long long DEADLINE = 50000000ull;
constexpr int HTB = 16384, STAGE_BYTES = 8 * HTB, LDS_BYTES = STAGE_BYTES + 512;
constexpr int AUX_SYS = 17, AUX_AGENT = 16;
#define MON_SLEEP 8
#define ODD_FAST 6
#define SLOW_SLEEP 127
#define POLL_FAST_FROM 56
#define POLL_SLEEP 16
#define PULL_LD_AUX 17
#define PULL_ST_AUX 16

struct Params { const unsigned short* x; const unsigned short* w; unsigned char* ws[NR]; unsigned short* out; int rank; };

__device__ __forceinline__ int lds_byte(int r, int c) { const int st = (r >> 4) * 2 + (c >> 5), rr = r & 15, cc = c & 31, ob = rr * 64 + cc * 2; return st * 1024 + (ob ^ (((ob >> 9) & 1) << 5)); }
__device__ __forceinline__ void stage_rc(int b, int& R, int& C) { const int st = b / 1024, sb = b % 1024, swz = sb ^ (((sb >> 9) & 1) << 5); R = (st >> 1) * 16 + swz / 64; C = (st & 1) * 32 + (swz % 64) / 2; }

__device__ __forceinline__ unsigned ld_sys(const unsigned* p) { return __hip_atomic_load(p, __ATOMIC_RELAXED, __HIP_MEMORY_SCOPE_SYSTEM); }
__device__ __forceinline__ void st_sys(unsigned* p, unsigned v) { __hip_atomic_store(p, v, __ATOMIC_RELAXED, __HIP_MEMORY_SCOPE_SYSTEM); }
__device__ __forceinline__ void wait_ge(const unsigned* flag, unsigned val, unsigned* tmo, unsigned call) {
    const unsigned long long t0 = __builtin_amdgcn_s_memrealtime();
    while (ld_sys(flag) < val) {
        if (__hip_atomic_load(tmo, __ATOMIC_RELAXED, __HIP_MEMORY_SCOPE_AGENT) == call || __builtin_amdgcn_s_memrealtime() - t0 > DEADLINE) {
            __hip_atomic_store(tmo, call, __ATOMIC_RELAXED, __HIP_MEMORY_SCOPE_AGENT); break; }
        __builtin_amdgcn_s_sleep(2);
    }
}
__device__ __forceinline__ unsigned cvt_pk_bf16(float lo, float hi) { unsigned r; asm volatile("v_cvt_pk_bf16_f32 %0, %1, %2" : "=v"(r) : "v"(lo), "v"(hi)); return r; }
typedef float f32x2 __attribute__((ext_vector_type(2)));
__device__ __forceinline__ unsigned gelu2_bf16(float y0, float y1) {
    const f32x2 y = {y0, y1};
    const f32x2 t = (y * y) * -0.1029432395800235f + -2.302208198144325f;
    const f32x2 a = y * t;
    f32x2 d; d.x = __builtin_amdgcn_exp2f(a.x); d.y = __builtin_amdgcn_exp2f(a.y);
    d = d + 1.0f;
    f32x2 r; r.x = __builtin_amdgcn_rcpf(d.x); r.y = __builtin_amdgcn_rcpf(d.y);
    r = r * y;
    return cvt_pk_bf16(r.x, r.y);
}
#define STAMP(i) do { } while (0)
#define WAIT_V(n) asm volatile("s_waitcnt vmcnt(" #n ")" ::: "memory")
#define WAIT_L(n) asm volatile("s_waitcnt lgkmcnt(" #n ")" ::: "memory")
#define BAR __builtin_amdgcn_s_barrier()
#define SCHED __builtin_amdgcn_sched_barrier(0)
#define CFENCE asm volatile("" ::: "memory")

__device__ __forceinline__ void stage_units(const Params& p, LAS unsigned char* lds, unsigned call, int first) {
    const int tid = threadIdx.x, rank = p.rank;
    unsigned char* wsl = p.ws[rank]; unsigned* fl = (unsigned*)wsl;
    const auto rs = __builtin_amdgcn_make_buffer_rsrc(wsl, 0, (int)WS_NEED, 0x00020000);
    for (int u = first; u < NUNITS; u += NSTAGER) {
        const int c = u / UNITS_PER_CHUNK, v = u % UNITS_PER_CHUNK;
        if (v >= 64) {
            const int vv = v - 64;
            const unsigned dbase = OFF_A + (unsigned)(rank * 8 + c) * SLAB + (unsigned)vv * 4u * IMG;
            u32x4 t[8];
#pragma unroll
            for (int i = 0; i < 8; ++i) { const int pc = tid + 512 * i, img = pc >> 10; int R, C; stage_rc((pc & 1023) * 16, R, C);
                t[i] = *(const u32x4*)(p.x + (size_t)(vv * 512 + img * 128 + R) * KS + c * 64 + C); }
#pragma unroll
            for (int i = 0; i < 8; ++i) __builtin_amdgcn_raw_buffer_store_b128(t[i], rs, (int)(dbase + (unsigned)(tid + 512 * i) * 16u), 0, AUX_SYS);
        } else {
            const int d = v >> 3, kb = tid >> 6, nb = tid & 63;
            const unsigned dbase = (d == rank ? OFF_B + (unsigned)(rank * 8 + c) * SLAB : OFF_W + (unsigned)(d * 8 + c) * SLAB) + (unsigned)(v & 7) * 4u * IMG;
            u32x4 r[8];
#pragma unroll
            for (int i = 0; i < 8; ++i) r[i] = *(const u32x4*)(p.w + (size_t)(c * 64 + kb * 8 + i) * NN + v * 512 + nb * 8);
#pragma unroll
            for (int j = 0; j < 8; ++j) {
                u32x4 o;
#pragma unroll
                for (int e = 0; e < 4; ++e) o[e] = __builtin_amdgcn_perm(r[2 * e + 1][j >> 1], r[2 * e][j >> 1], (j & 1) ? 0x07060302u : 0x05040100u);
                const int nl = nb * 8 + j, img = nl >> 7, rowi = nl & 127, cc = rowi & 31;
                const int rho = (rowi & ~31) + 16 * ((cc >> 2) & 1) + 4 * (cc >> 3) + (cc & 3);
                *(LAS u32x4*)(lds + img * IMG + lds_byte(rho, kb * 8)) = o;
            }
            __syncthreads();
#pragma unroll
            for (int i = 0; i < 8; ++i) { const unsigned b = (unsigned)(tid + 512 * i) * 16u; const u32x4 t = *(const LAS u32x4*)(lds + b);
                __builtin_amdgcn_raw_buffer_store_b128(t, rs, (int)(dbase + b), 0, AUX_SYS); }
        }
        WAIT_V(0);
        __syncthreads();
        if (tid == 0) {
            const unsigned old = __hip_atomic_fetch_add(fl + (F_SCNT + c) * 32, 1u, __ATOMIC_RELAXED, __HIP_MEMORY_SCOPE_AGENT);
            if (old + 1u == call * (unsigned)UNITS_PER_CHUNK) {
                __builtin_amdgcn_fence(__ATOMIC_RELEASE, "");
                WAIT_V(0);
#pragma unroll
                for (int d = 0; d < NR; ++d) st_sys((unsigned*)p.ws[d] + (F_SFLAG + rank * 8 + c) * 32, call);
            }
        }
    }
}

struct Desc { unsigned soff, doff; int kind; int flag; int sig; };
constexpr int NITEM = 1536, NFETCH = NITEM + 32;
#define ND 4
constexpr int I_E0 = (8 - ND) * 120, I_O0 = I_E0 + ND * 88;
constexpr unsigned INE_ITEMS = 88, INO_ITEMS = 32, INB_ITEMS = 64;
__device__ __forceinline__ Desc puller_desc(int it, int rank, int s) {
    Desc d; d.sig = -1; d.kind = 0;
    if (it < 960) {
        int c, j;
        if (it < I_E0) { c = it / 120; j = it - c * 120; }
        else if (it < I_O0) { c = (8 - ND) + (it - I_E0) / 88; j = (it - I_E0) - (c - (8 - ND)) * 88; j = j < 32 ? ((j >> 2) << 3) + (j & 3) : j + 32; }
        else { c = (8 - ND) + ((it - I_O0) >> 5); j = (it - I_O0) & 31; j = ((j >> 2) << 3) + 4 + (j & 3); }
        d.flag = F_SFLAG + s * 8 + c; d.sig = ((j < 64 && (j & 4)) ? F_INO : F_INCNT) + s * 8 + c;
        if (j < 64) { d.soff = OFF_A + (unsigned)(8 * s + c) * SLAB + (unsigned)j * 8192u; d.doff = d.soff; }
        else { const unsigned off = (unsigned)(j - 64) * 8192u; d.soff = OFF_W + (unsigned)(rank * 8 + c) * SLAB + off; d.doff = OFF_B + (unsigned)(8 * s + c) * SLAB + off; }
    } else if (it < 1024) {
        const int c = (it - 960) >> 3, j = (it - 960) & 7; const unsigned off = 28u * IMG + (unsigned)j * 8192u;
        d.flag = F_SFLAG + s * 8 + c; d.sig = F_INB + s;
        d.soff = OFF_W + (unsigned)(rank * 8 + c) * SLAB + off; d.doff = OFF_B + (unsigned)(8 * s + c) * SLAB + off;
    } else if (it < 1472) {
        const int idx = (it - 1024) >> 4, j = (it - 1024) & 15, pml = idx / 14, pn = idx - pml * 14;
        d.kind = 1; d.flag = F_OUT + s * 64 + idx;
        d.soff = OFF_O + (unsigned)((2 * rank + pml) * 16 + pn) * 131072u + (unsigned)j * 8192u;
        d.doff = (unsigned)(pml * 256 + j * 16) * 65536u + (unsigned)(s * 4096 + pn * 256) * 2u;
    } else {
        const int idx = (it - 1472) >> 2, j = (it - 1472) & 3, al = idx >> 2, b = 28 + (idx & 3);
        d.kind = 2; d.flag = F_OUT + s * 64 + 28 + idx;
        d.soff = OFF_O + (unsigned)((2 * rank + (al >> 1)) * 16 + (b >> 1)) * 131072u + (unsigned)((al & 1) * 2 + (b & 1)) * 32768u + (unsigned)j * 8192u;
        d.doff = (unsigned)(al * 128 + j * 32) * 65536u + (unsigned)(s * 4096 + b * 128) * 2u;
    }
    return d;
}
typedef int i32x4 __attribute__((ext_vector_type(4)));
__device__ __forceinline__ i32x4 mk_rsrc(const void* ptr, unsigned bytes) {
    const unsigned long long a = (unsigned long long)ptr; i32x4 r;
    r.x = __builtin_amdgcn_readfirstlane((int)(unsigned)a); r.y = __builtin_amdgcn_readfirstlane((int)(unsigned)(a >> 32) & 0xffff); r.z = (int)bytes; r.w = 0x00020000; return r;
}
__device__ __forceinline__ void puller(const Params& p, unsigned call, int l) {
    const int lane = threadIdx.x & 63, rank = p.rank, s = (rank + 1 + l) & 7;
    unsigned* fl = (unsigned*)p.ws[rank]; unsigned* tmo = fl + F_TMO * 32; unsigned* qctr = fl + (F_Q + l) * 32;
    const unsigned qbase = (call - 1u) * (unsigned)NFETCH;
    const i32x4 rsrc = mk_rsrc(p.ws[s], WS_NEED), rdst = mk_rsrc(p.ws[rank], WS_NEED), rout = mk_rsrc(p.out, OUT_BYTES);
    const unsigned l0 = (unsigned)lane * 16u, l1 = (unsigned)(lane >> 5) * 65536u + (unsigned)(lane & 31) * 16u, l2 = (unsigned)(lane >> 4) * 65536u + (unsigned)(lane & 15) * 16u;
    u32x4 v[4];
    int cur, nxt;
    { unsigned t = 0; if (lane == 0) t = __hip_atomic_fetch_add(qctr, 1u, __ATOMIC_RELAXED, __HIP_MEMORY_SCOPE_AGENT); cur = (int)(__builtin_amdgcn_readfirstlane(t) - qbase); nxt = NITEM;
      if (cur < NITEM) { if (lane == 0) t = __hip_atomic_fetch_add(qctr, 1u, __ATOMIC_RELAXED, __HIP_MEMORY_SCOPE_AGENT); nxt = (int)(__builtin_amdgcn_readfirstlane(t) - qbase); } }
    bool cur_ready = false, have_prev = false, pbad = false, bad = false; int lastph = -1; (void)lastph;
    unsigned pdoff = 0, plane = 0, pstep = 0; int pkind = 0, psig = -1;
#define P_STORE(r, j, doff_, kind_, lane_, step_, bad_) do { \
        if ((kind_) == 0) { asm volatile("buffer_store_dwordx4 %0, %1, %2, 0 offen sc1\n\ts_nop 1" :: "v"(v[r]), "v"((doff_) + (lane_) + (unsigned)(j) * (step_)), "s"(rdst) : "memory"); } \
        else { u32x4 t_ = v[r]; if (bad_) t_ = (u32x4){0x7fc07fc0u, 0x7fc07fc0u, 0x7fc07fc0u, 0x7fc07fc0u}; \
            asm volatile("buffer_store_dwordx4 %0, %1, %2, 0 offen\n\ts_nop 1" :: "v"(t_), "v"((doff_) + (lane_) + (unsigned)(j) * (step_)), "s"(rout) : "memory"); } } while (0)
#define P_FLUSH() do { if (have_prev) { WAIT_V(0); _Pragma("unroll") for (int r = 0; r < 4; ++r) { asm volatile("" : "+v"(v[r])); P_STORE(r, 4 + r, pdoff, pkind, plane, pstep, pbad); } \
            WAIT_V(0); if (psig >= 0 && lane == 0) __hip_atomic_fetch_add(fl + psig * 32, 1u, __ATOMIC_RELAXED, __HIP_MEMORY_SCOPE_AGENT); have_prev = false; } } while (0)
    while (cur < NITEM) {
        const Desc dc = puller_desc(cur, rank, s);
        if (!cur_ready) {
            P_FLUSH();
            if (lane == 0) wait_ge(fl + dc.flag * 32, call, tmo, call);
            bad = __builtin_amdgcn_readfirstlane(__hip_atomic_load(tmo, __ATOMIC_RELAXED, __HIP_MEMORY_SCOPE_AGENT)) == call;
        }
        unsigned fv = 0, tk = 0;
        if (nxt < NITEM) {
            const Desc dn = puller_desc(nxt, rank, s);
            asm volatile("global_load_dword %0, %1, off sc0 sc1" : "=v"(fv) : "v"(fl + dn.flag * 32) : "memory");
            if (lane == 0) asm volatile("global_atomic_add %0, %1, %2, off sc0" : "=v"(tk) : "v"(qctr), "v"(1u) : "memory");
        }
        const unsigned clane = dc.kind == 0 ? l0 : (dc.kind == 1 ? l1 : l2), cstep = dc.kind == 0 ? 1024u : (dc.kind == 1 ? 131072u : 262144u);
#pragma unroll
        for (int j = 0; j < 8; ++j) {
            const int r = j & 3;
            if (j >= 4 || have_prev) {
                asm volatile("s_waitcnt vmcnt(3)" : "+v"(v[r]) :: "memory");
                if (j >= 4) P_STORE(r, j - 4, dc.doff, dc.kind, clane, cstep, bad); else P_STORE(r, 4 + j, pdoff, pkind, plane, pstep, pbad);
            }
            asm volatile("buffer_load_dwordx4 %0, %1, %2, 0 offen sc0 sc1" : "=v"(v[r]) : "v"(dc.soff + l0 + (unsigned)j * 1024u), "s"(rsrc) : "memory");
        }
        if (have_prev && psig >= 0 && lane == 0) __hip_atomic_fetch_add(fl + psig * 32, 1u, __ATOMIC_RELAXED, __HIP_MEMORY_SCOPE_AGENT);
        asm volatile("" : "+v"(fv), "+v"(tk));
        have_prev = true; pdoff = dc.doff; pkind = dc.kind; plane = clane; pstep = cstep; psig = dc.sig; pbad = bad;
        cur_ready = (unsigned)__builtin_amdgcn_readfirstlane(fv) >= call;
        const int nn = nxt < NITEM ? (int)(__builtin_amdgcn_readfirstlane(tk) - qbase) : NITEM;
        if ((cur >= I_O0 - 32 && cur < I_O0) || (cur >= 960 - 32 && cur < 960) || (cur >= 1024 - 32 && cur < 1024)) P_FLUSH();
        cur = nxt; nxt = nn;
    }
    P_FLUSH();
}

__device__ __forceinline__ unsigned long long ld_sum(const unsigned* fl, int w, unsigned call) {
    const unsigned long long v = __hip_atomic_load((const unsigned long long*)(fl + F_SUM * 32) + w, __ATOMIC_RELAXED, __HIP_MEMORY_SCOPE_AGENT);
    return (unsigned)(v >> 56) == (call & 0xffu) ? (v & 0x00ffffffffffffffull) : 0ull;
}
__device__ __forceinline__ void monitor(const Params& p, unsigned call) {
    const int lane = threadIdx.x & 63, rank = p.rank;
    unsigned* fl = (unsigned*)p.ws[rank]; unsigned* tmo = fl + F_TMO * 32;
    unsigned long long* sum = (unsigned long long*)(fl + F_SUM * 32);
    const int c = lane / 7, l = lane - c * 7, s = (rank + 1 + l) & 7;
    const unsigned* pe = fl + (F_INCNT + s * 8 + (c & 7)) * 32; const unsigned* po = fl + (F_INO + s * 8 + (c & 7)) * 32; const unsigned* pb = fl + (F_INB + s) * 32;
    const unsigned long long tag = (unsigned long long)(call & 0xffu) << 56, full = 0x00ffffffffffffffull;
    unsigned long long me = 0, mo = 0, mb = 0;
    const unsigned long long t0 = __builtin_amdgcn_s_memrealtime();
    for (;;) {
        const int fe = (int)__builtin_ctzll(~me), fo = (int)__builtin_ctzll(~mo);
        const bool ae = lane < 56 && !((me >> lane) & 1ull) && lane < fe + 14, ao = lane < 56 && !((mo >> lane) & 1ull) && lane < fo + 14 && ((me >> lane) & 1ull);
        const bool ab = lane < 7 && !((mb >> lane) & 1ull) && me == full;
        const bool be = ae && ld_sys(pe) >= INE_ITEMS * call, bo = ao && ld_sys(po) >= INO_ITEMS * call, bb = ab && ld_sys(pb) >= INB_ITEMS * call;
        const unsigned long long ne = me | __ballot(be), no = mo | __ballot(bo), nb = mb | __ballot(bb);
        if (lane == 0) {
            if (ne != me) __hip_atomic_store(sum + 0, ne | tag, __ATOMIC_RELAXED, __HIP_MEMORY_SCOPE_AGENT);
            if (no != mo) __hip_atomic_store(sum + 1, no | tag, __ATOMIC_RELAXED, __HIP_MEMORY_SCOPE_AGENT);
            if (nb != mb) __hip_atomic_store(sum + 2, nb | tag, __ATOMIC_RELAXED, __HIP_MEMORY_SCOPE_AGENT);
        }
        me = ne; mo = no; mb = nb;
        if (me == full && mo == full && mb == 0x7full) break;
        if (__builtin_amdgcn_readfirstlane(__hip_atomic_load(tmo, __ATOMIC_RELAXED, __HIP_MEMORY_SCOPE_AGENT)) == call || __builtin_amdgcn_s_memrealtime() - t0 > DEADLINE) {
            if (lane == 0) __hip_atomic_store(tmo, call, __ATOMIC_RELAXED, __HIP_MEMORY_SCOPE_AGENT); break; }
        __builtin_amdgcn_s_sleep(MON_SLEEP);
    }
}

__device__ __forceinline__ int kt_of(int t, int rank) {
    if (t < 8) return rank * 8 + t;
    const int tt = t - 8, c = tt / 7, l = tt - c * 7; return ((rank + 1 + l) & 7) * 8 + c;
}
__device__ __forceinline__ void store_tile_rows(const f32x4& v0, const f32x4& v1, bool bad, u32x4& w) {
    w.x = gelu2_bf16(v0[0], v0[1]); w.y = gelu2_bf16(v0[2], v0[3]); w.z = gelu2_bf16(v1[0], v1[1]); w.w = gelu2_bf16(v1[2], v1[3]);
    if (bad) w = (u32x4){0x7fc07fc0u, 0x7fc07fc0u, 0x7fc07fc0u, 0x7fc07fc0u};
}

__device__ __forceinline__ void gemm_r1(const Params& p, LAS unsigned char* lds, unsigned call, int pm, int pn) {
    const int tid = threadIdx.x, wid = __builtin_amdgcn_readfirstlane(tid >> 6), lane = tid & 63, wr = wid >> 2, wc = wid & 3, fr = lane & 15, fq = lane >> 4;
    const int rank = p.rank;
    unsigned char* wsl = p.ws[rank]; unsigned* fl = (unsigned*)wsl; unsigned* tmo = fl + F_TMO * 32;
    volatile LAS unsigned* lready = (volatile LAS unsigned*)(lds + STAGE_BYTES);
    const char* gA = (const char*)wsl + OFF_A + (size_t)pm * 2 * IMG + tid * 16;
    const char* gB = (const char*)wsl + OFF_B + (size_t)pn * 2 * IMG + tid * 16;
    const unsigned ldsw = (unsigned)wid * 1024u;
    const int aoff = lds_byte(wr * 64 + fr, fq * 8), boff = lds_byte(wc * 32 + fr, fq * 8);
    const unsigned* myflag = fl + (F_SFLAG + rank * 8 + (lane & 7)) * 32;
    int ready = 0;
#define SA(b, h) (((b) * 2 + (h)) * HTB)
#define SB(b, h) ((4 + (b) * 2 + (h)) * HTB)
#define STAGE(bufoff, gptr) do { _Pragma("unroll") for (int _i = 0; _i < 2; ++_i) \
        __builtin_amdgcn_global_load_lds((const unsigned*)((gptr) + _i * 8192), (LAS unsigned*)(lds + (bufoff) + ldsw + _i * 8192), 16, 0, 0); } while (0)
#define LDA(dst, b, h) do { _Pragma("unroll") for (int m = 0; m < 4; ++m) _Pragma("unroll") for (int k = 0; k < 2; ++k) dst[m][k] = *(const LAS bf16x8*)(lds + SA(b, h) + aoff + m * 2048 + k * 1024); } while (0)
#define LDB(dst, b, h) do { _Pragma("unroll") for (int n = 0; n < 2; ++n) _Pragma("unroll") for (int k = 0; k < 2; ++k) dst[n][k] = *(const LAS bf16x8*)(lds + SB(b, h) + boff + n * 2048 + k * 1024); } while (0)
#define MMA(ai, bj, At, Bt) do { __builtin_amdgcn_s_setprio(1); _Pragma("unroll") for (int m = 0; m < 4; ++m) _Pragma("unroll") for (int n = 0; n < 2; ++n) _Pragma("unroll") for (int k = 0; k < 2; ++k) \
        acc[ai][bj][m][n] = __builtin_amdgcn_mfma_f32_16x16x32_bf16(Bt[n][k], At[m][k], acc[ai][bj][m][n], 0, 0, 0); __builtin_amdgcn_s_setprio(0); } while (0)
    volatile LAS int* lorder = (volatile LAS int*)(lds + STAGE_BYTES + 64);
    unsigned long long sched = 0ull;
    const int mykt = kt_of(lane, rank);
#define NEED(nd) do { if ((nd) > ready) { \
        if (wid == 0) { const unsigned long long t0 = __builtin_amdgcn_s_memrealtime(); int cnt = ready; \
            for (;;) { \
                const unsigned long long sm = ld_sum(fl, pm & 1, call) << 8; \
                bool ok = !((sched >> lane) & 1ull) && (lane < 8 ? ld_sys(myflag) >= call : ((sm >> lane) & 1ull) != 0ull); \
                bool giveup = false; \
                if (cnt + __builtin_popcountll(__ballot(ok)) < (nd)) { \
                    giveup = __builtin_amdgcn_readfirstlane(__hip_atomic_load(tmo, __ATOMIC_RELAXED, __HIP_MEMORY_SCOPE_AGENT)) == call || __builtin_amdgcn_s_memrealtime() - t0 > DEADLINE; \
                    if (giveup) { if (lane == 0) __hip_atomic_store(tmo, call, __ATOMIC_RELAXED, __HIP_MEMORY_SCOPE_AGENT); ok = !((sched >> lane) & 1ull); } } \
                const unsigned long long bal = __ballot(ok); \
                if (ok) lorder[cnt + __builtin_popcountll(bal & ((1ull << lane) - 1ull))] = mykt; \
                sched |= bal; cnt += __builtin_popcountll(bal); \
                if (cnt >= (nd)) break; \
                if ((nd) > 10 && (nd) < ((pm & 1) ? 64 - 7 * ND + ODD_FAST : POLL_FAST_FROM)) __builtin_amdgcn_s_sleep(SLOW_SLEEP); else __builtin_amdgcn_s_sleep(POLL_SLEEP); } \
            __builtin_amdgcn_fence(__ATOMIC_ACQUIRE, "agent"); \
            if (lane == 0) *lready = (unsigned)cnt; \
            asm volatile("s_waitcnt vmcnt(0) lgkmcnt(0)" ::: "memory"); } \
        CFENCE; BAR; CFENCE; \
        ready = __builtin_amdgcn_readfirstlane((int)*lready); } } while (0)
#define KOFF(t) ((size_t)(unsigned)__builtin_amdgcn_readfirstlane(lorder[t]) * SLAB)

    f32x4 acc[2][2][4][2];
#pragma unroll
    for (int a = 0; a < 2; ++a)
#pragma unroll
        for (int b = 0; b < 2; ++b)
#pragma unroll
            for (int m = 0; m < 4; ++m)
#pragma unroll
                for (int n = 0; n < 2; ++n) acc[a][b][m][n] = (f32x4){0.f, 0.f, 0.f, 0.f};
    bf16x8 At[4][2], B0[2][2], B1[2][2];
    NEED(4);
    size_t o1, o2, o3;
    {
        const size_t k0 = KOFF(0); o1 = KOFF(1); o2 = KOFF(2); o3 = KOFF(3);
        STAGE(SB(0, 0), gB + k0); STAGE(SA(0, 0), gA + k0); STAGE(SB(0, 1), gB + k0 + IMG); STAGE(SA(0, 1), gA + k0 + IMG);
        if (wr == 1) BAR;
        WAIT_V(4); BAR;
        STAGE(SB(1, 0), gB + o1); STAGE(SA(1, 0), gA + o1); STAGE(SB(1, 1), gB + o1 + IMG);
        WAIT_V(6); BAR;
    }
    STAMP(2);
    for (int t = 0; t < 64; t += 2) {
        LDB(B0, 0, 0); SCHED; LDA(At, 0, 0); STAGE(SA(1, 1), gA + o1 + IMG);
        WAIT_L(8); BAR; WAIT_L(0); MMA(0, 0, At, B0); BAR; SCHED;
        LDB(B1, 0, 1); STAGE(SB(0, 0), gB + o2);
        BAR; WAIT_L(0); MMA(0, 1, At, B1); BAR;
        LDA(At, 0, 1); STAGE(SA(0, 0), gA + o2);
        BAR; WAIT_L(0); MMA(1, 0, At, B0); BAR; SCHED;
        STAGE(SB(0, 1), gB + o2 + IMG);
        WAIT_V(6); BAR; MMA(1, 1, At, B1); BAR;
        LDB(B0, 1, 0); SCHED; LDA(At, 1, 0); STAGE(SA(0, 1), gA + o2 + IMG);
        WAIT_L(8); BAR; WAIT_L(0); MMA(0, 0, At, B0); BAR; SCHED;
        LDB(B1, 1, 1); STAGE(SB(1, 0), gB + o3);
        BAR; WAIT_L(0); MMA(0, 1, At, B1); BAR;
        LDA(At, 1, 1); STAGE(SA(1, 0), gA + o3);
        BAR; WAIT_L(0); MMA(1, 0, At, B0); BAR; SCHED;
        STAGE(SB(1, 1), gB + o3 + IMG);
        WAIT_V(6); BAR; MMA(1, 1, At, B1); BAR;
        o1 = o3;
        if (t + 6 <= 64) { NEED(t + 6); o2 = KOFF(t + 4); o3 = KOFF(t + 5); }
    }
    STAMP(3);
    {
        int tz = threadIdx.x; asm volatile("" : "+v"(tz));
        const int lane2 = tz & 63, fr2 = lane2 & 15, fq2 = lane2 >> 4;
        const bool bad = __hip_atomic_load(tmo, __ATOMIC_RELAXED, __HIP_MEMORY_SCOPE_AGENT) == call;
        const int owner = pm >> 1;
        if (owner == rank) {
            const auto rout = __builtin_amdgcn_make_buffer_rsrc(p.out, 0, (int)OUT_BYTES, 0x00020000);
            const unsigned base = (unsigned)((pm & 1) * 256 + wr * 64 + fr2) * 65536u + (unsigned)(rank * 4096 + pn * 256 + wc * 32 + fq2 * 8) * 2u;
#pragma unroll
            for (int ai = 0; ai < 2; ++ai)
#pragma unroll
                for (int m = 0; m < 4; ++m)
#pragma unroll
                    for (int bj = 0; bj < 2; ++bj) { u32x4 w; store_tile_rows(acc[ai][bj][m][0], acc[ai][bj][m][1], bad, w);
                        __builtin_amdgcn_raw_buffer_store_b128(w, rout, (int)(base + (unsigned)(ai * 128 + m * 16) * 65536u + (unsigned)bj * 256u), 0, 0); }
        } else {
            const auto rs = __builtin_amdgcn_make_buffer_rsrc(wsl, 0, (int)WS_NEED, 0x00020000);
            const unsigned base = OFF_O + (unsigned)(pm * 16 + pn) * 131072u + (unsigned)(wr * 64 + fr2) * 512u + (unsigned)(wc * 32 + fq2 * 8) * 2u;
#pragma unroll
            for (int ai = 0; ai < 2; ++ai)
#pragma unroll
                for (int m = 0; m < 4; ++m)
#pragma unroll
                    for (int bj = 0; bj < 2; ++bj) { u32x4 w; store_tile_rows(acc[ai][bj][m][0], acc[ai][bj][m][1], bad, w);
                        __builtin_amdgcn_raw_buffer_store_b128(w, rs, (int)(base + (unsigned)(ai * 128 + m * 16) * 512u + (unsigned)bj * 256u), 0, AUX_SYS); }
        }
        WAIT_V(0);
        if (wr == 0) BAR;
        BAR; CFENCE;
        if (owner != rank && tz == 0) {
            __builtin_amdgcn_fence(__ATOMIC_RELEASE, "");
            WAIT_V(0);
            st_sys((unsigned*)p.ws[owner] + (F_OUT + rank * 64 + (pm & 1) * 14 + pn) * 32, call);
        }
    }
#undef NEED
}

__device__ __forceinline__ void gemm_r2(const Params& p, LAS unsigned char* lds, unsigned call, int a, int b) {
    const int tid = threadIdx.x, wid = __builtin_amdgcn_readfirstlane(tid >> 6), lane = tid & 63, wr = wid >> 2, wc = wid & 3, fr = lane & 15, fq = lane >> 4;
    const int rank = p.rank;
    unsigned char* wsl = p.ws[rank]; unsigned* fl = (unsigned*)wsl; unsigned* tmo = fl + F_TMO * 32;
    if (tid < 64) {
        if (lane < 8) wait_ge(fl + (F_SFLAG + rank * 8 + lane) * 32, call, tmo, call);
        const unsigned long long t0 = __builtin_amdgcn_s_memrealtime();
        while (ld_sum(fl, (a >> 1) & 1, call) != 0x00ffffffffffffffull || ld_sum(fl, 2, call) != 0x7full) {
            if (__builtin_amdgcn_readfirstlane(__hip_atomic_load(tmo, __ATOMIC_RELAXED, __HIP_MEMORY_SCOPE_AGENT)) == call || __builtin_amdgcn_s_memrealtime() - t0 > DEADLINE) {
                if (lane == 0) __hip_atomic_store(tmo, call, __ATOMIC_RELAXED, __HIP_MEMORY_SCOPE_AGENT); break; }
            __builtin_amdgcn_s_sleep(16);
        }
        __builtin_amdgcn_fence(__ATOMIC_ACQUIRE, "agent"); WAIT_V(0);
    }
    CFENCE; BAR; CFENCE;
    const char* gA = (const char*)wsl + OFF_A + (size_t)a * IMG + tid * 16;
    const char* gB = (const char*)wsl + OFF_B + (size_t)b * IMG + tid * 16;
    const unsigned ldsw = (unsigned)wid * 1024u;
    const int aoff = lds_byte(wr * 64 + fr, fq * 8), boff = lds_byte(wc * 32 + fr, fq * 8);
    f32x4 acc[4][2];
#pragma unroll
    for (int m = 0; m < 4; ++m)
#pragma unroll
        for (int n = 0; n < 2; ++n) acc[m][n] = (f32x4){0.f, 0.f, 0.f, 0.f};
#define STAGE2(slot, kt) do { STAGE((slot) * 32768, gA + (size_t)(kt) * SLAB); STAGE((slot) * 32768 + 16384, gB + (size_t)(kt) * SLAB); } while (0)
#define FRAG2(Af, Bf, slot) do { \
        _Pragma("unroll") for (int n = 0; n < 2; ++n) _Pragma("unroll") for (int k = 0; k < 2; ++k) Bf[n][k] = *(const LAS bf16x8*)(lds + (slot) * 32768 + 16384 + boff + n * 2048 + k * 1024); \
        _Pragma("unroll") for (int m = 0; m < 4; ++m) _Pragma("unroll") for (int k = 0; k < 2; ++k) Af[m][k] = *(const LAS bf16x8*)(lds + (slot) * 32768 + aoff + m * 2048 + k * 1024); } while (0)
#define MMA2(Af, Bf) do { __builtin_amdgcn_s_setprio(1); _Pragma("unroll") for (int k = 0; k < 2; ++k) _Pragma("unroll") for (int m = 0; m < 4; ++m) _Pragma("unroll") for (int n = 0; n < 2; ++n) \
        acc[m][n] = __builtin_amdgcn_mfma_f32_16x16x32_bf16(Bf[n][k], Af[m][k], acc[m][n], 0, 0, 0); __builtin_amdgcn_s_setprio(0); } while (0)
    bf16x8 A0[4][2], Bf0[2][2], A1[4][2], Bf1[2][2];
    STAMP(6);
    STAGE2(0, 0); STAGE2(1, 1); STAGE2(2, 2); STAGE2(3, 3);
    WAIT_V(12); BAR; CFENCE;
    FRAG2(A0, Bf0, 0);
    for (int t = 0; t < 64; t += 4) {
        __builtin_amdgcn_s_waitcnt(0xC07F); WAIT_V(8); BAR; SCHED; STAGE2(0, (t + 4) & 63); FRAG2(A1, Bf1, 1); SCHED; MMA2(A0, Bf0); SCHED;
        __builtin_amdgcn_s_waitcnt(0xC07F); WAIT_V(8); BAR; SCHED; STAGE2(1, (t + 5) & 63); FRAG2(A0, Bf0, 2); SCHED; MMA2(A1, Bf1); SCHED;
        __builtin_amdgcn_s_waitcnt(0xC07F); WAIT_V(8); BAR; SCHED; STAGE2(2, (t + 6) & 63); FRAG2(A1, Bf1, 3); SCHED; MMA2(A0, Bf0); SCHED;
        __builtin_amdgcn_s_waitcnt(0xC07F); WAIT_V(8); BAR; SCHED; STAGE2(3, (t + 7) & 63); FRAG2(A0, Bf0, 0); SCHED; MMA2(A1, Bf1); SCHED;
    }
    WAIT_L(0);
    STAMP(7);
    const bool bad = __hip_atomic_load(tmo, __ATOMIC_RELAXED, __HIP_MEMORY_SCOPE_AGENT) == call;
    const int owner = a >> 2;
    if (owner == rank) {
        const auto rout = __builtin_amdgcn_make_buffer_rsrc(p.out, 0, (int)OUT_BYTES, 0x00020000);
        const unsigned base = (unsigned)((a & 3) * 128 + wr * 64 + fr) * 65536u + (unsigned)(rank * 4096 + b * 128 + wc * 32 + fq * 8) * 2u;
#pragma unroll
        for (int m = 0; m < 4; ++m) { u32x4 w; store_tile_rows(acc[m][0], acc[m][1], bad, w);
            __builtin_amdgcn_raw_buffer_store_b128(w, rout, (int)(base + (unsigned)(m * 16) * 65536u), 0, 0); }
    } else {
        const auto rs = __builtin_amdgcn_make_buffer_rsrc(wsl, 0, (int)WS_NEED, 0x00020000);
        const unsigned base = OFF_O + (unsigned)((a >> 1) * 16 + (b >> 1)) * 131072u + (unsigned)((a & 1) * 2 + (b & 1)) * 32768u + (unsigned)(wr * 64 + fr) * 256u + (unsigned)(wc * 32 + fq * 8) * 2u;
#pragma unroll
        for (int m = 0; m < 4; ++m) { u32x4 w; store_tile_rows(acc[m][0], acc[m][1], bad, w);
            __builtin_amdgcn_raw_buffer_store_b128(w, rs, (int)(base + (unsigned)(m * 16) * 256u), 0, AUX_SYS); }
    }
    WAIT_V(0); BAR; CFENCE;
    if (owner != rank && tid == 0) {
        __builtin_amdgcn_fence(__ATOMIC_RELEASE, "");
        WAIT_V(0);
        st_sys((unsigned*)p.ws[owner] + (F_OUT + rank * 64 + 28 + (a & 3) * 4 + (b - 28)) * 32, call);
    }
}

__global__ __launch_bounds__(512, 2) void fused_gemm_a2a(Params p) {
    extern __shared__ __attribute__((aligned(16))) unsigned char shm[];
    LAS unsigned char* lds = (LAS unsigned char*)shm;
    const int tid = threadIdx.x, bid = blockIdx.x;
    unsigned* fl = (unsigned*)p.ws[p.rank];
    volatile LAS unsigned* lcall = (volatile LAS unsigned*)(lds + STAGE_BYTES + 16);
    if (tid == 0) *lcall = __hip_atomic_fetch_add(fl + F_TICKET * 32, 1u, __ATOMIC_RELAXED, __HIP_MEMORY_SCOPE_AGENT) / (unsigned)NWG + 1u;
    __syncthreads();
    const unsigned call = __builtin_amdgcn_readfirstlane(*lcall);
    STAMP(0);
    if (bid < NPULL) { puller(p, call, bid % 7); STAMP(19); return; }
    stage_units(p, lds, call, bid - NPULL);
    STAMP(1);
    __syncthreads();
    const int xcd = bid & 7, idx = (bid >> 3) - (xcd < 4 ? 4 : 3), rg = xcd >> 1, cg = xcd & 1;
    const int n1 = 28, nw = xcd < 4 ? 28 : 29;
    if (idx < n1) {
        gemm_r1(p, lds, call, rg * 4 + (idx & 3), cg * 7 + (idx >> 2));
        STAMP(4);
    } else if (bid == 252 && tid < 64) monitor(p, call);
    volatile LAS unsigned* lq = (volatile LAS unsigned*)(lds + STAGE_BYTES + 32);
    for (;;) {
        __syncthreads();
        if (tid == 0) *lq = __hip_atomic_fetch_add(fl + (F_Q2 + xcd) * 32, 1u, __ATOMIC_RELAXED, __HIP_MEMORY_SCOPE_AGENT) - (call - 1u) * (unsigned)(16 + nw);
        __syncthreads();
        const unsigned r2 = __builtin_amdgcn_readfirstlane(*lq);
        if (r2 >= 16u) break;
        gemm_r2(p, lds, call, rg * 8 + (int)(r2 & 7u), 28 + cg * 2 + (int)(r2 >> 3));
        STAMP(5);
    }
}
}

extern "C" void kernel_launch(int rank, int nranks, ncclComm_t comm,
                              hipStream_t stream,
                              void* const* d_in, const int* in_sizes, int n_in,
                              void* d_out, int out_size,
                              const int* in_dtypes, int out_dtype,
                              void* const* d_out_all,
                              void* const* d_ws_all, size_t ws_bytes) {
    if (nranks != NR || n_in != 2 || ws_bytes < (size_t)WS_NEED || (unsigned)rank >= (unsigned)NR) { fprintf(stderr, "kernel_launch: unexpected configuration\n"); return; }
    Params p;
    p.x = (const unsigned short*)d_in[0]; p.w = (const unsigned short*)d_in[1]; p.out = (unsigned short*)d_out; p.rank = rank;
    for (int i = 0; i < NR; ++i) p.ws[i] = (unsigned char*)d_ws_all[i];
    HIP_CHECK(hipFuncSetAttribute((const void*)fused_gemm_a2a, hipFuncAttributeMaxDynamicSharedMemorySize, LDS_BYTES));
    hipLaunchKernelGGL(fused_gemm_a2a, dim3(NWG), dim3(512), LDS_BYTES, stream, p);
    HIP_CHECK(hipGetLastError());
}
```

```cpp
#include <hip/hip_runtime.h>
#include <rccl/rccl.h>
#include <cstdio>
#include <cstdint>

#define HIP_CHECK(expr) do { hipError_t _e = (expr); if (_e != hipSuccess) { \
    fprintf(stderr, "HIP error %s at %s:%d\n", hipGetErrorString(_e), __FILE__, __LINE__); \
    return; } } while (0)

namespace {
typedef unsigned u32x4 __attribute__((ext_vector_type(4)));
typedef short bf16x8 __attribute__((ext_vector_type(8)));
typedef float f32x4 __attribute__((ext_vector_type(4)));
#define LAS __attribute__((address_space(3)))

constexpr int NR = 8;
constexpr int MM = 4096, KS = 512, NN = 32768, NS = 4096, MS = 512;
constexpr unsigned IMG = 16384;
constexpr unsigned SLAB = 32 * IMG;
constexpr unsigned OFF_A = 1u << 20;
constexpr unsigned OFF_B = OFF_A + 64 * SLAB;
constexpr unsigned OFF_W = OFF_B + 64 * SLAB;
constexpr unsigned OFF_O = OFF_W + 64 * SLAB;
constexpr unsigned WS_NEED = OFF_O + 256u * 131072u;
constexpr unsigned OUT_BYTES = (unsigned)MS * NN * 2u;
constexpr int F_TICKET = 0, F_TMO = 1, F_SCNT = 2, F_SFLAG = 16, F_INCNT = 80, F_INB = 144, F_Q = 152, F_OUT = 160  , F_INO = 672, F_Q2 = 736, F_SUM = 744;
constexpr int NPULL = 28, NWG = 256;
constexpr int NSTAGER = NWG - NPULL;
constexpr int UNITS_PER_CHUNK = 72, NUNITS = 8 * UNITS_PER_CHUNK;
constexpr unsigned long long DEADLINE = 50000000ull;
constexpr int HTB = 16384, STAGE_BYTES = 8 * HTB, LDS_BYTES = STAGE_BYTES + 512;
constexpr int AUX_SYS = 17, AUX_AGENT = 16;
#define MON_SLEEP 8
#define ODD_FAST 6
#define SLOW_SLEEP 127
#define POLL_FAST_FROM 56
#define POLL_SLEEP 16
#define PULL_LD_AUX 17
#define PULL_ST_AUX 16

struct Params { const unsigned short* x; const unsigned short* w; unsigned char* ws[NR]; unsigned short* out; int rank; };

__device__ __forceinline__ int lds_byte(int r, int c) { const int st = (r >> 4) * 2 + (c >> 5), rr = r & 15, cc = c & 31, ob = rr * 64 + cc * 2; return st * 1024 + (ob ^ (((ob >> 9) & 1) << 5)); }
__device__ __forceinline__ void stage_rc(int b, int& R, int& C) { const int st = b / 1024, sb = b % 1024, swz = sb ^ (((sb >> 9) & 1) << 5); R = (st >> 1) * 16 + swz / 64; C = (st & 1) * 32 + (swz % 64) / 2; }

__device__ __forceinline__ unsigned ld_sys(const unsigned* p) { return __hip_atomic_load(p, __ATOMIC_RELAXED, __HIP_MEMORY_SCOPE_SYSTEM); }
__device__ __forceinline__ void st_sys(unsigned* p, unsigned v) { __hip_atomic_store(p, v, __ATOMIC_RELAXED, __HIP_MEMORY_SCOPE_SYSTEM); }
__device__ __forceinline__ void wait_ge(const unsigned* flag, unsigned val, unsigned* tmo, unsigned call) {
    const unsigned long long t0 = __builtin_amdgcn_s_memrealtime();
    while (ld_sys(flag) < val) {
        if (__hip_atomic_load(tmo, __ATOMIC_RELAXED, __HIP_MEMORY_SCOPE_AGENT) == call || __builtin_amdgcn_s_memrealtime() - t0 > DEADLINE) {
            __hip_atomic_store(tmo, call, __ATOMIC_RELAXED, __HIP_MEMORY_SCOPE_AGENT); break; }
        __builtin_amdgcn_s_sleep(2);
    }
}
__device__ __forceinline__ unsigned cvt_pk_bf16(float lo, float hi) { unsigned r; asm volatile("v_cvt_pk_bf16_f32 %0, %1, %2" : "=v"(r) : "v"(lo), "v"(hi)); return r; }
typedef float f32x2 __attribute__((ext_vector_type(2)));
__device__ __forceinline__ unsigned gelu2_bf16(float y0, float y1) {
    const f32x2 y = {y0, y1};
    const f32x2 t = (y * y) * -0.1029432395800235f + -2.302208198144325f;
    const f32x2 a = y * t;
    f32x2 d; d.x = __builtin_amdgcn_exp2f(a.x); d.y = __builtin_amdgcn_exp2f(a.y);
    d = d + 1.0f;
    f32x2 r; r.x = __builtin_amdgcn_rcpf(d.x); r.y = __builtin_amdgcn_rcpf(d.y);
    r = r * y;
    return cvt_pk_bf16(r.x, r.y);
}
#define STAMP(i) do { } while (0)
#define WAIT_V(n) asm volatile("s_waitcnt vmcnt(" #n ")" ::: "memory")
#define WAIT_L(n) asm volatile("s_waitcnt lgkmcnt(" #n ")" ::: "memory")
#define BAR __builtin_amdgcn_s_barrier()
#define SCHED __builtin_amdgcn_sched_barrier(0)
#define CFENCE asm volatile("" ::: "memory")
#define ACQ_AGENT CFENCE

__device__ __forceinline__ void stage_units(const Params& p, LAS unsigned char* lds, unsigned call, int first) {
    const int tid = threadIdx.x, rank = p.rank;
    unsigned char* wsl = p.ws[rank]; unsigned* fl = (unsigned*)wsl;
    const auto rs = __builtin_amdgcn_make_buffer_rsrc(wsl, 0, (int)WS_NEED, 0x00020000);
    for (int u = first; u < NUNITS; u += NSTAGER) {
        const int c = u / UNITS_PER_CHUNK, v = u % UNITS_PER_CHUNK;
        if (v >= 64) {
            const int vv = v - 64;
            const unsigned dbase = OFF_A + (unsigned)(rank * 8 + c) * SLAB + (unsigned)vv * 4u * IMG;
            u32x4 t[8];
#pragma unroll
            for (int i = 0; i < 8; ++i) { const int pc = tid + 512 * i, img = pc >> 10; int R, C; stage_rc((pc & 1023) * 16, R, C);
                t[i] = *(const u32x4*)(p.x + (size_t)(vv * 512 + img * 128 + R) * KS + c * 64 + C); }
#pragma unroll
            for (int i = 0; i < 8; ++i) __builtin_amdgcn_raw_buffer_store_b128(t[i], rs, (int)(dbase + (unsigned)(tid + 512 * i) * 16u), 0, AUX_SYS);
        } else {
            const int d = v >> 3, kb = tid >> 6, nb = tid & 63;
            const unsigned dbase = (d == rank ? OFF_B + (unsigned)(rank * 8 + c) * SLAB : OFF_W + (unsigned)(d * 8 + c) * SLAB) + (unsigned)(v & 7) * 4u * IMG;
            u32x4 r[8];
#pragma unroll
            for (int i = 0; i < 8; ++i) r[i] = *(const u32x4*)(p.w + (size_t)(c * 64 + kb * 8 + i) * NN + v * 512 + nb * 8);
#pragma unroll
            for (int j = 0; j < 8; ++j) {
                u32x4 o;
#pragma unroll
                for (int e = 0; e < 4; ++e) o[e] = __builtin_amdgcn_perm(r[2 * e + 1][j >> 1], r[2 * e][j >> 1], (j & 1) ? 0x07060302u : 0x05040100u);
                const int nl = nb * 8 + j, img = nl >> 7, rowi = nl & 127, cc = rowi & 31;
                const int rho = (rowi & ~31) + 16 * ((cc >> 2) & 1) + 4 * (cc >> 3) + (cc & 3);
                *(LAS u32x4*)(lds + img * IMG + lds_byte(rho, kb * 8)) = o;
            }
            __syncthreads();
#pragma unroll
            for (int i = 0; i < 8; ++i) { const unsigned b = (unsigned)(tid + 512 * i) * 16u; const u32x4 t = *(const LAS u32x4*)(lds + b);
                __builtin_amdgcn_raw_buffer_store_b128(t, rs, (int)(dbase + b), 0, AUX_SYS); }
        }
        WAIT_V(0);
        __syncthreads();
        if (tid == 0) {
            const unsigned old = __hip_atomic_fetch_add(fl + (F_SCNT + c) * 32, 1u, __ATOMIC_RELAXED, __HIP_MEMORY_SCOPE_AGENT);
            if (old + 1u == call * (unsigned)UNITS_PER_CHUNK) {
#pragma unroll
                for (int d = 0; d < NR; ++d) st_sys((unsigned*)p.ws[d] + (F_SFLAG + rank * 8 + c) * 32, call);
            }
        }
    }
}

struct Desc { unsigned soff, doff; int kind; int flag; int sig; int half; };
#define RD 4
#define NPW 8
constexpr int NITEM = 1536, NFETCH = NITEM + 4 * NPW;
#define ND 4
constexpr int I_E0 = (8 - ND) * 120, I_O0 = I_E0 + ND * 88;
constexpr unsigned INE_ITEMS = 88, INO_ITEMS = 32, INB_ITEMS = 64;
__device__ __forceinline__ Desc puller_desc(int it, int rank, int s) {
    Desc d; d.sig = -1; d.kind = 0; d.half = -1;
    if (it < 960) {
        int c, j;
        if (it < I_E0) { c = it / 120; j = it - c * 120; }
        else if (it < I_O0) { c = (8 - ND) + (it - I_E0) / 88; j = (it - I_E0) - (c - (8 - ND)) * 88; j = j < 32 ? ((j >> 2) << 3) + (j & 3) : j + 32; }
        else { c = (8 - ND) + ((it - I_O0) >> 5); j = (it - I_O0) & 31; j = ((j >> 2) << 3) + 4 + (j & 3); }
        d.flag = F_SFLAG + s * 8 + c; d.sig = ((j < 64 && (j & 4)) ? F_INO : F_INCNT) + s * 8 + c;
        if (j < 64) { d.soff = OFF_A + (unsigned)(8 * s + c) * SLAB + (unsigned)j * 8192u; d.doff = d.soff; }
        else { const unsigned off = (unsigned)(j - 64) * 8192u; d.soff = OFF_W + (unsigned)(rank * 8 + c) * SLAB + off; d.doff = OFF_B + (unsigned)(8 * s + c) * SLAB + off; }
    } else if (it < 1024) {
        const int c = (it - 960) >> 3, j = (it - 960) & 7; const unsigned off = 28u * IMG + (unsigned)j * 8192u;
        d.flag = F_SFLAG + s * 8 + c; d.sig = F_INB + s;
        d.soff = OFF_W + (unsigned)(rank * 8 + c) * SLAB + off; d.doff = OFF_B + (unsigned)(8 * s + c) * SLAB + off;
    } else if (it < 1472) {
        const int idx = (it - 1024) >> 4, j = (it - 1024) & 15, pml = idx / 14, pn = idx - pml * 14;
        d.kind = 1; d.flag = F_OUT + s * 64 + idx; d.half = j >> 3;
        d.soff = OFF_O + (unsigned)((2 * rank + pml) * 16 + pn) * 131072u + (unsigned)j * 8192u;
        d.doff = (unsigned)(pml * 256 + j * 16) * 65536u + (unsigned)(s * 4096 + pn * 256) * 2u;
    } else {
        const int idx = (it - 1472) >> 2, j = (it - 1472) & 3, al = idx >> 2, b = 28 + (idx & 3);
        d.kind = 2; d.flag = F_OUT + s * 64 + 28 + idx;
        d.soff = OFF_O + (unsigned)((2 * rank + (al >> 1)) * 16 + (b >> 1)) * 131072u + (unsigned)((al & 1) * 2 + (b & 1)) * 32768u + (unsigned)j * 8192u;
        d.doff = (unsigned)(al * 128 + j * 32) * 65536u + (unsigned)(s * 4096 + b * 128) * 2u;
    }
    return d;
}
#define STR2(x) #x
#define STR(x) STR2(x)
typedef int i32x4 __attribute__((ext_vector_type(4)));
__device__ __forceinline__ i32x4 mk_rsrc(const void* ptr, unsigned bytes) {
    const unsigned long long a = (unsigned long long)ptr; i32x4 r;
    r.x = __builtin_amdgcn_readfirstlane((int)(unsigned)a); r.y = __builtin_amdgcn_readfirstlane((int)(unsigned)(a >> 32) & 0xffff); r.z = (int)bytes; r.w = 0x00020000; return r;
}
__device__ __forceinline__ void puller(const Params& p, unsigned call, int l) {
    const int lane = threadIdx.x & 63, rank = p.rank, s = (rank + 1 + l) & 7;
    unsigned* fl = (unsigned*)p.ws[rank]; unsigned* tmo = fl + F_TMO * 32; unsigned* qctr = fl + (F_Q + l) * 32;
    const unsigned qbase = (call - 1u) * (unsigned)NFETCH;
    const i32x4 rsrc = mk_rsrc(p.ws[s], WS_NEED), rdst = mk_rsrc(p.ws[rank], WS_NEED), rout = mk_rsrc(p.out, OUT_BYTES);
    const unsigned l0 = (unsigned)lane * 16u, l1 = (unsigned)(lane >> 5) * 65536u + (unsigned)(lane & 31) * 16u, l2 = (unsigned)(lane >> 4) * 65536u + (unsigned)(lane & 15) * 16u;
    u32x4 v[RD];
    int cur, nxt;
    if ((int)(threadIdx.x >> 6) >= NPW) return;
    { unsigned t = 0; if (lane == 0) t = __hip_atomic_fetch_add(qctr, 1u, __ATOMIC_RELAXED, __HIP_MEMORY_SCOPE_AGENT); cur = (int)(__builtin_amdgcn_readfirstlane(t) - qbase); nxt = NITEM;
      if (cur < NITEM) { if (lane == 0) t = __hip_atomic_fetch_add(qctr, 1u, __ATOMIC_RELAXED, __HIP_MEMORY_SCOPE_AGENT); nxt = (int)(__builtin_amdgcn_readfirstlane(t) - qbase); } }
    bool cur_ready = false, have_prev = false, pbad = false, bad = false; int lastph = -1; (void)lastph;
    unsigned pdoff = 0, plane = 0, pstep = 0; int pkind = 0, psig = -1;
#define P_STORE(r, j, doff_, kind_, lane_, step_, bad_) do { \
        if ((kind_) == 0) { asm volatile("buffer_store_dwordx4 %0, %1, %2, 0 offen sc1\n\ts_nop 1" :: "v"(v[r]), "v"((doff_) + (lane_) + (unsigned)(j) * (step_)), "s"(rdst) : "memory"); } \
        else { u32x4 t_ = v[r]; if (bad_) t_ = (u32x4){0x7fc07fc0u, 0x7fc07fc0u, 0x7fc07fc0u, 0x7fc07fc0u}; \
            asm volatile("buffer_store_dwordx4 %0, %1, %2, 0 offen\n\ts_nop 1" :: "v"(t_), "v"((doff_) + (lane_) + (unsigned)(j) * (step_)), "s"(rout) : "memory"); } } while (0)
#define P_SIG(sg) do { if ((sg) >= 0 && lane == 0) __hip_atomic_fetch_add(fl + (sg) * 32, 1u, __ATOMIC_RELAXED, __HIP_MEMORY_SCOPE_AGENT); } while (0)
#define P_FLUSH() do { if (have_prev) { WAIT_V(0); _Pragma("unroll") for (int r = 0; r < RD; ++r) { asm volatile("" : "+v"(v[r])); P_STORE(r, 8 - RD + r, pdoff, pkind, plane, pstep, pbad); } \
            WAIT_V(0); P_SIG(psig); P_SIG(psig2); psig2 = -1; have_prev = false; } } while (0)
    int psig2 = -1;
    while (cur < NITEM) {
        const Desc dc = puller_desc(cur, rank, s);
        if (!cur_ready) {
            P_FLUSH();
            if (lane == 0) wait_ge(fl + dc.flag * 32, dc.half < 0 ? call : 2u * call - 1u + (unsigned)dc.half, tmo, call);
            bad = __builtin_amdgcn_readfirstlane(__hip_atomic_load(tmo, __ATOMIC_RELAXED, __HIP_MEMORY_SCOPE_AGENT)) == call;
        }
        unsigned fv = 0, tk = 0, ftgt = call;
        if (nxt < NITEM) {
            const Desc dn = puller_desc(nxt, rank, s);
            if (dn.half >= 0) ftgt = 2u * call - 1u + (unsigned)dn.half;
            asm volatile("global_load_dword %0, %1, off sc0 sc1" : "=v"(fv) : "v"(fl + dn.flag * 32) : "memory");
            if (lane == 0) asm volatile("global_atomic_add %0, %1, %2, off sc0" : "=v"(tk) : "v"(qctr), "v"(1u) : "memory");
        }
        const unsigned clane = dc.kind == 0 ? l0 : (dc.kind == 1 ? l1 : l2), cstep = dc.kind == 0 ? 1024u : (dc.kind == 1 ? 131072u : 262144u);
#pragma unroll
        for (int j = 0; j < 8; ++j) {
            const int r = j & (RD - 1);
            if (j >= RD || have_prev) {
                asm volatile("s_waitcnt vmcnt(" STR(RD) "-1)" : "+v"(v[r]) :: "memory");
                if (j >= RD) P_STORE(r, j - RD, dc.doff, dc.kind, clane, cstep, bad); else P_STORE(r, 8 - RD + j, pdoff, pkind, plane, pstep, pbad);
            }
            asm volatile("buffer_load_dwordx4 %0, %1, %2, 0 offen sc0 sc1" : "=v"(v[r]) : "v"(dc.soff + l0 + (unsigned)j * 1024u), "s"(rsrc) : "memory");
        }
        if (have_prev) P_SIG(psig);
        asm volatile("" : "+v"(fv), "+v"(tk));
        have_prev = true; pdoff = dc.doff; pkind = dc.kind; plane = clane; pstep = cstep; psig = dc.sig; pbad = bad;
        cur_ready = (unsigned)__builtin_amdgcn_readfirstlane(fv) >= ftgt;
        const int nn = nxt < NITEM ? (int)(__builtin_amdgcn_readfirstlane(tk) - qbase) : NITEM;
        if ((cur >= I_O0 - 32 && cur < I_O0) || (cur >= 960 - 32 && cur < 960) || (cur >= 1024 - 32 && cur < 1024)) P_FLUSH();
        cur = nxt; nxt = nn;
    }
    P_FLUSH();
}

__device__ __forceinline__ unsigned long long ld_sum(const unsigned* fl, int w, unsigned call) {
    const unsigned long long v = __hip_atomic_load((const unsigned long long*)(fl + F_SUM * 32) + w, __ATOMIC_RELAXED, __HIP_MEMORY_SCOPE_AGENT);
    return (unsigned)(v >> 56) == (call & 0xffu) ? (v & 0x00ffffffffffffffull) : 0ull;
}
__device__ __forceinline__ void monitor(const Params& p, unsigned call) {
    const int lane = threadIdx.x & 63, rank = p.rank;
    unsigned* fl = (unsigned*)p.ws[rank]; unsigned* tmo = fl + F_TMO * 32;
    unsigned long long* sum = (unsigned long long*)(fl + F_SUM * 32);
    const int c = lane / 7, l = lane - c * 7, s = (rank + 1 + l) & 7;
    const unsigned* pe = fl + (F_INCNT + s * 8 + (c & 7)) * 32; const unsigned* po = fl + (F_INO + s * 8 + (c & 7)) * 32; const unsigned* pb = fl + (F_INB + s) * 32;
    const unsigned long long tag = (unsigned long long)(call & 0xffu) << 56, full = 0x00ffffffffffffffull;
    unsigned long long me = 0, mo = 0, mb = 0;
    const unsigned long long t0 = __builtin_amdgcn_s_memrealtime();
    for (;;) {
        const int fe = (int)__builtin_ctzll(~me), fo = (int)__builtin_ctzll(~mo);
        const bool ae = lane < 56 && !((me >> lane) & 1ull) && lane < fe + 14, ao = lane < 56 && !((mo >> lane) & 1ull) && lane < fo + 14 && ((me >> lane) & 1ull);
        const bool ab = lane < 7 && !((mb >> lane) & 1ull) && me == full;
        const bool be = ae && ld_sys(pe) >= INE_ITEMS * call, bo = ao && ld_sys(po) >= INO_ITEMS * call, bb = ab && ld_sys(pb) >= INB_ITEMS * call;
        const unsigned long long ne = me | __ballot(be), no = mo | __ballot(bo), nb = mb | __ballot(bb);
        if (lane == 0) {
            if (ne != me) __hip_atomic_store(sum + 0, ne | tag, __ATOMIC_RELAXED, __HIP_MEMORY_SCOPE_AGENT);
            if (no != mo) __hip_atomic_store(sum + 1, no | tag, __ATOMIC_RELAXED, __HIP_MEMORY_SCOPE_AGENT);
            if (nb != mb) __hip_atomic_store(sum + 2, nb | tag, __ATOMIC_RELAXED, __HIP_MEMORY_SCOPE_AGENT);
        }
        me = ne; mo = no; mb = nb;
        if (me == full && mo == full && mb == 0x7full) break;
        if (__builtin_amdgcn_readfirstlane(__hip_atomic_load(tmo, __ATOMIC_RELAXED, __HIP_MEMORY_SCOPE_AGENT)) == call || __builtin_amdgcn_s_memrealtime() - t0 > DEADLINE) {
            if (lane == 0) __hip_atomic_store(tmo, call, __ATOMIC_RELAXED, __HIP_MEMORY_SCOPE_AGENT); break; }
        __builtin_amdgcn_s_sleep(MON_SLEEP);
    }
}

__device__ __forceinline__ int kt_of(int t, int rank) {
    if (t < 8) return rank * 8 + t;
    const int tt = t - 8, c = tt / 7, l = tt - c * 7; return ((rank + 1 + l) & 7) * 8 + c;
}
__device__ __forceinline__ void store_tile_rows(const f32x4& v0, const f32x4& v1, bool bad, u32x4& w) {
    w.x = gelu2_bf16(v0[0], v0[1]); w.y = gelu2_bf16(v0[2], v0[3]); w.z = gelu2_bf16(v1[0], v1[1]); w.w = gelu2_bf16(v1[2], v1[3]);
    if (bad) w = (u32x4){0x7fc07fc0u, 0x7fc07fc0u, 0x7fc07fc0u, 0x7fc07fc0u};
}

__device__ __forceinline__ void gemm_r1(const Params& p, LAS unsigned char* lds, unsigned call, int pm, int pn) {
    const int tid = threadIdx.x, wid = __builtin_amdgcn_readfirstlane(tid >> 6), lane = tid & 63, wr = wid >> 2, wc = wid & 3, fr = lane & 15, fq = lane >> 4;
    const int rank = p.rank;
    unsigned char* wsl = p.ws[rank]; unsigned* fl = (unsigned*)wsl; unsigned* tmo = fl + F_TMO * 32;
    volatile LAS unsigned* lready = (volatile LAS unsigned*)(lds + STAGE_BYTES);
    const char* gA = (const char*)wsl + OFF_A + (size_t)pm * 2 * IMG + tid * 16;
    const char* gB = (const char*)wsl + OFF_B + (size_t)pn * 2 * IMG + tid * 16;
    const unsigned ldsw = (unsigned)wid * 1024u;
    const int aoff = lds_byte(wr * 64 + fr, fq * 8), boff = lds_byte(wc * 32 + fr, fq * 8);
    const unsigned* myflag = fl + (F_SFLAG + rank * 8 + (lane & 7)) * 32;
    int ready = 0;
#define SA(b, h) (((b) * 2 + (h)) * HTB)
#define SB(b, h) ((4 + (b) * 2 + (h)) * HTB)
#define STAGE(bufoff, gptr) do { _Pragma("unroll") for (int _i = 0; _i < 2; ++_i) \
        __builtin_amdgcn_global_load_lds((const unsigned*)((gptr) + _i * 8192), (LAS unsigned*)(lds + (bufoff) + ldsw + _i * 8192), 16, 0, 0); } while (0)
#define LDA(dst, b, h) do { _Pragma("unroll") for (int m = 0; m < 4; ++m) _Pragma("unroll") for (int k = 0; k < 2; ++k) dst[m][k] = *(const LAS bf16x8*)(lds + SA(b, h) + aoff + m * 2048 + k * 1024); } while (0)
#define LDB(dst, b, h) do { _Pragma("unroll") for (int n = 0; n < 2; ++n) _Pragma("unroll") for (int k = 0; k < 2; ++k) dst[n][k] = *(const LAS bf16x8*)(lds + SB(b, h) + boff + n * 2048 + k * 1024); } while (0)
#define MMA(ai, bj, At, Bt) do { __builtin_amdgcn_s_setprio(1); _Pragma("unroll") for (int m = 0; m < 4; ++m) _Pragma("unroll") for (int n = 0; n < 2; ++n) _Pragma("unroll") for (int k = 0; k < 2; ++k) \
        acc[ai][bj][m][n] = __builtin_amdgcn_mfma_f32_16x16x32_bf16(Bt[n][k], At[m][k], acc[ai][bj][m][n], 0, 0, 0); __builtin_amdgcn_s_setprio(0); } while (0)
    volatile LAS int* lorder = (volatile LAS int*)(lds + STAGE_BYTES + 64);
    unsigned long long sched = 0ull;
    const int mykt = kt_of(lane, rank);
#define NEED(nd) do { if ((nd) > ready) { \
        if (wid == 0) { const unsigned long long t0 = __builtin_amdgcn_s_memrealtime(); int cnt = ready; \
            for (;;) { \
                const unsigned long long sm = ld_sum(fl, pm & 1, call) << 8; \
                bool ok = !((sched >> lane) & 1ull) && (lane < 8 ? ld_sys(myflag) >= call : ((sm >> lane) & 1ull) != 0ull); \
                bool giveup = false; \
                if (cnt + __builtin_popcountll(__ballot(ok)) < (nd)) { \
                    giveup = __builtin_amdgcn_readfirstlane(__hip_atomic_load(tmo, __ATOMIC_RELAXED, __HIP_MEMORY_SCOPE_AGENT)) == call || __builtin_amdgcn_s_memrealtime() - t0 > DEADLINE; \
                    if (giveup) { if (lane == 0) { __hip_atomic_store(tmo, call, __ATOMIC_RELAXED, __HIP_MEMORY_SCOPE_AGENT); lready[1] = 1u; } ok = !((sched >> lane) & 1ull); } } \
                const unsigned long long bal = __ballot(ok); \
                if (ok) lorder[cnt + __builtin_popcountll(bal & ((1ull << lane) - 1ull))] = mykt; \
                sched |= bal; cnt += __builtin_popcountll(bal); \
                if (cnt >= (nd)) break; \
                if ((nd) > 10 && (nd) < ((pm & 1) ? 64 - 7 * ND + ODD_FAST : POLL_FAST_FROM)) __builtin_amdgcn_s_sleep(SLOW_SLEEP); else __builtin_amdgcn_s_sleep(POLL_SLEEP); } \
            ACQ_AGENT; \
            if (lane == 0) *lready = (unsigned)cnt; \
            asm volatile("s_waitcnt vmcnt(0) lgkmcnt(0)" ::: "memory"); } \
        CFENCE; BAR; CFENCE; \
        ready = __builtin_amdgcn_readfirstlane((int)*lready); } } while (0)
#define KOFF(t) ((size_t)(unsigned)__builtin_amdgcn_readfirstlane(lorder[t]) * SLAB)

    f32x4 acc[2][2][4][2];
#pragma unroll
    for (int a = 0; a < 2; ++a)
#pragma unroll
        for (int b = 0; b < 2; ++b)
#pragma unroll
            for (int m = 0; m < 4; ++m)
#pragma unroll
                for (int n = 0; n < 2; ++n) acc[a][b][m][n] = (f32x4){0.f, 0.f, 0.f, 0.f};
    bf16x8 At[4][2], B0[2][2], B1[2][2];
    if (tid == 0) lready[1] = 0u;
    NEED(4);
    size_t o1, o2, o3;
    {
        const size_t k0 = KOFF(0); o1 = KOFF(1); o2 = KOFF(2); o3 = KOFF(3);
        STAGE(SB(0, 0), gB + k0); STAGE(SA(0, 0), gA + k0); STAGE(SB(0, 1), gB + k0 + IMG); STAGE(SA(0, 1), gA + k0 + IMG);
        if (wr == 1) BAR;
        WAIT_V(4); BAR;
        STAGE(SB(1, 0), gB + o1); STAGE(SA(1, 0), gA + o1); STAGE(SB(1, 1), gB + o1 + IMG);
        WAIT_V(6); BAR;
    }
    STAMP(2);
    for (int t = 0; t < 64; t += 2) {
        LDB(B0, 0, 0); SCHED; LDA(At, 0, 0); STAGE(SA(1, 1), gA + o1 + IMG);
        WAIT_L(8); BAR; WAIT_L(0); MMA(0, 0, At, B0); BAR; SCHED;
        LDB(B1, 0, 1); STAGE(SB(0, 0), gB + o2);
        BAR; WAIT_L(0); MMA(0, 1, At, B1); BAR;
        LDA(At, 0, 1); STAGE(SA(0, 0), gA + o2);
        BAR; WAIT_L(0); MMA(1, 0, At, B0); BAR; SCHED;
        STAGE(SB(0, 1), gB + o2 + IMG);
        WAIT_V(6); BAR; MMA(1, 1, At, B1); BAR;
        LDB(B0, 1, 0); SCHED; LDA(At, 1, 0); STAGE(SA(0, 1), gA + o2 + IMG);
        WAIT_L(8); BAR; WAIT_L(0); MMA(0, 0, At, B0); BAR; SCHED;
        LDB(B1, 1, 1); STAGE(SB(1, 0), gB + o3);
        BAR; WAIT_L(0); MMA(0, 1, At, B1); BAR;
        LDA(At, 1, 1); STAGE(SA(1, 0), gA + o3);
        BAR; WAIT_L(0); MMA(1, 0, At, B0); BAR; SCHED;
        STAGE(SB(1, 1), gB + o3 + IMG);
        WAIT_V(6); BAR; MMA(1, 1, At, B1); BAR;
        o1 = o3;
        if (t + 6 <= 64) { NEED(t + 6); o2 = KOFF(t + 4); o3 = KOFF(t + 5); }
    }
    STAMP(3);
    {
        int tz = threadIdx.x; asm volatile("" : "+v"(tz));
        const int lane2 = tz & 63, fr2 = lane2 & 15, fq2 = lane2 >> 4;
        const bool bad = lready[1] != 0u;
        const int owner = pm >> 1;
        if (wr == 0) BAR;
        if (owner == rank) {
            const auto rout = __builtin_amdgcn_make_buffer_rsrc(p.out, 0, (int)OUT_BYTES, 0x00020000);
            const unsigned base = (unsigned)((pm & 1) * 256 + wr * 64 + fr2) * 65536u + (unsigned)(rank * 4096 + pn * 256 + wc * 32 + fq2 * 8) * 2u;
#pragma unroll
            for (int ai = 0; ai < 2; ++ai)
#pragma unroll
                for (int m = 0; m < 4; ++m)
#pragma unroll
                    for (int bj = 0; bj < 2; ++bj) { u32x4 w; store_tile_rows(acc[ai][bj][m][0], acc[ai][bj][m][1], bad, w);
                        __builtin_amdgcn_raw_buffer_store_b128(w, rout, (int)(base + (unsigned)(ai * 128 + m * 16) * 65536u + (unsigned)bj * 256u), 0, 0); }
            WAIT_V(0); BAR; CFENCE;
        } else {
            const auto rs = __builtin_amdgcn_make_buffer_rsrc(wsl, 0, (int)WS_NEED, 0x00020000);
            const unsigned base = OFF_O + (unsigned)(pm * 16 + pn) * 131072u + (unsigned)(wr * 64 + fr2) * 512u + (unsigned)(wc * 32 + fq2 * 8) * 2u;
            unsigned* oflag = (unsigned*)p.ws[owner] + (F_OUT + rank * 64 + (pm & 1) * 14 + pn) * 32;
#pragma unroll
            for (int ai = 0; ai < 2; ++ai) {
#pragma unroll
                for (int m = 0; m < 4; ++m)
#pragma unroll
                    for (int bj = 0; bj < 2; ++bj) { u32x4 w; store_tile_rows(acc[ai][bj][m][0], acc[ai][bj][m][1], bad, w);
                        __builtin_amdgcn_raw_buffer_store_b128(w, rs, (int)(base + (unsigned)(ai * 128 + m * 16) * 512u + (unsigned)bj * 256u), 0, AUX_SYS); }
                WAIT_V(0); BAR; CFENCE;
                if (tz == 0) {
                    st_sys(oflag, 2u * call - 1u + (unsigned)ai);
                }
            }
        }
    }
#undef NEED
}

__device__ __forceinline__ void gemm_r2(const Params& p, LAS unsigned char* lds, unsigned call, int a, int b) {
    const int tid = threadIdx.x, wid = __builtin_amdgcn_readfirstlane(tid >> 6), lane = tid & 63, wr = wid >> 2, wc = wid & 3, fr = lane & 15, fq = lane >> 4;
    const int rank = p.rank;
    unsigned char* wsl = p.ws[rank]; unsigned* fl = (unsigned*)wsl; unsigned* tmo = fl + F_TMO * 32;
    if (tid < 64) {
        if (lane < 8) wait_ge(fl + (F_SFLAG + rank * 8 + lane) * 32, call, tmo, call);
        const unsigned long long t0 = __builtin_amdgcn_s_memrealtime();
        while (ld_sum(fl, (a >> 1) & 1, call) != 0x00ffffffffffffffull || ld_sum(fl, 2, call) != 0x7full) {
            if (__builtin_amdgcn_readfirstlane(__hip_atomic_load(tmo, __ATOMIC_RELAXED, __HIP_MEMORY_SCOPE_AGENT)) == call || __builtin_amdgcn_s_memrealtime() - t0 > DEADLINE) {
                if (lane == 0) __hip_atomic_store(tmo, call, __ATOMIC_RELAXED, __HIP_MEMORY_SCOPE_AGENT); break; }
            __builtin_amdgcn_s_sleep(16);
        }
        ACQ_AGENT; WAIT_V(0);
    }
    CFENCE; BAR; CFENCE;
    const char* gA = (const char*)wsl + OFF_A + (size_t)a * IMG + tid * 16;
    const char* gB = (const char*)wsl + OFF_B + (size_t)b * IMG + tid * 16;
    const unsigned ldsw = (unsigned)wid * 1024u;
    const int aoff = lds_byte(wr * 64 + fr, fq * 8), boff = lds_byte(wc * 32 + fr, fq * 8);
    f32x4 acc[4][2];
#pragma unroll
    for (int m = 0; m < 4; ++m)
#pragma unroll
        for (int n = 0; n < 2; ++n) acc[m][n] = (f32x4){0.f, 0.f, 0.f, 0.f};
#define STAGE2(slot, kt) do { STAGE((slot) * 32768, gA + (size_t)(kt) * SLAB); STAGE((slot) * 32768 + 16384, gB + (size_t)(kt) * SLAB); } while (0)
#define FRAG2(Af, Bf, slot) do { \
        _Pragma("unroll") for (int n = 0; n < 2; ++n) _Pragma("unroll") for (int k = 0; k < 2; ++k) Bf[n][k] = *(const LAS bf16x8*)(lds + (slot) * 32768 + 16384 + boff + n * 2048 + k * 1024); \
        _Pragma("unroll") for (int m = 0; m < 4; ++m) _Pragma("unroll") for (int k = 0; k < 2; ++k) Af[m][k] = *(const LAS bf16x8*)(lds + (slot) * 32768 + aoff + m * 2048 + k * 1024); } while (0)
#define MMA2(Af, Bf) do { __builtin_amdgcn_s_setprio(1); _Pragma("unroll") for (int k = 0; k < 2; ++k) _Pragma("unroll") for (int m = 0; m < 4; ++m) _Pragma("unroll") for (int n = 0; n < 2; ++n) \
        acc[m][n] = __builtin_amdgcn_mfma_f32_16x16x32_bf16(Bf[n][k], Af[m][k], acc[m][n], 0, 0, 0); __builtin_amdgcn_s_setprio(0); } while (0)
    bf16x8 A0[4][2], Bf0[2][2], A1[4][2], Bf1[2][2];
    STAMP(6);
    STAGE2(0, 0); STAGE2(1, 1); STAGE2(2, 2); STAGE2(3, 3);
    WAIT_V(12); BAR; CFENCE;
    FRAG2(A0, Bf0, 0);
    for (int t = 0; t < 64; t += 4) {
        __builtin_amdgcn_s_waitcnt(0xC07F); WAIT_V(8); BAR; SCHED; STAGE2(0, (t + 4) & 63); FRAG2(A1, Bf1, 1); SCHED; MMA2(A0, Bf0); SCHED;
        __builtin_amdgcn_s_waitcnt(0xC07F); WAIT_V(8); BAR; SCHED; STAGE2(1, (t + 5) & 63); FRAG2(A0, Bf0, 2); SCHED; MMA2(A1, Bf1); SCHED;
        __builtin_amdgcn_s_waitcnt(0xC07F); WAIT_V(8); BAR; SCHED; STAGE2(2, (t + 6) & 63); FRAG2(A1, Bf1, 3); SCHED; MMA2(A0, Bf0); SCHED;
        __builtin_amdgcn_s_waitcnt(0xC07F); WAIT_V(8); BAR; SCHED; STAGE2(3, (t + 7) & 63); FRAG2(A0, Bf0, 0); SCHED; MMA2(A1, Bf1); SCHED;
    }
    WAIT_L(0);
    STAMP(7);
    const bool bad = __hip_atomic_load(tmo, __ATOMIC_RELAXED, __HIP_MEMORY_SCOPE_AGENT) == call;
    const int owner = a >> 2;
    if (owner == rank) {
        const auto rout = __builtin_amdgcn_make_buffer_rsrc(p.out, 0, (int)OUT_BYTES, 0x00020000);
        const unsigned base = (unsigned)((a & 3) * 128 + wr * 64 + fr) * 65536u + (unsigned)(rank * 4096 + b * 128 + wc * 32 + fq * 8) * 2u;
#pragma unroll
        for (int m = 0; m < 4; ++m) { u32x4 w; store_tile_rows(acc[m][0], acc[m][1], bad, w);
            __builtin_amdgcn_raw_buffer_store_b128(w, rout, (int)(base + (unsigned)(m * 16) * 65536u), 0, 0); }
    } else {
        const auto rs = __builtin_amdgcn_make_buffer_rsrc(wsl, 0, (int)WS_NEED, 0x00020000);
        const unsigned base = OFF_O + (unsigned)((a >> 1) * 16 + (b >> 1)) * 131072u + (unsigned)((a & 1) * 2 + (b & 1)) * 32768u + (unsigned)(wr * 64 + fr) * 256u + (unsigned)(wc * 32 + fq * 8) * 2u;
#pragma unroll
        for (int m = 0; m < 4; ++m) { u32x4 w; store_tile_rows(acc[m][0], acc[m][1], bad, w);
            __builtin_amdgcn_raw_buffer_store_b128(w, rs, (int)(base + (unsigned)(m * 16) * 256u), 0, AUX_SYS); }
    }
    WAIT_V(0); BAR; CFENCE;
    if (owner != rank && tid == 0) {
        st_sys((unsigned*)p.ws[owner] + (F_OUT + rank * 64 + 28 + (a & 3) * 4 + (b - 28)) * 32, call);
    }
}

__global__ __launch_bounds__(512, 2) void fused_gemm_a2a(Params p) {
    extern __shared__ __attribute__((aligned(16))) unsigned char shm[];
    LAS unsigned char* lds = (LAS unsigned char*)shm;
    const int tid = threadIdx.x, bid = blockIdx.x;
    unsigned* fl = (unsigned*)p.ws[p.rank];
    volatile LAS unsigned* lcall = (volatile LAS unsigned*)(lds + STAGE_BYTES + 16);
    if (tid == 0) *lcall = __hip_atomic_fetch_add(fl + F_TICKET * 32, 1u, __ATOMIC_RELAXED, __HIP_MEMORY_SCOPE_AGENT) / (unsigned)NWG + 1u;
    __syncthreads();
    const unsigned call = __builtin_amdgcn_readfirstlane(*lcall);
    STAMP(0);
    if (bid < NPULL) { puller(p, call, bid % 7); STAMP(19); return; }
    stage_units(p, lds, call, bid - NPULL);
    STAMP(1);
    __syncthreads();
    const int xcd = bid & 7, idx = (bid >> 3) - (xcd < 4 ? 4 : 3), rg = xcd >> 1, cg = xcd & 1;
    const int n1 = 28, nw = xcd < 4 ? 28 : 29;
    if (idx < n1) {
        gemm_r1(p, lds, call, rg * 4 + (idx & 3), cg * 7 + (idx >> 2));
        STAMP(4);
    } else if (bid == 252 && tid < 64) monitor(p, call);
    volatile LAS unsigned* lq = (volatile LAS unsigned*)(lds + STAGE_BYTES + 32);
    for (;;) {
        __syncthreads();
        if (tid == 0) *lq = __hip_atomic_fetch_add(fl + (F_Q2 + xcd) * 32, 1u, __ATOMIC_RELAXED, __HIP_MEMORY_SCOPE_AGENT) - (call - 1u) * (unsigned)(16 + nw);
        __syncthreads();
        const unsigned r2 = __builtin_amdgcn_readfirstlane(*lq);
        if (r2 >= 16u) break;
        gemm_r2(p, lds, call, rg * 8 + (int)(r2 & 7u), 28 + cg * 2 + (int)(r2 >> 3));
        STAMP(5);
    }
}
}

extern "C" void kernel_launch(int rank, int nranks, ncclComm_t comm,
                              hipStream_t stream,
                              void* const* d_in, const int* in_sizes, int n_in,
                              void* d_out, int out_size,
                              const int* in_dtypes, int out_dtype,
                              void* const* d_out_all,
                              void* const* d_ws_all, size_t ws_bytes) {
    if (nranks != NR || n_in != 2 || ws_bytes < (size_t)WS_NEED || (unsigned)rank >= (unsigned)NR) { fprintf(stderr, "kernel_launch: unexpected configuration\n"); return; }
    Params p;
    p.x = (const unsigned short*)d_in[0]; p.w = (const unsigned short*)d_in[1]; p.out = (unsigned short*)d_out; p.rank = rank;
    for (int i = 0; i < NR; ++i) p.ws[i] = (unsigned char*)d_ws_all[i];
    HIP_CHECK(hipFuncSetAttribute((const void*)fused_gemm_a2a, hipFuncAttributeMaxDynamicSharedMemorySize, LDS_BYTES));
    hipLaunchKernelGGL(fused_gemm_a2a, dim3(NWG), dim3(512), LDS_BYTES, stream, p);
    HIP_CHECK(hipGetLastError());
}
```

```cpp
#include <hip/hip_runtime.h>
#include <rccl/rccl.h>
#include <cstdio>
#include <cstdint>

#define HIP_CHECK(expr) do { hipError_t _e = (expr); if (_e != hipSuccess) { \
    fprintf(stderr, "HIP error %s at %s:%d\n", hipGetErrorString(_e), __FILE__, __LINE__); \
    return; } } while (0)

namespace {
typedef unsigned u32x4 __attribute__((ext_vector_type(4)));
typedef short bf16x8 __attribute__((ext_vector_type(8)));
typedef float f32x4 __attribute__((ext_vector_type(4)));
#define LAS __attribute__((address_space(3)))

constexpr int NR = 8;
constexpr int MM = 4096, KS = 512, NN = 32768, NS = 4096, MS = 512;
constexpr unsigned IMG = 16384;
constexpr unsigned SLAB = 32 * IMG;
constexpr unsigned OFF_A = 1u << 20;
constexpr unsigned OFF_B = OFF_A + 64 * SLAB;
constexpr unsigned OFF_W = OFF_B + 64 * SLAB;
constexpr unsigned OFF_O = OFF_W + 64 * SLAB;
constexpr unsigned WS_NEED = OFF_O + 256u * 131072u;
constexpr unsigned OUT_BYTES = (unsigned)MS * NN * 2u;
constexpr int F_TICKET = 0, F_TMO = 1, F_SCNT = 2, F_SFLAG = 16, F_INCNT = 80, F_INB = 144, F_Q = 152, F_OUT = 160  , F_INO = 672, F_Q2 = 736, F_SUM = 744, F_UF = 1024  ;
constexpr int NPULL = 28, NWG = 256;
constexpr int NSTAGER = NWG - NPULL;
constexpr int UNITS_PER_CHUNK = 72, NUNITS = 8 * UNITS_PER_CHUNK;
constexpr unsigned long long DEADLINE = 50000000ull;
constexpr int HTB = 16384, STAGE_BYTES = 8 * HTB, LDS_BYTES = STAGE_BYTES + 512;
constexpr int AUX_SYS = 17, AUX_AGENT = 16;
#define MON_SLEEP 8
#define ODD_FAST 6
#define SLOW_SLEEP 127
#define POLL_FAST_FROM 56
#define POLL_SLEEP 16
#define PULL_LD_AUX 17
#define PULL_ST_AUX 16

struct Params { const unsigned short* x; const unsigned short* w; unsigned char* ws[NR]; unsigned short* out; int rank; };

__device__ __forceinline__ int lds_byte(int r, int c) { const int st = (r >> 4) * 2 + (c >> 5), rr = r & 15, cc = c & 31, ob = rr * 64 + cc * 2; return st * 1024 + (ob ^ (((ob >> 9) & 1) << 5)); }
__device__ __forceinline__ void stage_rc(int b, int& R, int& C) { const int st = b / 1024, sb = b % 1024, swz = sb ^ (((sb >> 9) & 1) << 5); R = (st >> 1) * 16 + swz / 64; C = (st & 1) * 32 + (swz % 64) / 2; }

__device__ __forceinline__ unsigned ld_sys(const unsigned* p) { return __hip_atomic_load(p, __ATOMIC_RELAXED, __HIP_MEMORY_SCOPE_SYSTEM); }
__device__ __forceinline__ void st_sys(unsigned* p, unsigned v) { __hip_atomic_store(p, v, __ATOMIC_RELAXED, __HIP_MEMORY_SCOPE_SYSTEM); }
__device__ __forceinline__ void wait_ge(const unsigned* flag, unsigned val, unsigned* tmo, unsigned call) {
    const unsigned long long t0 = __builtin_amdgcn_s_memrealtime();
    while (ld_sys(flag) < val) {
        if (__hip_atomic_load(tmo, __ATOMIC_RELAXED, __HIP_MEMORY_SCOPE_AGENT) == call || __builtin_amdgcn_s_memrealtime() - t0 > DEADLINE) {
            __hip_atomic_store(tmo, call, __ATOMIC_RELAXED, __HIP_MEMORY_SCOPE_AGENT); break; }
        __builtin_amdgcn_s_sleep(2);
    }
}
__device__ __forceinline__ unsigned cvt_pk_bf16(float lo, float hi) { unsigned r; asm volatile("v_cvt_pk_bf16_f32 %0, %1, %2" : "=v"(r) : "v"(lo), "v"(hi)); return r; }
typedef float f32x2 __attribute__((ext_vector_type(2)));
__device__ __forceinline__ unsigned gelu2_bf16(float y0, float y1) {
    const f32x2 y = {y0, y1};
    const f32x2 t = (y * y) * -0.1029432395800235f + -2.302208198144325f;
    const f32x2 a = y * t;
    f32x2 d; d.x = __builtin_amdgcn_exp2f(a.x); d.y = __builtin_amdgcn_exp2f(a.y);
    d = d + 1.0f;
    f32x2 r; r.x = __builtin_amdgcn_rcpf(d.x); r.y = __builtin_amdgcn_rcpf(d.y);
    r = r * y;
    return cvt_pk_bf16(r.x, r.y);
}
#define STAMP(i) do { } while (0)
#define WAIT_V(n) asm volatile("s_waitcnt vmcnt(" #n ")" ::: "memory")
#define WAIT_L(n) asm volatile("s_waitcnt lgkmcnt(" #n ")" ::: "memory")
#define BAR __builtin_amdgcn_s_barrier()
#define SCHED __builtin_amdgcn_sched_barrier(0)
#define CFENCE asm volatile("" ::: "memory")
#define ACQ_AGENT CFENCE

__device__ __forceinline__ void stage_units(const Params& p, LAS unsigned char* lds, volatile LAS unsigned* lcall, int first) {
    const int tid = threadIdx.x, rank = p.rank;
    unsigned char* wsl = p.ws[rank]; unsigned* fl = (unsigned*)wsl;
    const auto rs = __builtin_amdgcn_make_buffer_rsrc(wsl, 0, (int)WS_NEED, 0x00020000);
    for (int u = first; u < NUNITS; u += NSTAGER) {
        const int c = u / UNITS_PER_CHUNK, v0 = u % UNITS_PER_CHUNK, v = v0 < 8 ? 64 + v0 : v0 - 8;
        if (v >= 64) {
            const int vv = v - 64;
            const unsigned dbase = OFF_A + (unsigned)(rank * 8 + c) * SLAB + (unsigned)vv * 4u * IMG;
            u32x4 t[8];
#pragma unroll
            for (int i = 0; i < 8; ++i) { const int pc = tid + 512 * i, img = pc >> 10; int R, C; stage_rc((pc & 1023) * 16, R, C);
                t[i] = *(const u32x4*)(p.x + (size_t)(vv * 512 + img * 128 + R) * KS + c * 64 + C); }
#pragma unroll
            for (int i = 0; i < 8; ++i) __builtin_amdgcn_raw_buffer_store_b128(t[i], rs, (int)(dbase + (unsigned)(tid + 512 * i) * 16u), 0, AUX_SYS);
        } else {
            const int d = v >> 3, kb = tid >> 6, nb = tid & 63;
            const unsigned dbase = (d == rank ? OFF_B + (unsigned)(rank * 8 + c) * SLAB : OFF_W + (unsigned)(d * 8 + c) * SLAB) + (unsigned)(v & 7) * 4u * IMG;
            u32x4 r[8];
#pragma unroll
            for (int i = 0; i < 8; ++i) r[i] = *(const u32x4*)(p.w + (size_t)(c * 64 + kb * 8 + i) * NN + v * 512 + nb * 8);
#pragma unroll
            for (int j = 0; j < 8; ++j) {
                u32x4 o;
#pragma unroll
                for (int e = 0; e < 4; ++e) o[e] = __builtin_amdgcn_perm(r[2 * e + 1][j >> 1], r[2 * e][j >> 1], (j & 1) ? 0x07060302u : 0x05040100u);
                const int nl = nb * 8 + j, img = nl >> 7, rowi = nl & 127, cc = rowi & 31;
                const int rho = (rowi & ~31) + 16 * ((cc >> 2) & 1) + 4 * (cc >> 3) + (cc & 3);
                *(LAS u32x4*)(lds + img * IMG + lds_byte(rho, kb * 8)) = o;
            }
            __syncthreads();
#pragma unroll
            for (int i = 0; i < 8; ++i) { const unsigned b = (unsigned)(tid + 512 * i) * 16u; const u32x4 t = *(const LAS u32x4*)(lds + b);
                __builtin_amdgcn_raw_buffer_store_b128(t, rs, (int)(dbase + b), 0, AUX_SYS); }
        }
        WAIT_V(0);
        __syncthreads();
        const unsigned call = *lcall;
        if (tid < NR) {
            if (tid != rank && (v >= 64 || tid == (v >> 3)))
                st_sys((unsigned*)p.ws[tid] + (F_UF + (rank * 8 + c) * 16 + (v >= 64 ? v - 64 : 8 + (v & 7))) * 32, call);
        } else if (tid == 64) {
            const unsigned old = __hip_atomic_fetch_add(fl + (F_SCNT + c) * 32, 1u, __ATOMIC_RELAXED, __HIP_MEMORY_SCOPE_AGENT);
            if (old + 1u == call * (unsigned)UNITS_PER_CHUNK) st_sys(fl + (F_SFLAG + rank * 8 + c) * 32, call);
        }
    }
}

struct Desc { unsigned soff, doff; int kind; int flag; int sig; int half; };
constexpr int NITEM = 1536, NFETCH = NITEM + 32;
#define ND 4
constexpr int I_E0 = (8 - ND) * 120, I_O0 = I_E0 + ND * 88;
constexpr unsigned INE_ITEMS = 88, INO_ITEMS = 32, INB_ITEMS = 64;
__device__ __forceinline__ Desc puller_desc(int it, int rank, int s) {
    Desc d; d.sig = -1; d.kind = 0; d.half = -1;
    if (it < 960) {
        int c, j;
        if (it < I_E0) { c = it / 120; j = it - c * 120; }
        else if (it < I_O0) { c = (8 - ND) + (it - I_E0) / 88; j = (it - I_E0) - (c - (8 - ND)) * 88; j = j < 32 ? ((j >> 2) << 3) + (j & 3) : j + 32; }
        else { c = (8 - ND) + ((it - I_O0) >> 5); j = (it - I_O0) & 31; j = ((j >> 2) << 3) + 4 + (j & 3); }
        d.flag = F_UF + (s * 8 + c) * 16 + (j >> 3); d.sig = ((j < 64 && (j & 4)) ? F_INO : F_INCNT) + s * 8 + c;
        if (j < 64) { d.soff = OFF_A + (unsigned)(8 * s + c) * SLAB + (unsigned)j * 8192u; d.doff = d.soff; }
        else { const unsigned off = (unsigned)(j - 64) * 8192u; d.soff = OFF_W + (unsigned)(rank * 8 + c) * SLAB + off; d.doff = OFF_B + (unsigned)(8 * s + c) * SLAB + off; }
    } else if (it < 1024) {
        const int c = (it - 960) >> 3, j = (it - 960) & 7; const unsigned off = 28u * IMG + (unsigned)j * 8192u;
        d.flag = F_UF + (s * 8 + c) * 16 + 15; d.sig = F_INB + s;
        d.soff = OFF_W + (unsigned)(rank * 8 + c) * SLAB + off; d.doff = OFF_B + (unsigned)(8 * s + c) * SLAB + off;
    } else if (it < 1472) {
        const int idx = (it - 1024) >> 4, j = (it - 1024) & 15, pml = idx / 14, pn = idx - pml * 14;
        d.kind = 1; d.flag = F_OUT + s * 64 + idx; d.half = j >> 3;
        d.soff = OFF_O + (unsigned)((2 * rank + pml) * 16 + pn) * 131072u + (unsigned)j * 8192u;
        d.doff = (unsigned)(pml * 256 + j * 16) * 65536u + (unsigned)(s * 4096 + pn * 256) * 2u;
    } else {
        const int idx = (it - 1472) >> 2, j = (it - 1472) & 3, al = idx >> 2, b = 28 + (idx & 3);
        d.kind = 2; d.flag = F_OUT + s * 64 + 28 + idx;
        d.soff = OFF_O + (unsigned)((2 * rank + (al >> 1)) * 16 + (b >> 1)) * 131072u + (unsigned)((al & 1) * 2 + (b & 1)) * 32768u + (unsigned)j * 8192u;
        d.doff = (unsigned)(al * 128 + j * 32) * 65536u + (unsigned)(s * 4096 + b * 128) * 2u;
    }
    return d;
}
#define STR2(x) #x
#define STR(x) STR2(x)
typedef int i32x4 __attribute__((ext_vector_type(4)));
__device__ __forceinline__ i32x4 mk_rsrc(const void* ptr, unsigned bytes) {
    const unsigned long long a = (unsigned long long)ptr; i32x4 r;
    r.x = __builtin_amdgcn_readfirstlane((int)(unsigned)a); r.y = __builtin_amdgcn_readfirstlane((int)(unsigned)(a >> 32) & 0xffff); r.z = (int)bytes; r.w = 0x00020000; return r;
}
#define RING_CLOB "v200","v201","v202","v203","v204","v205","v206","v207","v208","v209","v210","v211","v212","v213","v214","v215"
#define R_LD(r, off_) asm volatile("buffer_load_dwordx4 v[%c2:%c3], %0, %1, 0 offen sc0 sc1" :: "v"(off_), "s"(rsrc), "i"(200 + 4 * (r)), "i"(203 + 4 * (r)) : "memory", RING_CLOB)
#define R_ST(r, off_, rs_, wait, bits) asm volatile(wait "buffer_store_dwordx4 v[%c2:%c3], %0, %1, 0 offen" bits "\n\ts_nop 1" :: "v"(off_), "s"(rs_), "i"(200 + 4 * (r)), "i"(203 + 4 * (r)) : "memory", RING_CLOB)
#define R_NAN(r, wait) asm volatile(wait "v_mov_b32 v%c0, 0x7fc07fc0\n\tv_mov_b32 v%c1, 0x7fc07fc0\n\tv_mov_b32 v%c2, 0x7fc07fc0\n\tv_mov_b32 v%c3, 0x7fc07fc0" \
        :: "i"(200 + 4 * (r)), "i"(201 + 4 * (r)), "i"(202 + 4 * (r)), "i"(203 + 4 * (r)) : "memory", RING_CLOB)
__device__ __forceinline__ void puller(const Params& p, unsigned call, int l) {
    const int lane = threadIdx.x & 63, rank = p.rank, s = (rank + 1 + l) & 7;
    unsigned* fl = (unsigned*)p.ws[rank]; unsigned* tmo = fl + F_TMO * 32; unsigned* qctr = fl + (F_Q + l) * 32;
    const unsigned qbase = (call - 1u) * (unsigned)NFETCH;
    const i32x4 rsrc = mk_rsrc(p.ws[s], WS_NEED), rdst = mk_rsrc(p.ws[rank], WS_NEED), rout = mk_rsrc(p.out, OUT_BYTES);
    const unsigned l0 = (unsigned)lane * 16u, l1 = (unsigned)(lane >> 5) * 65536u + (unsigned)(lane & 31) * 16u, l2 = (unsigned)(lane >> 4) * 65536u + (unsigned)(lane & 15) * 16u;
    int cur, nxt;
    { unsigned t = 0; if (lane == 0) t = __hip_atomic_fetch_add(qctr, 1u, __ATOMIC_RELAXED, __HIP_MEMORY_SCOPE_AGENT); cur = (int)(__builtin_amdgcn_readfirstlane(t) - qbase); nxt = NITEM;
      if (cur < NITEM) { if (lane == 0) t = __hip_atomic_fetch_add(qctr, 1u, __ATOMIC_RELAXED, __HIP_MEMORY_SCOPE_AGENT); nxt = (int)(__builtin_amdgcn_readfirstlane(t) - qbase); } }
    bool cur_ready = false, have_prev = false, pbad = false, bad = false; int lastph = -1; (void)lastph;
    unsigned pdoff = 0, plane = 0, pstep = 0; int pkind = 0, psig = -1;
#define P_STORE(r, j, doff_, kind_, lane_, step_, bad_, wait) do { const unsigned o_ = (doff_) + (lane_) + (unsigned)(j) * (step_); \
        if ((kind_) == 0) R_ST(r, o_, rdst, wait, " sc1"); else { if (bad_) R_NAN(r, wait); R_ST(r, o_, rout, wait, ""); } } while (0)
#define P_SIG(sg) do { if ((sg) >= 0 && lane == 0) __hip_atomic_fetch_add(fl + (sg) * 32, 1u, __ATOMIC_RELAXED, __HIP_MEMORY_SCOPE_AGENT); } while (0)
#define P_FLUSH() do { if (have_prev) { WAIT_V(0); _Pragma("unroll") for (int r = 0; r < 4; ++r) P_STORE(r, 4 + r, pdoff, pkind, plane, pstep, pbad, ""); \
            WAIT_V(0); P_SIG(psig); have_prev = false; } } while (0)
    while (cur < NITEM) {
        const Desc dc = puller_desc(cur, rank, s);
        if (!cur_ready) {
            P_FLUSH();
            if (lane == 0) wait_ge(fl + dc.flag * 32, dc.half < 0 ? call : 2u * call - 1u + (unsigned)dc.half, tmo, call);
            bad = __builtin_amdgcn_readfirstlane(__hip_atomic_load(tmo, __ATOMIC_RELAXED, __HIP_MEMORY_SCOPE_AGENT)) == call;
        }
        unsigned ftgt = call;
        if (nxt < NITEM) {
            const Desc dn = puller_desc(nxt, rank, s);
            if (dn.half >= 0) ftgt = 2u * call - 1u + (unsigned)dn.half;
            asm volatile("v_mov_b32 v216, -1\n\tv_mov_b32 v217, -1\n\ts_nop 0\n\tglobal_load_dword v216, %0, off sc0 sc1" :: "v"(fl + dn.flag * 32) : "memory", "v216", "v217");
            if (lane == 0) asm volatile("global_atomic_add v217, %0, %1, off sc0" :: "v"(qctr), "v"(1u) : "memory", "v217");
        }
        const unsigned clane = dc.kind == 0 ? l0 : (dc.kind == 1 ? l1 : l2), cstep = dc.kind == 0 ? 1024u : (dc.kind == 1 ? 131072u : 262144u);
#pragma unroll
        for (int j = 0; j < 8; ++j) {
            const int r = j & 3;
            if (j >= 4) P_STORE(r, j - 4, dc.doff, dc.kind, clane, cstep, bad, "s_waitcnt vmcnt(3)\n\t");
            else if (have_prev) P_STORE(r, 4 + j, pdoff, pkind, plane, pstep, pbad, "s_waitcnt vmcnt(3)\n\t");
            R_LD(r, dc.soff + l0 + (unsigned)j * 1024u);
        }
        if (have_prev) P_SIG(psig);
        unsigned fv, tk;
        asm volatile("v_mov_b32 %0, v216\n\tv_mov_b32 %1, v217" : "=&v"(fv), "=&v"(tk) :: "memory", "v216", "v217");
        if (nxt < NITEM && (__builtin_amdgcn_readfirstlane(fv) == -1 || __builtin_amdgcn_readfirstlane(tk) == -1))
            asm volatile("s_waitcnt vmcnt(0)\n\tv_mov_b32 %0, v216\n\tv_mov_b32 %1, v217" : "=&v"(fv), "=&v"(tk) :: "memory", "v216", "v217");
        have_prev = true; pdoff = dc.doff; pkind = dc.kind; plane = clane; pstep = cstep; psig = dc.sig; pbad = bad;
        cur_ready = nxt < NITEM && (unsigned)__builtin_amdgcn_readfirstlane(fv) >= ftgt;
        const int nn = nxt < NITEM ? (int)(__builtin_amdgcn_readfirstlane(tk) - qbase) : NITEM;
        if ((cur >= I_O0 - 32 && cur < I_O0) || (cur >= 960 - 32 && cur < 960) || (cur >= 1024 - 32 && cur < 1024)) P_FLUSH();
        cur = nxt; nxt = nn;
    }
    P_FLUSH();
}

__device__ __forceinline__ unsigned long long ld_sum(const unsigned* fl, int w, unsigned call) {
    const unsigned long long v = __hip_atomic_load((const unsigned long long*)(fl + F_SUM * 32) + w, __ATOMIC_RELAXED, __HIP_MEMORY_SCOPE_AGENT);
    return (unsigned)(v >> 56) == (call & 0xffu) ? (v & 0x00ffffffffffffffull) : 0ull;
}
__device__ __forceinline__ void monitor(const Params& p, unsigned call) {
    const int lane = threadIdx.x & 63, rank = p.rank;
    unsigned* fl = (unsigned*)p.ws[rank]; unsigned* tmo = fl + F_TMO * 32;
    unsigned long long* sum = (unsigned long long*)(fl + F_SUM * 32);
    const int c = lane / 7, l = lane - c * 7, s = (rank + 1 + l) & 7;
    const unsigned* pe = fl + (F_INCNT + s * 8 + (c & 7)) * 32; const unsigned* po = fl + (F_INO + s * 8 + (c & 7)) * 32; const unsigned* pb = fl + (F_INB + s) * 32;
    const unsigned long long tag = (unsigned long long)(call & 0xffu) << 56, full = 0x00ffffffffffffffull;
    unsigned long long me = 0, mo = 0, mb = 0;
    const unsigned long long t0 = __builtin_amdgcn_s_memrealtime();
    for (;;) {
        const int fe = (int)__builtin_ctzll(~me), fo = (int)__builtin_ctzll(~mo);
        const bool ae = lane < 56 && !((me >> lane) & 1ull) && lane < fe + 14, ao = lane < 56 && !((mo >> lane) & 1ull) && lane < fo + 14 && ((me >> lane) & 1ull);
        const bool ab = lane < 7 && !((mb >> lane) & 1ull) && me == full;
        const bool be = ae && ld_sys(pe) >= INE_ITEMS * call, bo = ao && ld_sys(po) >= INO_ITEMS * call, bb = ab && ld_sys(pb) >= INB_ITEMS * call;
        const unsigned long long ne = me | __ballot(be), no = mo | __ballot(bo), nb = mb | __ballot(bb);
        if (lane == 0) {
            if (ne != me) __hip_atomic_store(sum + 0, ne | tag, __ATOMIC_RELAXED, __HIP_MEMORY_SCOPE_AGENT);
            if (no != mo) __hip_atomic_store(sum + 1, no | tag, __ATOMIC_RELAXED, __HIP_MEMORY_SCOPE_AGENT);
            if (nb != mb) __hip_atomic_store(sum + 2, nb | tag, __ATOMIC_RELAXED, __HIP_MEMORY_SCOPE_AGENT);
        }
        me = ne; mo = no; mb = nb;
        if (me == full && mo == full && mb == 0x7full) break;
        if (__builtin_amdgcn_readfirstlane(__hip_atomic_load(tmo, __ATOMIC_RELAXED, __HIP_MEMORY_SCOPE_AGENT)) == call || __builtin_amdgcn_s_memrealtime() - t0 > DEADLINE) {
            if (lane == 0) __hip_atomic_store(tmo, call, __ATOMIC_RELAXED, __HIP_MEMORY_SCOPE_AGENT); break; }
        __builtin_amdgcn_s_sleep(MON_SLEEP);
    }
}

__device__ __forceinline__ int kt_of(int t, int rank) {
    if (t < 8) return rank * 8 + t;
    const int tt = t - 8, c = tt / 7, l = tt - c * 7; return ((rank + 1 + l) & 7) * 8 + c;
}
__device__ __forceinline__ void store_tile_rows(const f32x4& v0, const f32x4& v1, bool bad, u32x4& w) {
    w.x = gelu2_bf16(v0[0], v0[1]); w.y = gelu2_bf16(v0[2], v0[3]); w.z = gelu2_bf16(v1[0], v1[1]); w.w = gelu2_bf16(v1[2], v1[3]);
    if (bad) w = (u32x4){0x7fc07fc0u, 0x7fc07fc0u, 0x7fc07fc0u, 0x7fc07fc0u};
}

__device__ __forceinline__ void gemm_r1(const Params& p, LAS unsigned char* lds, unsigned call, int pm, int pn) {
    const int tid = threadIdx.x, wid = __builtin_amdgcn_readfirstlane(tid >> 6), lane = tid & 63, wr = wid >> 2, wc = wid & 3, fr = lane & 15, fq = lane >> 4;
    const int rank = p.rank;
    unsigned char* wsl = p.ws[rank]; unsigned* fl = (unsigned*)wsl; unsigned* tmo = fl + F_TMO * 32;
    volatile LAS unsigned* lready = (volatile LAS unsigned*)(lds + STAGE_BYTES);
    const char* gA = (const char*)wsl + OFF_A + (size_t)pm * 2 * IMG + tid * 16;
    const char* gB = (const char*)wsl + OFF_B + (size_t)pn * 2 * IMG + tid * 16;
    const unsigned ldsw = (unsigned)wid * 1024u;
    const int aoff = lds_byte(wr * 64 + fr, fq * 8), boff = lds_byte(wc * 32 + fr, fq * 8);
    const unsigned* myflag = fl + (F_SFLAG + rank * 8 + (lane & 7)) * 32;
    int ready = 0;
#define SA(b, h) (((b) * 2 + (h)) * HTB)
#define SB(b, h) ((4 + (b) * 2 + (h)) * HTB)
#define STAGE(bufoff, gptr) do { _Pragma("unroll") for (int _i = 0; _i < 2; ++_i) \
        __builtin_amdgcn_global_load_lds((const unsigned*)((gptr) + _i * 8192), (LAS unsigned*)(lds + (bufoff) + ldsw + _i * 8192), 16, 0, 0); } while (0)
#define LDA(dst, b, h) do { _Pragma("unroll") for (int m = 0; m < 4; ++m) _Pragma("unroll") for (int k = 0; k < 2; ++k) dst[m][k] = *(const LAS bf16x8*)(lds + SA(b, h) + aoff + m * 2048 + k * 1024); } while (0)
#define LDB(dst, b, h) do { _Pragma("unroll") for (int n = 0; n < 2; ++n) _Pragma("unroll") for (int k = 0; k < 2; ++k) dst[n][k] = *(const LAS bf16x8*)(lds + SB(b, h) + boff + n * 2048 + k * 1024); } while (0)
#define MMA(ai, bj, At, Bt) do { __builtin_amdgcn_s_setprio(1); _Pragma("unroll") for (int m = 0; m < 4; ++m) _Pragma("unroll") for (int n = 0; n < 2; ++n) _Pragma("unroll") for (int k = 0; k < 2; ++k) \
        acc[ai][bj][m][n] = __builtin_amdgcn_mfma_f32_16x16x32_bf16(Bt[n][k], At[m][k], acc[ai][bj][m][n], 0, 0, 0); __builtin_amdgcn_s_setprio(0); } while (0)
    volatile LAS int* lorder = (volatile LAS int*)(lds + STAGE_BYTES + 64);
    unsigned long long sched = 0ull;
    const int mykt = kt_of(lane, rank);
#define NEED(nd) do { if ((nd) > ready) { \
        if (wid == 0) { const unsigned long long t0 = __builtin_amdgcn_s_memrealtime(); int cnt = ready; \
            for (;;) { \
                const unsigned long long sm = ld_sum(fl, pm & 1, call) << 8; \
                bool ok = !((sched >> lane) & 1ull) && (lane < 8 ? ld_sys(myflag) >= call : ((sm >> lane) & 1ull) != 0ull); \
                bool giveup = false; \
                if (cnt + __builtin_popcountll(__ballot(ok)) < (nd)) { \
                    giveup = __builtin_amdgcn_readfirstlane(__hip_atomic_load(tmo, __ATOMIC_RELAXED, __HIP_MEMORY_SCOPE_AGENT)) == call || __builtin_amdgcn_s_memrealtime() - t0 > DEADLINE; \
                    if (giveup) { if (lane == 0) { __hip_atomic_store(tmo, call, __ATOMIC_RELAXED, __HIP_MEMORY_SCOPE_AGENT); lready[1] = 1u; } ok = !((sched >> lane) & 1ull); } } \
                const unsigned long long bal = __ballot(ok); \
                if (ok) lorder[cnt + __builtin_popcountll(bal & ((1ull << lane) - 1ull))] = mykt; \
                sched |= bal; cnt += __builtin_popcountll(bal); \
                if (cnt >= (nd)) break; \
                if ((nd) > 10 && (nd) < ((pm & 1) ? 64 - 7 * ND + ODD_FAST : POLL_FAST_FROM)) __builtin_amdgcn_s_sleep(SLOW_SLEEP); else __builtin_amdgcn_s_sleep(POLL_SLEEP); } \
            ACQ_AGENT; \
            if (lane == 0) *lready = (unsigned)cnt; \
            asm volatile("s_waitcnt vmcnt(0) lgkmcnt(0)" ::: "memory"); } \
        CFENCE; BAR; CFENCE; \
        ready = __builtin_amdgcn_readfirstlane((int)*lready); } } while (0)
#define KOFF(t) ((size_t)(unsigned)__builtin_amdgcn_readfirstlane(lorder[t]) * SLAB)

    f32x4 acc[2][2][4][2];
#pragma unroll
    for (int a = 0; a < 2; ++a)
#pragma unroll
        for (int b = 0; b < 2; ++b)
#pragma unroll
            for (int m = 0; m < 4; ++m)
#pragma unroll
                for (int n = 0; n < 2; ++n) acc[a][b][m][n] = (f32x4){0.f, 0.f, 0.f, 0.f};
    bf16x8 At[4][2], B0[2][2], B1[2][2];
    if (tid == 0) lready[1] = 0u;
    NEED(4);
    size_t o1, o2, o3;
    {
        const size_t k0 = KOFF(0); o1 = KOFF(1); o2 = KOFF(2); o3 = KOFF(3);
        STAGE(SB(0, 0), gB + k0); STAGE(SA(0, 0), gA + k0); STAGE(SB(0, 1), gB + k0 + IMG); STAGE(SA(0, 1), gA + k0 + IMG);
        if (wr == 1) BAR;
        WAIT_V(4); BAR;
        STAGE(SB(1, 0), gB + o1); STAGE(SA(1, 0), gA + o1); STAGE(SB(1, 1), gB + o1 + IMG);
        WAIT_V(6); BAR;
    }
    STAMP(2);
    for (int t = 0; t < 64; t += 2) {
        LDB(B0, 0, 0); SCHED; LDA(At, 0, 0); STAGE(SA(1, 1), gA + o1 + IMG);
        WAIT_L(8); BAR; WAIT_L(0); MMA(0, 0, At, B0); BAR; SCHED;
        LDB(B1, 0, 1); STAGE(SB(0, 0), gB + o2);
        BAR; WAIT_L(0); MMA(0, 1, At, B1); BAR;
        LDA(At, 0, 1); STAGE(SA(0, 0), gA + o2);
        BAR; WAIT_L(0); MMA(1, 0, At, B0); BAR; SCHED;
        STAGE(SB(0, 1), gB + o2 + IMG);
        WAIT_V(6); BAR; MMA(1, 1, At, B1); BAR;
        LDB(B0, 1, 0); SCHED; LDA(At, 1, 0); STAGE(SA(0, 1), gA + o2 + IMG);
        WAIT_L(8); BAR; WAIT_L(0); MMA(0, 0, At, B0); BAR; SCHED;
        LDB(B1, 1, 1); STAGE(SB(1, 0), gB + o3);
        BAR; WAIT_L(0); MMA(0, 1, At, B1); BAR;
        LDA(At, 1, 1); STAGE(SA(1, 0), gA + o3);
        BAR; WAIT_L(0); MMA(1, 0, At, B0); BAR; SCHED;
        STAGE(SB(1, 1), gB + o3 + IMG);
        WAIT_V(6); BAR; MMA(1, 1, At, B1); BAR;
        o1 = o3;
        if (t + 6 <= 64) { NEED(t + 6); o2 = KOFF(t + 4); o3 = KOFF(t + 5); }
    }
    STAMP(3);
    {
        int tz = threadIdx.x; asm volatile("" : "+v"(tz));
        const int lane2 = tz & 63, fr2 = lane2 & 15, fq2 = lane2 >> 4;
        const bool bad = lready[1] != 0u;
        const int owner = pm >> 1;
        if (wr == 0) BAR;
        if (owner == rank) {
            const auto rout = __builtin_amdgcn_make_buffer_rsrc(p.out, 0, (int)OUT_BYTES, 0x00020000);
            const unsigned base = (unsigned)((pm & 1) * 256 + wr * 64 + fr2) * 65536u + (unsigned)(rank * 4096 + pn * 256 + wc * 32 + fq2 * 8) * 2u;
#pragma unroll
            for (int ai = 0; ai < 2; ++ai)
#pragma unroll
                for (int m = 0; m < 4; ++m)
#pragma unroll
                    for (int bj = 0; bj < 2; ++bj) { u32x4 w; store_tile_rows(acc[ai][bj][m][0], acc[ai][bj][m][1], bad, w);
                        __builtin_amdgcn_raw_buffer_store_b128(w, rout, (int)(base + (unsigned)(ai * 128 + m * 16) * 65536u + (unsigned)bj * 256u), 0, 0); }
            WAIT_V(0); BAR; CFENCE;
        } else {
            const auto rs = __builtin_amdgcn_make_buffer_rsrc(wsl, 0, (int)WS_NEED, 0x00020000);
            const unsigned base = OFF_O + (unsigned)(pm * 16 + pn) * 131072u + (unsigned)(wr * 64 + fr2) * 512u + (unsigned)(wc * 32 + fq2 * 8) * 2u;
            unsigned* oflag = (unsigned*)p.ws[owner] + (F_OUT + rank * 64 + (pm & 1) * 14 + pn) * 32;
#pragma unroll
            for (int ai = 0; ai < 2; ++ai) {
#pragma unroll
                for (int m = 0; m < 4; ++m)
#pragma unroll
                    for (int bj = 0; bj < 2; ++bj) { u32x4 w; store_tile_rows(acc[ai][bj][m][0], acc[ai][bj][m][1], bad, w);
                        __builtin_amdgcn_raw_buffer_store_b128(w, rs, (int)(base + (unsigned)(ai * 128 + m * 16) * 512u + (unsigned)bj * 256u), 0, AUX_SYS); }
                WAIT_V(0); BAR; CFENCE;
                if (tz == 0) {
                    st_sys(oflag, 2u * call - 1u + (unsigned)ai);
                }
            }
        }
    }
#undef NEED
}

__device__ __forceinline__ void gemm_r2(const Params& p, LAS unsigned char* lds, unsigned call, int a, int b) {
    const int tid = threadIdx.x, wid = __builtin_amdgcn_readfirstlane(tid >> 6), lane = tid & 63, wr = wid >> 2, wc = wid & 3, fr = lane & 15, fq = lane >> 4;
    const int rank = p.rank;
    unsigned char* wsl = p.ws[rank]; unsigned* fl = (unsigned*)wsl; unsigned* tmo = fl + F_TMO * 32;
    if (tid < 64) {
        if (lane < 8) wait_ge(fl + (F_SFLAG + rank * 8 + lane) * 32, call, tmo, call);
        const unsigned long long t0 = __builtin_amdgcn_s_memrealtime();
        while (ld_sum(fl, (a >> 1) & 1, call) != 0x00ffffffffffffffull || ld_sum(fl, 2, call) != 0x7full) {
            if (__builtin_amdgcn_readfirstlane(__hip_atomic_load(tmo, __ATOMIC_RELAXED, __HIP_MEMORY_SCOPE_AGENT)) == call || __builtin_amdgcn_s_memrealtime() - t0 > DEADLINE) {
                if (lane == 0) __hip_atomic_store(tmo, call, __ATOMIC_RELAXED, __HIP_MEMORY_SCOPE_AGENT); break; }
            __builtin_amdgcn_s_sleep(16);
        }
        ACQ_AGENT; WAIT_V(0);
    }
    CFENCE; BAR; CFENCE;
    const char* gA = (const char*)wsl + OFF_A + (size_t)a * IMG + tid * 16;
    const char* gB = (const char*)wsl + OFF_B + (size_t)b * IMG + tid * 16;
    const unsigned ldsw = (unsigned)wid * 1024u;
    const int aoff = lds_byte(wr * 64 + fr, fq * 8), boff = lds_byte(wc * 32 + fr, fq * 8);
    f32x4 acc[4][2];
#pragma unroll
    for (int m = 0; m < 4; ++m)
#pragma unroll
        for (int n = 0; n < 2; ++n) acc[m][n] = (f32x4){0.f, 0.f, 0.f, 0.f};
#define STAGE2(slot, kt) do { STAGE((slot) * 32768, gA + (size_t)(kt) * SLAB); STAGE((slot) * 32768 + 16384, gB + (size_t)(kt) * SLAB); } while (0)
#define FRAG2(Af, Bf, slot) do { \
        _Pragma("unroll") for (int n = 0; n < 2; ++n) _Pragma("unroll") for (int k = 0; k < 2; ++k) Bf[n][k] = *(const LAS bf16x8*)(lds + (slot) * 32768 + 16384 + boff + n * 2048 + k * 1024); \
        _Pragma("unroll") for (int m = 0; m < 4; ++m) _Pragma("unroll") for (int k = 0; k < 2; ++k) Af[m][k] = *(const LAS bf16x8*)(lds + (slot) * 32768 + aoff + m * 2048 + k * 1024); } while (0)
#define MMA2(Af, Bf) do { __builtin_amdgcn_s_setprio(1); _Pragma("unroll") for (int k = 0; k < 2; ++k) _Pragma("unroll") for (int m = 0; m < 4; ++m) _Pragma("unroll") for (int n = 0; n < 2; ++n) \
        acc[m][n] = __builtin_amdgcn_mfma_f32_16x16x32_bf16(Bf[n][k], Af[m][k], acc[m][n], 0, 0, 0); __builtin_amdgcn_s_setprio(0); } while (0)
    bf16x8 A0[4][2], Bf0[2][2], A1[4][2], Bf1[2][2];
    STAMP(6);
    STAGE2(0, 0); STAGE2(1, 1); STAGE2(2, 2); STAGE2(3, 3);
    WAIT_V(12); BAR; CFENCE;
    FRAG2(A0, Bf0, 0);
    for (int t = 0; t < 64; t += 4) {
        __builtin_amdgcn_s_waitcnt(0xC07F); WAIT_V(8); BAR; SCHED; STAGE2(0, (t + 4) & 63); FRAG2(A1, Bf1, 1); SCHED; MMA2(A0, Bf0); SCHED;
        __builtin_amdgcn_s_waitcnt(0xC07F); WAIT_V(8); BAR; SCHED; STAGE2(1, (t + 5) & 63); FRAG2(A0, Bf0, 2); SCHED; MMA2(A1, Bf1); SCHED;
        __builtin_amdgcn_s_waitcnt(0xC07F); WAIT_V(8); BAR; SCHED; STAGE2(2, (t + 6) & 63); FRAG2(A1, Bf1, 3); SCHED; MMA2(A0, Bf0); SCHED;
        __builtin_amdgcn_s_waitcnt(0xC07F); WAIT_V(8); BAR; SCHED; STAGE2(3, (t + 7) & 63); FRAG2(A0, Bf0, 0); SCHED; MMA2(A1, Bf1); SCHED;
    }
    WAIT_L(0);
    STAMP(7);
    const bool bad = __hip_atomic_load(tmo, __ATOMIC_RELAXED, __HIP_MEMORY_SCOPE_AGENT) == call;
    const int owner = a >> 2;
    if (owner == rank) {
        const auto rout = __builtin_amdgcn_make_buffer_rsrc(p.out, 0, (int)OUT_BYTES, 0x00020000);
        const unsigned base = (unsigned)((a & 3) * 128 + wr * 64 + fr) * 65536u + (unsigned)(rank * 4096 + b * 128 + wc * 32 + fq * 8) * 2u;
#pragma unroll
        for (int m = 0; m < 4; ++m) { u32x4 w; store_tile_rows(acc[m][0], acc[m][1], bad, w);
            __builtin_amdgcn_raw_buffer_store_b128(w, rout, (int)(base + (unsigned)(m * 16) * 65536u), 0, 0); }
    } else {
        const auto rs = __builtin_amdgcn_make_buffer_rsrc(wsl, 0, (int)WS_NEED, 0x00020000);
        const unsigned base = OFF_O + (unsigned)((a >> 1) * 16 + (b >> 1)) * 131072u + (unsigned)((a & 1) * 2 + (b & 1)) * 32768u + (unsigned)(wr * 64 + fr) * 256u + (unsigned)(wc * 32 + fq * 8) * 2u;
#pragma unroll
        for (int m = 0; m < 4; ++m) { u32x4 w; store_tile_rows(acc[m][0], acc[m][1], bad, w);
            __builtin_amdgcn_raw_buffer_store_b128(w, rs, (int)(base + (unsigned)(m * 16) * 256u), 0, AUX_SYS); }
    }
    WAIT_V(0); BAR; CFENCE;
    if (owner != rank && tid == 0) {
        st_sys((unsigned*)p.ws[owner] + (F_OUT + rank * 64 + 28 + (a & 3) * 4 + (b - 28)) * 32, call);
    }
}

__global__ __launch_bounds__(512, 2) void fused_gemm_a2a(Params p) {
    extern __shared__ __attribute__((aligned(16))) unsigned char shm[];
    LAS unsigned char* lds = (LAS unsigned char*)shm;
    const int tid = threadIdx.x, bid = blockIdx.x;
    unsigned* fl = (unsigned*)p.ws[p.rank];
    volatile LAS unsigned* lcall = (volatile LAS unsigned*)(lds + STAGE_BYTES + 16);
    if (tid == 0) *lcall = __hip_atomic_fetch_add(fl + F_TICKET * 32, 1u, __ATOMIC_RELAXED, __HIP_MEMORY_SCOPE_AGENT) / (unsigned)NWG + 1u;
    STAMP(0);
    if (bid >= NPULL) stage_units(p, lds, lcall, bid - NPULL);
    __syncthreads();
    const unsigned call = __builtin_amdgcn_readfirstlane(*lcall);
    if (bid < NPULL) { puller(p, call, bid % 7); STAMP(19); return; }
    STAMP(1);
    __syncthreads();
    const int xcd = bid & 7, idx = (bid >> 3) - (xcd < 4 ? 4 : 3), rg = xcd >> 1, cg = xcd & 1;
    const int n1 = 28, nw = xcd < 4 ? 28 : 29;
    if (idx < n1) {
        gemm_r1(p, lds, call, rg * 4 + (idx & 3), cg * 7 + (idx >> 2));
        STAMP(4);
    } else if (bid == 252 && tid < 64) monitor(p, call);
    volatile LAS unsigned* lq = (volatile LAS unsigned*)(lds + STAGE_BYTES + 32);
    for (;;) {
        __syncthreads();
        if (tid == 0) *lq = __hip_atomic_fetch_add(fl + (F_Q2 + xcd) * 32, 1u, __ATOMIC_RELAXED, __HIP_MEMORY_SCOPE_AGENT) - (call - 1u) * (unsigned)(16 + nw);
        __syncthreads();
        const unsigned r2 = __builtin_amdgcn_readfirstlane(*lq);
        if (r2 >= 16u) break;
        gemm_r2(p, lds, call, rg * 8 + (int)(r2 & 7u), 28 + cg * 2 + (int)(r2 >> 3));
        STAMP(5);
    }
}
}

extern "C" void kernel_launch(int rank, int nranks, ncclComm_t comm,
                              hipStream_t stream,
                              void* const* d_in, const int* in_sizes, int n_in,
                              void* d_out, int out_size,
                              const int* in_dtypes, int out_dtype,
                              void* const* d_out_all,
                              void* const* d_ws_all, size_t ws_bytes) {
    if (nranks != NR || n_in != 2 || ws_bytes < (size_t)WS_NEED || (unsigned)rank >= (unsigned)NR) { fprintf(stderr, "kernel_launch: unexpected configuration\n"); return; }
    Params p;
    p.x = (const unsigned short*)d_in[0]; p.w = (const unsigned short*)d_in[1]; p.out = (unsigned short*)d_out; p.rank = rank;
    for (int i = 0; i < NR; ++i) p.ws[i] = (unsigned char*)d_ws_all[i];
    HIP_CHECK(hipFuncSetAttribute((const void*)fused_gemm_a2a, hipFuncAttributeMaxDynamicSharedMemorySize, LDS_BYTES));
    hipLaunchKernelGGL(fused_gemm_a2a, dim3(NWG), dim3(512), LDS_BYTES, stream, p);
    HIP_CHECK(hipGetLastError());
}
```

```cpp
#include <hip/hip_runtime.h>
#include <rccl/rccl.h>
#include <cstdio>
#include <cstdint>

#define HIP_CHECK(expr) do { hipError_t _e = (expr); if (_e != hipSuccess) { \
    fprintf(stderr, "HIP error %s at %s:%d\n", hipGetErrorString(_e), __FILE__, __LINE__); \
    return; } } while (0)

namespace {
typedef unsigned u32x4 __attribute__((ext_vector_type(4)));
typedef short bf16x8 __attribute__((ext_vector_type(8)));
typedef float f32x4 __attribute__((ext_vector_type(4)));
#define LAS __attribute__((address_space(3)))

constexpr int NR = 8;
constexpr int MM = 4096, KS = 512, NN = 32768, NS = 4096, MS = 512;
constexpr unsigned IMG = 16384;
constexpr unsigned SLAB = 32 * IMG;
constexpr unsigned OFF_A = 1u << 20;
constexpr unsigned OFF_B = OFF_A + 64 * SLAB;
constexpr unsigned OFF_W = OFF_B + 64 * SLAB;
constexpr unsigned OFF_O = OFF_W + 64 * SLAB;
constexpr unsigned WS_NEED = OFF_O + 256u * 131072u;
constexpr unsigned OUT_BYTES = (unsigned)MS * NN * 2u;
constexpr int F_TICKET = 0, F_TMO = 1, F_SCNT = 2, F_SFLAG = 16, F_INCNT = 80, F_INB = 144, F_Q = 152, F_OUT = 160  , F_INO = 672, F_Q2 = 736, F_SUM = 744, F_XF = 768  , F_UF = 1024  ;
constexpr int NPULL = 28, NWG = 256;
constexpr int NSTAGER = NWG - NPULL;
constexpr int UNITS_PER_CHUNK = 72, NUNITS = 8 * UNITS_PER_CHUNK;
constexpr unsigned long long DEADLINE = 50000000ull;
constexpr int HTB = 16384, STAGE_BYTES = 8 * HTB, LDS_BYTES = STAGE_BYTES + 512;
constexpr int AUX_SYS = 17, AUX_AGENT = 16;
#define MON_SLEEP 8
#define ODD_FAST 6
#define SLOW_SLEEP 127
#define POLL_FAST_FROM 56
#define POLL_SLEEP 16

struct Params { const unsigned short* x; const unsigned short* w; unsigned char* ws[NR]; unsigned short* out; int rank; };

__device__ __forceinline__ int lds_byte(int r, int c) { const int st = (r >> 4) * 2 + (c >> 5), rr = r & 15, cc = c & 31, ob = rr * 64 + cc * 2; return st * 1024 + (ob ^ (((ob >> 9) & 1) << 5)); }
__device__ __forceinline__ void stage_rc(int b, int& R, int& C) { const int st = b / 1024, sb = b % 1024, swz = sb ^ (((sb >> 9) & 1) << 5); R = (st >> 1) * 16 + swz / 64; C = (st & 1) * 32 + (swz % 64) / 2; }

__device__ __forceinline__ unsigned ld_sys(const unsigned* p) { return __hip_atomic_load(p, __ATOMIC_RELAXED, __HIP_MEMORY_SCOPE_SYSTEM); }
__device__ __forceinline__ void st_sys(unsigned* p, unsigned v) { __hip_atomic_store(p, v, __ATOMIC_RELAXED, __HIP_MEMORY_SCOPE_SYSTEM); }
__device__ __forceinline__ void wait_ge(const unsigned* flag, unsigned val, unsigned* tmo, unsigned call) {
    const unsigned long long t0 = __builtin_amdgcn_s_memrealtime();
    while (ld_sys(flag) < val) {
        if (__hip_atomic_load(tmo, __ATOMIC_RELAXED, __HIP_MEMORY_SCOPE_AGENT) == call || __builtin_amdgcn_s_memrealtime() - t0 > DEADLINE) {
            __hip_atomic_store(tmo, call, __ATOMIC_RELAXED, __HIP_MEMORY_SCOPE_AGENT); break; }
        __builtin_amdgcn_s_sleep(2);
    }
}
__device__ __forceinline__ unsigned cvt_pk_bf16(float lo, float hi) { unsigned r; asm volatile("v_cvt_pk_bf16_f32 %0, %1, %2" : "=v"(r) : "v"(lo), "v"(hi)); return r; }
typedef float f32x2 __attribute__((ext_vector_type(2)));
__device__ __forceinline__ unsigned gelu2_bf16(float y0, float y1) {
    const f32x2 y = {y0, y1};
    const f32x2 t = (y * y) * -0.1029432395800235f + -2.302208198144325f;
    const f32x2 a = y * t;
    f32x2 d; d.x = __builtin_amdgcn_exp2f(a.x); d.y = __builtin_amdgcn_exp2f(a.y);
    d = d + 1.0f;
    f32x2 r; r.x = __builtin_amdgcn_rcpf(d.x); r.y = __builtin_amdgcn_rcpf(d.y);
    r = r * y;
    return cvt_pk_bf16(r.x, r.y);
}
#define STAMP(i) do { } while (0)
#define WAIT_V(n) asm volatile("s_waitcnt vmcnt(" #n ")" ::: "memory")
#define WAIT_L(n) asm volatile("s_waitcnt lgkmcnt(" #n ")" ::: "memory")
#define BAR __builtin_amdgcn_s_barrier()
#define SCHED __builtin_amdgcn_sched_barrier(0)
#define CFENCE asm volatile("" ::: "memory")
#define ACQ_AGENT CFENCE

__device__ __forceinline__ void stage_units(const Params& p, LAS unsigned char* lds, volatile LAS unsigned* lcall, int first) {
    const int tid = threadIdx.x, rank = p.rank;
    unsigned char* wsl = p.ws[rank]; unsigned* fl = (unsigned*)wsl;
    const auto rs = __builtin_amdgcn_make_buffer_rsrc(wsl, 0, (int)WS_NEED, 0x00020000);
    if (first < 32) {
        const unsigned dbase = OFF_A + (unsigned)(rank * 8) * SLAB + (unsigned)first * IMG;
        u32x4 t[2];
#pragma unroll
        for (int i = 0; i < 2; ++i) { int R, C; stage_rc((tid + 512 * i) * 16, R, C); t[i] = *(const u32x4*)(p.x + (size_t)(first * 128 + R) * KS + C); }
#pragma unroll
        for (int i = 0; i < 2; ++i) __builtin_amdgcn_raw_buffer_store_b128(t[i], rs, (int)(dbase + (unsigned)(tid + 512 * i) * 16u), 0, AUX_SYS);
        WAIT_V(0);
        __syncthreads();
        const unsigned call = *lcall;
        if (tid < NR) { if (tid != rank) st_sys((unsigned*)p.ws[tid] + (F_XF + rank * 32 + first) * 32, call); }
        else if (tid == 64) {
            const unsigned old = __hip_atomic_fetch_add(fl + F_SCNT * 32, 1u, __ATOMIC_RELAXED, __HIP_MEMORY_SCOPE_AGENT);
            if (old + 1u == call * 96u) st_sys(fl + (F_SFLAG + rank * 8) * 32, call);
        }
    }
    for (int u = 8 + (first + NSTAGER - 32) % NSTAGER; u < NUNITS; u += NSTAGER) {
        const int c = u / UNITS_PER_CHUNK, v0 = u % UNITS_PER_CHUNK, v = v0 < 8 ? 64 + v0 : v0 - 8;
        if (v >= 64) {
            const int vv = v - 64;
            const unsigned dbase = OFF_A + (unsigned)(rank * 8 + c) * SLAB + (unsigned)vv * 4u * IMG;
            u32x4 t[8];
#pragma unroll
            for (int i = 0; i < 8; ++i) { const int pc = tid + 512 * i, img = pc >> 10; int R, C; stage_rc((pc & 1023) * 16, R, C);
                t[i] = *(const u32x4*)(p.x + (size_t)(vv * 512 + img * 128 + R) * KS + c * 64 + C); }
#pragma unroll
            for (int i = 0; i < 8; ++i) __builtin_amdgcn_raw_buffer_store_b128(t[i], rs, (int)(dbase + (unsigned)(tid + 512 * i) * 16u), 0, AUX_SYS);
        } else {
            const int d = v >> 3, kb = tid >> 6, nb = tid & 63;
            const unsigned dbase = (d == rank ? OFF_B + (unsigned)(rank * 8 + c) * SLAB : OFF_W + (unsigned)(d * 8 + c) * SLAB) + (unsigned)(v & 7) * 4u * IMG;
            u32x4 r[8];
#pragma unroll
            for (int i = 0; i < 8; ++i) r[i] = *(const u32x4*)(p.w + (size_t)(c * 64 + kb * 8 + i) * NN + v * 512 + nb * 8);
#pragma unroll
            for (int j = 0; j < 8; ++j) {
                u32x4 o;
#pragma unroll
                for (int e = 0; e < 4; ++e) o[e] = __builtin_amdgcn_perm(r[2 * e + 1][j >> 1], r[2 * e][j >> 1], (j & 1) ? 0x07060302u : 0x05040100u);
                const int nl = nb * 8 + j, img = nl >> 7, rowi = nl & 127, cc = rowi & 31;
                const int rho = (rowi & ~31) + 16 * ((cc >> 2) & 1) + 4 * (cc >> 3) + (cc & 3);
                *(LAS u32x4*)(lds + img * IMG + lds_byte(rho, kb * 8)) = o;
            }
            __syncthreads();
#pragma unroll
            for (int i = 0; i < 8; ++i) { const unsigned b = (unsigned)(tid + 512 * i) * 16u; const u32x4 t = *(const LAS u32x4*)(lds + b);
                __builtin_amdgcn_raw_buffer_store_b128(t, rs, (int)(dbase + b), 0, AUX_SYS); }
        }
        WAIT_V(0);
        __syncthreads();
        const unsigned call = *lcall;
        if (tid < NR) {
            if (tid != rank && (v >= 64 || tid == (v >> 3)))
                st_sys((unsigned*)p.ws[tid] + (F_UF + (rank * 8 + c) * 16 + (v >= 64 ? v - 64 : 8 + (v & 7))) * 32, call);
        } else if (tid == 64) {
            const unsigned old = __hip_atomic_fetch_add(fl + (F_SCNT + c) * 32, 1u, __ATOMIC_RELAXED, __HIP_MEMORY_SCOPE_AGENT);
            if (old + 1u == call * (c == 0 ? 96u : (unsigned)UNITS_PER_CHUNK)) st_sys(fl + (F_SFLAG + rank * 8 + c) * 32, call);
        }
    }
}

struct Desc { unsigned soff, doff; int kind; int flag; int sig; int half; };
constexpr int NITEM = 1536, NFETCH = NITEM - 64 + 32;
#define ND 4
constexpr int I_E0 = (8 - ND) * 120, I_O0 = I_E0 + ND * 88;
constexpr unsigned INE_ITEMS = 88, INO_ITEMS = 32, INB_ITEMS = 64;
__device__ __forceinline__ Desc puller_desc(int it, int rank, int s) {
    Desc d; d.sig = -1; d.kind = 0; d.half = -1;
    if (it < 960) {
        int c, j;
        if (it < I_E0) { c = it / 120; j = it - c * 120; }
        else if (it < I_O0) { c = (8 - ND) + (it - I_E0) / 88; j = (it - I_E0) - (c - (8 - ND)) * 88; j = j < 32 ? ((j >> 2) << 3) + (j & 3) : j + 32; }
        else { c = (8 - ND) + ((it - I_O0) >> 5); j = (it - I_O0) & 31; j = ((j >> 2) << 3) + 4 + (j & 3); }
        d.flag = it < 64 ? F_XF + s * 32 + (j >> 1) : F_UF + (s * 8 + c) * 16 + (j >> 3); d.sig = ((j < 64 && (j & 4)) ? F_INO : F_INCNT) + s * 8 + c;
        if (j < 64) { d.soff = OFF_A + (unsigned)(8 * s + c) * SLAB + (unsigned)j * 8192u; d.doff = d.soff; }
        else { const unsigned off = (unsigned)(j - 64) * 8192u; d.soff = OFF_W + (unsigned)(rank * 8 + c) * SLAB + off; d.doff = OFF_B + (unsigned)(8 * s + c) * SLAB + off; }
    } else if (it < 1024) {
        const int c = (it - 960) >> 3, j = (it - 960) & 7; const unsigned off = 28u * IMG + (unsigned)j * 8192u;
        d.flag = F_UF + (s * 8 + c) * 16 + 15; d.sig = F_INB + s;
        d.soff = OFF_W + (unsigned)(rank * 8 + c) * SLAB + off; d.doff = OFF_B + (unsigned)(8 * s + c) * SLAB + off;
    } else if (it < 1472) {
        const int idx = (it - 1024) >> 4, j = (it - 1024) & 15, pml = idx / 14, pn = idx - pml * 14;
        d.kind = 1; d.flag = F_OUT + s * 64 + idx; d.half = j >> 3;
        d.soff = OFF_O + (unsigned)((2 * rank + pml) * 16 + pn) * 131072u + (unsigned)j * 8192u;
        d.doff = (unsigned)(pml * 256 + j * 16) * 65536u + (unsigned)(s * 4096 + pn * 256) * 2u;
    } else {
        const int idx = (it - 1472) >> 2, j = (it - 1472) & 3, al = idx >> 2, b = 28 + (idx & 3);
        d.kind = 2; d.flag = F_OUT + s * 64 + 28 + idx;
        d.soff = OFF_O + (unsigned)((2 * rank + (al >> 1)) * 16 + (b >> 1)) * 131072u + (unsigned)((al & 1) * 2 + (b & 1)) * 32768u + (unsigned)j * 8192u;
        d.doff = (unsigned)(al * 128 + j * 32) * 65536u + (unsigned)(s * 4096 + b * 128) * 2u;
    }
    return d;
}
typedef int i32x4 __attribute__((ext_vector_type(4)));
__device__ __forceinline__ i32x4 mk_rsrc(const void* ptr, unsigned bytes) {
    const unsigned long long a = (unsigned long long)ptr; i32x4 r;
    r.x = __builtin_amdgcn_readfirstlane((int)(unsigned)a); r.y = __builtin_amdgcn_readfirstlane((int)(unsigned)(a >> 32) & 0xffff); r.z = (int)bytes; r.w = 0x00020000; return r;
}
#define RING_CLOB "v200","v201","v202","v203","v204","v205","v206","v207","v208","v209","v210","v211","v212","v213","v214","v215"
#define R_LD(r, off_) asm volatile("buffer_load_dwordx4 v[%c2:%c3], %0, %1, 0 offen sc0 sc1" :: "v"(off_), "s"(rsrc), "i"(200 + 4 * (r)), "i"(203 + 4 * (r)) : "memory", RING_CLOB)
#define R_ST(r, off_, rs_, wait, bits) asm volatile(wait "buffer_store_dwordx4 v[%c2:%c3], %0, %1, 0 offen" bits "\n\ts_nop 1" :: "v"(off_), "s"(rs_), "i"(200 + 4 * (r)), "i"(203 + 4 * (r)) : "memory", RING_CLOB)
#define R_NAN(r, wait) asm volatile(wait "v_mov_b32 v%c0, 0x7fc07fc0\n\tv_mov_b32 v%c1, 0x7fc07fc0\n\tv_mov_b32 v%c2, 0x7fc07fc0\n\tv_mov_b32 v%c3, 0x7fc07fc0" \
        :: "i"(200 + 4 * (r)), "i"(201 + 4 * (r)), "i"(202 + 4 * (r)), "i"(203 + 4 * (r)) : "memory", RING_CLOB)
__device__ __forceinline__ void puller(const Params& p, unsigned call, int l, int wl) {
    const int lane = threadIdx.x & 63, rank = p.rank, s = (rank + 1 + l) & 7;
    unsigned* fl = (unsigned*)p.ws[rank]; unsigned* tmo = fl + F_TMO * 32; unsigned* qctr = fl + (F_Q + l) * 32;
    const unsigned qbase = (call - 1u) * (unsigned)NFETCH - 64u;
    const i32x4 rsrc = mk_rsrc(p.ws[s], WS_NEED), rdst = mk_rsrc(p.ws[rank], WS_NEED), rout = mk_rsrc(p.out, OUT_BYTES);
    const unsigned l0 = (unsigned)lane * 16u, l1 = (unsigned)(lane >> 5) * 65536u + (unsigned)(lane & 31) * 16u, l2 = (unsigned)(lane >> 4) * 65536u + (unsigned)(lane & 15) * 16u;
    int cur = wl, nxt = cur + 32;
    bool cur_ready = false, have_prev = false, pbad = false, bad = false; int lastph = -1; (void)lastph;
    unsigned pdoff = 0, plane = 0, pstep = 0; int pkind = 0, psig = -1;
#define P_STORE(r, j, doff_, kind_, lane_, step_, bad_, wait) do { const unsigned o_ = (doff_) + (lane_) + (unsigned)(j) * (step_); \
        if ((kind_) == 0) R_ST(r, o_, rdst, wait, " sc1"); else { if (bad_) R_NAN(r, wait); R_ST(r, o_, rout, wait, ""); } } while (0)
#define P_SIG(sg) do { if ((sg) >= 0 && lane == 0) __hip_atomic_fetch_add(fl + (sg) * 32, 1u, __ATOMIC_RELAXED, __HIP_MEMORY_SCOPE_AGENT); } while (0)
#define P_FLUSH() do { if (have_prev) { WAIT_V(0); _Pragma("unroll") for (int r = 0; r < 4; ++r) P_STORE(r, 4 + r, pdoff, pkind, plane, pstep, pbad, ""); \
            WAIT_V(0); P_SIG(psig); have_prev = false; } } while (0)
    while (cur < NITEM) {
        const Desc dc = puller_desc(cur, rank, s);
        if (!cur_ready) {
            P_FLUSH();
            if (lane == 0) wait_ge(fl + dc.flag * 32, dc.half < 0 ? call : 2u * call - 1u + (unsigned)dc.half, tmo, call);
            bad = __builtin_amdgcn_readfirstlane(__hip_atomic_load(tmo, __ATOMIC_RELAXED, __HIP_MEMORY_SCOPE_AGENT)) == call;
        }
        unsigned ftgt = call;
        if (nxt < NITEM) {
            const Desc dn = puller_desc(nxt, rank, s);
            if (dn.half >= 0) ftgt = 2u * call - 1u + (unsigned)dn.half;
            asm volatile("v_mov_b32 v216, -1\n\tv_mov_b32 v217, -1\n\ts_nop 0\n\tglobal_load_dword v216, %0, off sc0 sc1" :: "v"(fl + dn.flag * 32) : "memory", "v216", "v217");
            if (lane == 0) asm volatile("global_atomic_add v217, %0, %1, off sc0" :: "v"(qctr), "v"(1u) : "memory", "v217");
        }
        const unsigned clane = dc.kind == 0 ? l0 : (dc.kind == 1 ? l1 : l2), cstep = dc.kind == 0 ? 1024u : (dc.kind == 1 ? 131072u : 262144u);
#pragma unroll
        for (int j = 0; j < 8; ++j) {
            const int r = j & 3;
            if (j >= 4) P_STORE(r, j - 4, dc.doff, dc.kind, clane, cstep, bad, "s_waitcnt vmcnt(3)\n\t");
            else if (have_prev) P_STORE(r, 4 + j, pdoff, pkind, plane, pstep, pbad, "s_waitcnt vmcnt(3)\n\t");
            R_LD(r, dc.soff + l0 + (unsigned)j * 1024u);
        }
        if (have_prev) P_SIG(psig);
        unsigned fv, tk;
        asm volatile("v_mov_b32 %0, v216\n\tv_mov_b32 %1, v217" : "=&v"(fv), "=&v"(tk) :: "memory", "v216", "v217");
        if (nxt < NITEM && (__builtin_amdgcn_readfirstlane(fv) == -1 || __builtin_amdgcn_readfirstlane(tk) == -1))
            asm volatile("s_waitcnt vmcnt(0)\n\tv_mov_b32 %0, v216\n\tv_mov_b32 %1, v217" : "=&v"(fv), "=&v"(tk) :: "memory", "v216", "v217");
        have_prev = true; pdoff = dc.doff; pkind = dc.kind; plane = clane; pstep = cstep; psig = dc.sig; pbad = bad;
        cur_ready = nxt < NITEM && (unsigned)__builtin_amdgcn_readfirstlane(fv) >= ftgt;
        const int nn = nxt < NITEM ? (int)(__builtin_amdgcn_readfirstlane(tk) - qbase) : NITEM;
        if ((cur >= I_O0 - 32 && cur < I_O0) || (cur >= 960 - 32 && cur < 960) || (cur >= 1024 - 32 && cur < 1024)) P_FLUSH();
        cur = nxt; nxt = nn;
    }
    P_FLUSH();
}

__device__ __forceinline__ unsigned long long ld_sum(const unsigned* fl, int w, unsigned call) {
    const unsigned long long v = __hip_atomic_load((const unsigned long long*)(fl + F_SUM * 32) + w, __ATOMIC_RELAXED, __HIP_MEMORY_SCOPE_AGENT);
    return (unsigned)(v >> 56) == (call & 0xffu) ? (v & 0x00ffffffffffffffull) : 0ull;
}
__device__ __forceinline__ void monitor(const Params& p, unsigned call) {
    const int lane = threadIdx.x & 63, rank = p.rank;
    unsigned* fl = (unsigned*)p.ws[rank]; unsigned* tmo = fl + F_TMO * 32;
    unsigned long long* sum = (unsigned long long*)(fl + F_SUM * 32);
    const int c = lane / 7, l = lane - c * 7, s = (rank + 1 + l) & 7;
    const unsigned* pe = fl + (F_INCNT + s * 8 + (c & 7)) * 32; const unsigned* po = fl + (F_INO + s * 8 + (c & 7)) * 32; const unsigned* pb = fl + (F_INB + s) * 32;
    const unsigned long long tag = (unsigned long long)(call & 0xffu) << 56, full = 0x00ffffffffffffffull;
    unsigned long long me = 0, mo = 0, mb = 0;
    const unsigned long long t0 = __builtin_amdgcn_s_memrealtime();
    for (;;) {
        const int fe = (int)__builtin_ctzll(~me), fo = (int)__builtin_ctzll(~mo);
        const bool ae = lane < 56 && !((me >> lane) & 1ull) && lane < fe + 14, ao = lane < 56 && !((mo >> lane) & 1ull) && lane < fo + 14 && ((me >> lane) & 1ull);
        const bool ab = lane < 7 && !((mb >> lane) & 1ull) && me == full;
        const bool be = ae && ld_sys(pe) >= INE_ITEMS * call, bo = ao && ld_sys(po) >= INO_ITEMS * call, bb = ab && ld_sys(pb) >= INB_ITEMS * call;
        const unsigned long long ne = me | __ballot(be), no = mo | __ballot(bo), nb = mb | __ballot(bb);
        if (lane == 0) {
            if (ne != me) __hip_atomic_store(sum + 0, ne | tag, __ATOMIC_RELAXED, __HIP_MEMORY_SCOPE_AGENT);
            if (no != mo) __hip_atomic_store(sum + 1, no | tag, __ATOMIC_RELAXED, __HIP_MEMORY_SCOPE_AGENT);
            if (nb != mb) __hip_atomic_store(sum + 2, nb | tag, __ATOMIC_RELAXED, __HIP_MEMORY_SCOPE_AGENT);
        }
        me = ne; mo = no; mb = nb;
        if (me == full && mo == full && mb == 0x7full) break;
        if (__builtin_amdgcn_readfirstlane(__hip_atomic_load(tmo, __ATOMIC_RELAXED, __HIP_MEMORY_SCOPE_AGENT)) == call || __builtin_amdgcn_s_memrealtime() - t0 > DEADLINE) {
            if (lane == 0) __hip_atomic_store(tmo, call, __ATOMIC_RELAXED, __HIP_MEMORY_SCOPE_AGENT); break; }
        __builtin_amdgcn_s_sleep(MON_SLEEP);
    }
}

__device__ __forceinline__ int kt_of(int t, int rank) {
    if (t < 8) return rank * 8 + t;
    const int tt = t - 8, c = tt / 7, l = tt - c * 7; return ((rank + 1 + l) & 7) * 8 + c;
}
__device__ __forceinline__ void store_tile_rows(const f32x4& v0, const f32x4& v1, bool bad, u32x4& w) {
    w.x = gelu2_bf16(v0[0], v0[1]); w.y = gelu2_bf16(v0[2], v0[3]); w.z = gelu2_bf16(v1[0], v1[1]); w.w = gelu2_bf16(v1[2], v1[3]);
    if (bad) w = (u32x4){0x7fc07fc0u, 0x7fc07fc0u, 0x7fc07fc0u, 0x7fc07fc0u};
}

__device__ __forceinline__ void gemm_r1(const Params& p, LAS unsigned char* lds, unsigned call, int pm, int pn) {
    const int tid = threadIdx.x, wid = __builtin_amdgcn_readfirstlane(tid >> 6), lane = tid & 63, wr = wid >> 2, wc = wid & 3, fr = lane & 15, fq = lane >> 4;
    const int rank = p.rank;
    unsigned char* wsl = p.ws[rank]; unsigned* fl = (unsigned*)wsl; unsigned* tmo = fl + F_TMO * 32;
    volatile LAS unsigned* lready = (volatile LAS unsigned*)(lds + STAGE_BYTES);
    const char* gA = (const char*)wsl + OFF_A + (size_t)pm * 2 * IMG + tid * 16;
    const char* gB = (const char*)wsl + OFF_B + (size_t)pn * 2 * IMG + tid * 16;
    const unsigned ldsw = (unsigned)wid * 1024u;
    const int aoff = lds_byte(wr * 64 + fr, fq * 8), boff = lds_byte(wc * 32 + fr, fq * 8);
    const unsigned* myflag = fl + (F_SFLAG + rank * 8 + (lane & 7)) * 32;
    int ready = 0;
#define SA(b, h) (((b) * 2 + (h)) * HTB)
#define SB(b, h) ((4 + (b) * 2 + (h)) * HTB)
#define STAGE(bufoff, gptr) do { _Pragma("unroll") for (int _i = 0; _i < 2; ++_i) \
        __builtin_amdgcn_global_load_lds((const unsigned*)((gptr) + _i * 8192), (LAS unsigned*)(lds + (bufoff) + ldsw + _i * 8192), 16, 0, 0); } while (0)
#define LDA(dst, b, h) do { _Pragma("unroll") for (int m = 0; m < 4; ++m) _Pragma("unroll") for (int k = 0; k < 2; ++k) dst[m][k] = *(const LAS bf16x8*)(lds + SA(b, h) + aoff + m * 2048 + k * 1024); } while (0)
#define LDB(dst, b, h) do { _Pragma("unroll") for (int n = 0; n < 2; ++n) _Pragma("unroll") for (int k = 0; k < 2; ++k) dst[n][k] = *(const LAS bf16x8*)(lds + SB(b, h) + boff + n * 2048 + k * 1024); } while (0)
#define MMA(ai, bj, At, Bt) do { __builtin_amdgcn_s_setprio(1); _Pragma("unroll") for (int m = 0; m < 4; ++m) _Pragma("unroll") for (int n = 0; n < 2; ++n) _Pragma("unroll") for (int k = 0; k < 2; ++k) \
        acc[ai][bj][m][n] = __builtin_amdgcn_mfma_f32_16x16x32_bf16(Bt[n][k], At[m][k], acc[ai][bj][m][n], 0, 0, 0); __builtin_amdgcn_s_setprio(0); } while (0)
    volatile LAS int* lorder = (volatile LAS int*)(lds + STAGE_BYTES + 64);
    unsigned long long sched = 0ull;
    const int mykt = kt_of(lane, rank);
#define NEED(nd) do { if ((nd) > ready) { \
        if (wid == 0) { const unsigned long long t0 = __builtin_amdgcn_s_memrealtime(); int cnt = ready; \
            for (;;) { \
                const unsigned long long sm = ld_sum(fl, pm & 1, call) << 8; \
                bool ok = !((sched >> lane) & 1ull) && (lane < 8 ? ld_sys(myflag) >= call : ((sm >> lane) & 1ull) != 0ull); \
                bool giveup = false; \
                if (cnt + __builtin_popcountll(__ballot(ok)) < (nd)) { \
                    giveup = __builtin_amdgcn_readfirstlane(__hip_atomic_load(tmo, __ATOMIC_RELAXED, __HIP_MEMORY_SCOPE_AGENT)) == call || __builtin_amdgcn_s_memrealtime() - t0 > DEADLINE; \
                    if (giveup) { if (lane == 0) { __hip_atomic_store(tmo, call, __ATOMIC_RELAXED, __HIP_MEMORY_SCOPE_AGENT); lready[1] = 1u; } ok = !((sched >> lane) & 1ull); } } \
                const unsigned long long bal = __ballot(ok); \
                if (ok) lorder[cnt + __builtin_popcountll(bal & ((1ull << lane) - 1ull))] = mykt; \
                sched |= bal; cnt += __builtin_popcountll(bal); \
                if (cnt >= (nd)) break; \
                if ((nd) > 10 && (nd) < ((pm & 1) ? 64 - 7 * ND + ODD_FAST : POLL_FAST_FROM)) __builtin_amdgcn_s_sleep(SLOW_SLEEP); else __builtin_amdgcn_s_sleep(POLL_SLEEP); } \
            ACQ_AGENT; \
            if (lane == 0) { *lready = (unsigned)cnt; \
                if (cnt == 64 && __hip_atomic_load(tmo, __ATOMIC_RELAXED, __HIP_MEMORY_SCOPE_AGENT) == call) lready[1] = 1u; }   \
            asm volatile("s_waitcnt vmcnt(0) lgkmcnt(0)" ::: "memory"); } \
        CFENCE; BAR; CFENCE; \
        ready = __builtin_amdgcn_readfirstlane((int)*lready); } } while (0)
#define KOFF(t) ((size_t)(unsigned)__builtin_amdgcn_readfirstlane(lorder[t]) * SLAB)

    f32x4 acc[2][2][4][2];
#pragma unroll
    for (int a = 0; a < 2; ++a)
#pragma unroll
        for (int b = 0; b < 2; ++b)
#pragma unroll
            for (int m = 0; m < 4; ++m)
#pragma unroll
                for (int n = 0; n < 2; ++n) acc[a][b][m][n] = (f32x4){0.f, 0.f, 0.f, 0.f};
    bf16x8 At[4][2], B0[2][2], B1[2][2];
    if (tid == 0) lready[1] = 0u;
    NEED(4);
    size_t o1, o2, o3;
    {
        const size_t k0 = KOFF(0); o1 = KOFF(1); o2 = KOFF(2); o3 = KOFF(3);
        STAGE(SB(0, 0), gB + k0); STAGE(SA(0, 0), gA + k0); STAGE(SB(0, 1), gB + k0 + IMG); STAGE(SA(0, 1), gA + k0 + IMG);
        if (wr == 1) BAR;
        WAIT_V(4); BAR;
        STAGE(SB(1, 0), gB + o1); STAGE(SA(1, 0), gA + o1); STAGE(SB(1, 1), gB + o1 + IMG);
        WAIT_V(6); BAR;
    }
    STAMP(2);
    for (int t = 0; t < 64; t += 2) {
        LDB(B0, 0, 0); SCHED; LDA(At, 0, 0); STAGE(SA(1, 1), gA + o1 + IMG);
        WAIT_L(8); BAR; WAIT_L(0); MMA(0, 0, At, B0); BAR; SCHED;
        LDB(B1, 0, 1); STAGE(SB(0, 0), gB + o2);
        BAR; WAIT_L(0); MMA(0, 1, At, B1); BAR;
        LDA(At, 0, 1); STAGE(SA(0, 0), gA + o2);
        BAR; WAIT_L(0); MMA(1, 0, At, B0); BAR; SCHED;
        STAGE(SB(0, 1), gB + o2 + IMG);
        WAIT_V(6); BAR; MMA(1, 1, At, B1); BAR;
        LDB(B0, 1, 0); SCHED; LDA(At, 1, 0); STAGE(SA(0, 1), gA + o2 + IMG);
        WAIT_L(8); BAR; WAIT_L(0); MMA(0, 0, At, B0); BAR; SCHED;
        LDB(B1, 1, 1); STAGE(SB(1, 0), gB + o3);
        BAR; WAIT_L(0); MMA(0, 1, At, B1); BAR;
        LDA(At, 1, 1); STAGE(SA(1, 0), gA + o3);
        BAR; WAIT_L(0); MMA(1, 0, At, B0); BAR; SCHED;
        STAGE(SB(1, 1), gB + o3 + IMG);
        WAIT_V(6); BAR; MMA(1, 1, At, B1); BAR;
        o1 = o3;
        if (t + 6 <= 64) { NEED(t + 6); o2 = KOFF(t + 4); o3 = KOFF(t + 5); }
    }
    STAMP(3);
    {
        int tz = threadIdx.x; asm volatile("" : "+v"(tz));
        const int lane2 = tz & 63, fr2 = lane2 & 15, fq2 = lane2 >> 4;
        const bool bad = lready[1] != 0u;
        const int owner = pm >> 1;
        if (wr == 0) BAR;
        if (owner == rank) {
            const auto rout = __builtin_amdgcn_make_buffer_rsrc(p.out, 0, (int)OUT_BYTES, 0x00020000);
            const unsigned base = (unsigned)((pm & 1) * 256 + wr * 64 + fr2) * 65536u + (unsigned)(rank * 4096 + pn * 256 + wc * 32 + fq2 * 8) * 2u;
#pragma unroll
            for (int ai = 0; ai < 2; ++ai)
#pragma unroll
                for (int m = 0; m < 4; ++m)
#pragma unroll
                    for (int bj = 0; bj < 2; ++bj) { u32x4 w; store_tile_rows(acc[ai][bj][m][0], acc[ai][bj][m][1], bad, w);
                        __builtin_amdgcn_raw_buffer_store_b128(w, rout, (int)(base + (unsigned)(ai * 128 + m * 16) * 65536u + (unsigned)bj * 256u), 0, 0); }
            WAIT_V(0); BAR; CFENCE;
        } else {
            const auto rs = __builtin_amdgcn_make_buffer_rsrc(wsl, 0, (int)WS_NEED, 0x00020000);
            const unsigned base = OFF_O + (unsigned)(pm * 16 + pn) * 131072u + (unsigned)(wr * 64 + fr2) * 512u + (unsigned)(wc * 32 + fq2 * 8) * 2u;
            unsigned* oflag = (unsigned*)p.ws[owner] + (F_OUT + rank * 64 + (pm & 1) * 14 + pn) * 32;
#pragma unroll
            for (int ai = 0; ai < 2; ++ai) {
#pragma unroll
                for (int m = 0; m < 4; ++m)
#pragma unroll
                    for (int bj = 0; bj < 2; ++bj) { u32x4 w; store_tile_rows(acc[ai][bj][m][0], acc[ai][bj][m][1], bad, w);
                        __builtin_amdgcn_raw_buffer_store_b128(w, rs, (int)(base + (unsigned)(ai * 128 + m * 16) * 512u + (unsigned)bj * 256u), 0, AUX_SYS); }
                WAIT_V(0); BAR; CFENCE;
                if (tz == 0) {
                    st_sys(oflag, 2u * call - 1u + (unsigned)ai);
                }
            }
        }
    }
#undef NEED
}

__device__ __forceinline__ void gemm_r2(const Params& p, LAS unsigned char* lds, unsigned call, int a, int b) {
    const int tid = threadIdx.x, wid = __builtin_amdgcn_readfirstlane(tid >> 6), lane = tid & 63, wr = wid >> 2, wc = wid & 3, fr = lane & 15, fq = lane >> 4;
    const int rank = p.rank;
    unsigned char* wsl = p.ws[rank]; unsigned* fl = (unsigned*)wsl; unsigned* tmo = fl + F_TMO * 32;
    if (tid < 64) {
        if (lane < 8) wait_ge(fl + (F_SFLAG + rank * 8 + lane) * 32, call, tmo, call);
        const unsigned long long t0 = __builtin_amdgcn_s_memrealtime();
        while (ld_sum(fl, (a >> 1) & 1, call) != 0x00ffffffffffffffull || ld_sum(fl, 2, call) != 0x7full) {
            if (__builtin_amdgcn_readfirstlane(__hip_atomic_load(tmo, __ATOMIC_RELAXED, __HIP_MEMORY_SCOPE_AGENT)) == call || __builtin_amdgcn_s_memrealtime() - t0 > DEADLINE) {
                if (lane == 0) __hip_atomic_store(tmo, call, __ATOMIC_RELAXED, __HIP_MEMORY_SCOPE_AGENT); break; }
            __builtin_amdgcn_s_sleep(16);
        }
        ACQ_AGENT; WAIT_V(0);
    }
    CFENCE; BAR; CFENCE;
    const char* gA = (const char*)wsl + OFF_A + (size_t)a * IMG + tid * 16;
    const char* gB = (const char*)wsl + OFF_B + (size_t)b * IMG + tid * 16;
    const unsigned ldsw = (unsigned)wid * 1024u;
    const int aoff = lds_byte(wr * 64 + fr, fq * 8), boff = lds_byte(wc * 32 + fr, fq * 8);
    f32x4 acc[4][2];
#pragma unroll
    for (int m = 0; m < 4; ++m)
#pragma unroll
        for (int n = 0; n < 2; ++n) acc[m][n] = (f32x4){0.f, 0.f, 0.f, 0.f};
#define STAGE2(slot, kt) do { STAGE((slot) * 32768, gA + (size_t)(kt) * SLAB); STAGE((slot) * 32768 + 16384, gB + (size_t)(kt) * SLAB); } while (0)
#define FRAG2(Af, Bf, slot) do { \
        _Pragma("unroll") for (int n = 0; n < 2; ++n) _Pragma("unroll") for (int k = 0; k < 2; ++k) Bf[n][k] = *(const LAS bf16x8*)(lds + (slot) * 32768 + 16384 + boff + n * 2048 + k * 1024); \
        _Pragma("unroll") for (int m = 0; m < 4; ++m) _Pragma("unroll") for (int k = 0; k < 2; ++k) Af[m][k] = *(const LAS bf16x8*)(lds + (slot) * 32768 + aoff + m * 2048 + k * 1024); } while (0)
#define MMA2(Af, Bf) do { __builtin_amdgcn_s_setprio(1); _Pragma("unroll") for (int k = 0; k < 2; ++k) _Pragma("unroll") for (int m = 0; m < 4; ++m) _Pragma("unroll") for (int n = 0; n < 2; ++n) \
        acc[m][n] = __builtin_amdgcn_mfma_f32_16x16x32_bf16(Bf[n][k], Af[m][k], acc[m][n], 0, 0, 0); __builtin_amdgcn_s_setprio(0); } while (0)
    bf16x8 A0[4][2], Bf0[2][2], A1[4][2], Bf1[2][2];
    STAMP(6);
    STAGE2(0, 0); STAGE2(1, 1); STAGE2(2, 2); STAGE2(3, 3);
    WAIT_V(12); BAR; CFENCE;
    FRAG2(A0, Bf0, 0);
    for (int t = 0; t < 64; t += 4) {
        __builtin_amdgcn_s_waitcnt(0xC07F); WAIT_V(8); BAR; SCHED; STAGE2(0, (t + 4) & 63); FRAG2(A1, Bf1, 1); SCHED; MMA2(A0, Bf0); SCHED;
        __builtin_amdgcn_s_waitcnt(0xC07F); WAIT_V(8); BAR; SCHED; STAGE2(1, (t + 5) & 63); FRAG2(A0, Bf0, 2); SCHED; MMA2(A1, Bf1); SCHED;
        __builtin_amdgcn_s_waitcnt(0xC07F); WAIT_V(8); BAR; SCHED; STAGE2(2, (t + 6) & 63); FRAG2(A1, Bf1, 3); SCHED; MMA2(A0, Bf0); SCHED;
        __builtin_amdgcn_s_waitcnt(0xC07F); WAIT_V(8); BAR; SCHED; STAGE2(3, (t + 7) & 63); FRAG2(A0, Bf0, 0); SCHED; MMA2(A1, Bf1); SCHED;
    }
    WAIT_L(0);
    STAMP(7);
    const bool bad = __hip_atomic_load(tmo, __ATOMIC_RELAXED, __HIP_MEMORY_SCOPE_AGENT) == call;
    const int owner = a >> 2;
    if (owner == rank) {
        const auto rout = __builtin_amdgcn_make_buffer_rsrc(p.out, 0, (int)OUT_BYTES, 0x00020000);
        const unsigned base = (unsigned)((a & 3) * 128 + wr * 64 + fr) * 65536u + (unsigned)(rank * 4096 + b * 128 + wc * 32 + fq * 8) * 2u;
#pragma unroll
        for (int m = 0; m < 4; ++m) { u32x4 w; store_tile_rows(acc[m][0], acc[m][1], bad, w);
            __builtin_amdgcn_raw_buffer_store_b128(w, rout, (int)(base + (unsigned)(m * 16) * 65536u), 0, 0); }
    } else {
        const auto rs = __builtin_amdgcn_make_buffer_rsrc(wsl, 0, (int)WS_NEED, 0x00020000);
        const unsigned base = OFF_O + (unsigned)((a >> 1) * 16 + (b >> 1)) * 131072u + (unsigned)((a & 1) * 2 + (b & 1)) * 32768u + (unsigned)(wr * 64 + fr) * 256u + (unsigned)(wc * 32 + fq * 8) * 2u;
#pragma unroll
        for (int m = 0; m < 4; ++m) { u32x4 w; store_tile_rows(acc[m][0], acc[m][1], bad, w);
            __builtin_amdgcn_raw_buffer_store_b128(w, rs, (int)(base + (unsigned)(m * 16) * 256u), 0, AUX_SYS); }
    }
    WAIT_V(0); BAR; CFENCE;
    if (owner != rank && tid == 0) {
        st_sys((unsigned*)p.ws[owner] + (F_OUT + rank * 64 + 28 + (a & 3) * 4 + (b - 28)) * 32, call);
    }
}

__global__ __launch_bounds__(512, 2) void fused_gemm_a2a(Params p) {
    extern __shared__ __attribute__((aligned(16))) unsigned char shm[];
    LAS unsigned char* lds = (LAS unsigned char*)shm;
    const int tid = threadIdx.x, bid = blockIdx.x;
    unsigned* fl = (unsigned*)p.ws[p.rank];
    volatile LAS unsigned* lcall = (volatile LAS unsigned*)(lds + STAGE_BYTES + 16);
    if (tid == 0) *lcall = __hip_atomic_fetch_add(fl + F_TICKET * 32, 1u, __ATOMIC_RELAXED, __HIP_MEMORY_SCOPE_AGENT) / (unsigned)NWG + 1u;
    STAMP(0);
    const bool is_puller = bid >= NSTAGER;
    if (!is_puller) stage_units(p, lds, lcall, bid);
    __syncthreads();
    const unsigned call = __builtin_amdgcn_readfirstlane(*lcall);
    if (is_puller) {
        const int pi = bid - NSTAGER, w = __builtin_amdgcn_readfirstlane(tid >> 6);
        puller(p, call, pi % 7, (pi / 7) * 8 + w);
        STAMP(19); return;
    }
    STAMP(1);
    __syncthreads();
    const int xcd = bid & 7, idx = bid >> 3, rg = xcd >> 1, cg = xcd & 1;
    const int n1 = 28, nw = xcd < 4 ? 29 : 28;
    if (idx < n1) {
        gemm_r1(p, lds, call, rg * 4 + (idx & 3), cg * 7 + (idx >> 2));
        STAMP(4);
    } else if (bid == 224 && tid < 64) monitor(p, call);
    volatile LAS unsigned* lq = (volatile LAS unsigned*)(lds + STAGE_BYTES + 32);
    for (;;) {
        __syncthreads();
        if (tid == 0) *lq = __hip_atomic_fetch_add(fl + (F_Q2 + xcd) * 32, 1u, __ATOMIC_RELAXED, __HIP_MEMORY_SCOPE_AGENT) - (call - 1u) * (unsigned)(16 + nw);
        __syncthreads();
        const unsigned r2 = __builtin_amdgcn_readfirstlane(*lq);
        if (r2 >= 16u) break;
        gemm_r2(p, lds, call, rg * 8 + (int)(r2 & 7u), 28 + cg * 2 + (int)(r2 >> 3));
        STAMP(5);
    }
}
}

extern "C" void kernel_launch(int rank, int nranks, ncclComm_t comm,
                              hipStream_t stream,
                              void* const* d_in, const int* in_sizes, int n_in,
                              void* d_out, int out_size,
                              const int* in_dtypes, int out_dtype,
                              void* const* d_out_all,
                              void* const* d_ws_all, size_t ws_bytes) {
    if (nranks != NR || n_in != 2 || ws_bytes < (size_t)WS_NEED || (unsigned)rank >= (unsigned)NR) { fprintf(stderr, "kernel_launch: unexpected configuration\n"); return; }
    Params p;
    p.x = (const unsigned short*)d_in[0]; p.w = (const unsigned short*)d_in[1]; p.out = (unsigned short*)d_out; p.rank = rank;
    for (int i = 0; i < NR; ++i) p.ws[i] = (unsigned char*)d_ws_all[i];
    HIP_CHECK(hipFuncSetAttribute((const void*)fused_gemm_a2a, hipFuncAttributeMaxDynamicSharedMemorySize, LDS_BYTES));
    hipLaunchKernelGGL(fused_gemm_a2a, dim3(NWG), dim3(512), LDS_BYTES, stream, p);
    HIP_CHECK(hipGetLastError());
}
```

```cpp
#include <hip/hip_runtime.h>
#include <rccl/rccl.h>
#include <cstdio>
#include <cstdint>

#define HIP_CHECK(expr) do { hipError_t _e = (expr); if (_e != hipSuccess) { \
    fprintf(stderr, "HIP error %s at %s:%d\n", hipGetErrorString(_e), __FILE__, __LINE__); \
    return; } } while (0)

namespace {
typedef unsigned u32x4 __attribute__((ext_vector_type(4)));
typedef short bf16x8 __attribute__((ext_vector_type(8)));
typedef float f32x4 __attribute__((ext_vector_type(4)));
#define LAS __attribute__((address_space(3)))

constexpr int NR = 8;
constexpr int MM = 4096, KS = 512, NN = 32768, NS = 4096, MS = 512;
constexpr unsigned IMG = 16384;
constexpr unsigned SLAB = 32 * IMG;
constexpr unsigned OFF_A = 1u << 20;
constexpr unsigned OFF_B = OFF_A + 64 * SLAB;
constexpr unsigned OFF_W = OFF_B + 64 * SLAB;
constexpr unsigned OFF_O = OFF_W + 64 * SLAB;
constexpr unsigned WS_NEED = OFF_O + 256u * 131072u;
constexpr unsigned OUT_BYTES = (unsigned)MS * NN * 2u;
constexpr int F_TICKET = 0, F_TMO = 1, F_SCNT = 2, F_SFLAG = 16, F_INCNT = 80, F_INB = 144, F_Q = 152, F_OUT = 160  , F_INO = 672, F_Q2 = 736, F_SUM = 744, F_XF = 768  , F_UF = 1024  ;
constexpr int NPULL = 28, NWG = 256;
constexpr int NSTAGER = NWG - NPULL;
constexpr int UNITS_PER_CHUNK = 72, NUNITS = 8 * UNITS_PER_CHUNK;
constexpr unsigned long long DEADLINE = 50000000ull;
constexpr int HTB = 16384, STAGE_BYTES = 8 * HTB, LDS_BYTES = STAGE_BYTES + 512;
constexpr int AUX_SYS = 17, AUX_AGENT = 16;
#define MON_SLEEP 8
#define ODD_FAST 6
#define SLOW_SLEEP 127
#define POLL_FAST_FROM 56
#define POLL_SLEEP 16

struct Params { const unsigned short* x; const unsigned short* w; unsigned char* ws[NR]; unsigned short* out; int rank; };

__device__ __forceinline__ int lds_byte(int r, int c) { const int st = (r >> 4) * 2 + (c >> 5), rr = r & 15, cc = c & 31, ob = rr * 64 + cc * 2; return st * 1024 + (ob ^ (((ob >> 9) & 1) << 5)); }
__device__ __forceinline__ void stage_rc(int b, int& R, int& C) { const int st = b / 1024, sb = b % 1024, swz = sb ^ (((sb >> 9) & 1) << 5); R = (st >> 1) * 16 + swz / 64; C = (st & 1) * 32 + (swz % 64) / 2; }

__device__ __forceinline__ unsigned ld_sys(const unsigned* p) { return __hip_atomic_load(p, __ATOMIC_RELAXED, __HIP_MEMORY_SCOPE_SYSTEM); }
__device__ __forceinline__ void st_sys(unsigned* p, unsigned v) { __hip_atomic_store(p, v, __ATOMIC_RELAXED, __HIP_MEMORY_SCOPE_SYSTEM); }
__device__ __forceinline__ void wait_ge(const unsigned* flag, unsigned val, unsigned* tmo, unsigned call) {
    const unsigned long long t0 = __builtin_amdgcn_s_memrealtime();
    while (ld_sys(flag) < val) {
        if (__hip_atomic_load(tmo, __ATOMIC_RELAXED, __HIP_MEMORY_SCOPE_AGENT) == call || __builtin_amdgcn_s_memrealtime() - t0 > DEADLINE) {
            __hip_atomic_store(tmo, call, __ATOMIC_RELAXED, __HIP_MEMORY_SCOPE_AGENT); break; }
        __builtin_amdgcn_s_sleep(2);
    }
}
__device__ __forceinline__ unsigned cvt_pk_bf16(float lo, float hi) { unsigned r; asm volatile("v_cvt_pk_bf16_f32 %0, %1, %2" : "=v"(r) : "v"(lo), "v"(hi)); return r; }
typedef float f32x2 __attribute__((ext_vector_type(2)));
__device__ __forceinline__ unsigned gelu2_bf16(float y0, float y1) {
    const f32x2 y = {y0, y1};
    const f32x2 t = (y * y) * -0.1029432395800235f + -2.302208198144325f;
    const f32x2 a = y * t;
    f32x2 d; d.x = __builtin_amdgcn_exp2f(a.x); d.y = __builtin_amdgcn_exp2f(a.y);
    d = d + 1.0f;
    f32x2 r; r.x = __builtin_amdgcn_rcpf(d.x); r.y = __builtin_amdgcn_rcpf(d.y);
    r = r * y;
    return cvt_pk_bf16(r.x, r.y);
}
#define STAMP(i) do { } while (0)
#define WAIT_V(n) asm volatile("s_waitcnt vmcnt(" #n ")" ::: "memory")
#define WAIT_L(n) asm volatile("s_waitcnt lgkmcnt(" #n ")" ::: "memory")
#define BAR __builtin_amdgcn_s_barrier()
#define SCHED __builtin_amdgcn_sched_barrier(0)
#define CFENCE asm volatile("" ::: "memory")
#define ACQ_AGENT CFENCE

__device__ __forceinline__ void stage_units(const Params& p, LAS unsigned char* lds, volatile LAS unsigned* lcall, int first) {
    const int tid = threadIdx.x, rank = p.rank;
    unsigned char* wsl = p.ws[rank]; unsigned* fl = (unsigned*)wsl;
    const auto rs = __builtin_amdgcn_make_buffer_rsrc(wsl, 0, (int)WS_NEED, 0x00020000);
    if (first < 32) {
        const unsigned dbase = OFF_A + (unsigned)(rank * 8) * SLAB + (unsigned)first * IMG;
        u32x4 t[2];
#pragma unroll
        for (int i = 0; i < 2; ++i) { int R, C; stage_rc((tid + 512 * i) * 16, R, C); t[i] = *(const u32x4*)(p.x + (size_t)(first * 128 + R) * KS + C); }
#pragma unroll
        for (int i = 0; i < 2; ++i) __builtin_amdgcn_raw_buffer_store_b128(t[i], rs, (int)(dbase + (unsigned)(tid + 512 * i) * 16u), 0, AUX_SYS);
        WAIT_V(0);
        __syncthreads();
        const unsigned call = *lcall;
        if (tid < NR) { if (tid != rank) st_sys((unsigned*)p.ws[tid] + (F_XF + rank * 32 + first) * 32, call); }
        else if (tid == 64) {
            const unsigned old = __hip_atomic_fetch_add(fl + F_SCNT * 32, 1u, __ATOMIC_RELAXED, __HIP_MEMORY_SCOPE_AGENT);
            if (old + 1u == call * 96u) st_sys(fl + (F_SFLAG + rank * 8) * 32, call);
        }
    }
    for (int u = 8 + (first + NSTAGER - 32) % NSTAGER; u < NUNITS; u += NSTAGER) {
        const int c = u / UNITS_PER_CHUNK, v0 = u % UNITS_PER_CHUNK, v = v0 < 8 ? 64 + v0 : v0 - 8;
        if (v >= 64) {
            const int vv = v - 64;
            const unsigned dbase = OFF_A + (unsigned)(rank * 8 + c) * SLAB + (unsigned)vv * 4u * IMG;
            u32x4 t[8];
#pragma unroll
            for (int i = 0; i < 8; ++i) { const int pc = tid + 512 * i, img = pc >> 10; int R, C; stage_rc((pc & 1023) * 16, R, C);
                t[i] = *(const u32x4*)(p.x + (size_t)(vv * 512 + img * 128 + R) * KS + c * 64 + C); }
#pragma unroll
            for (int i = 0; i < 8; ++i) __builtin_amdgcn_raw_buffer_store_b128(t[i], rs, (int)(dbase + (unsigned)(tid + 512 * i) * 16u), 0, AUX_SYS);
        } else {
            const int d = v >> 3, kb = tid >> 6, nb = tid & 63;
            const unsigned dbase = (d == rank ? OFF_B + (unsigned)(rank * 8 + c) * SLAB : OFF_W + (unsigned)(d * 8 + c) * SLAB) + (unsigned)(v & 7) * 4u * IMG;
            u32x4 r[8];
#pragma unroll
            for (int i = 0; i < 8; ++i) r[i] = *(const u32x4*)(p.w + (size_t)(c * 64 + kb * 8 + i) * NN + v * 512 + nb * 8);
#pragma unroll
            for (int j = 0; j < 8; ++j) {
                u32x4 o;
#pragma unroll
                for (int e = 0; e < 4; ++e) o[e] = __builtin_amdgcn_perm(r[2 * e + 1][j >> 1], r[2 * e][j >> 1], (j & 1) ? 0x07060302u : 0x05040100u);
                const int nl = nb * 8 + j, img = nl >> 7, rowi = nl & 127, cc = rowi & 31;
                const int rho = (rowi & ~31) + 16 * ((cc >> 2) & 1) + 4 * (cc >> 3) + (cc & 3);
                *(LAS u32x4*)(lds + img * IMG + lds_byte(rho, kb * 8)) = o;
            }
            __syncthreads();
#pragma unroll
            for (int i = 0; i < 8; ++i) { const unsigned b = (unsigned)(tid + 512 * i) * 16u; const u32x4 t = *(const LAS u32x4*)(lds + b);
                __builtin_amdgcn_raw_buffer_store_b128(t, rs, (int)(dbase + b), 0, AUX_SYS); }
        }
        WAIT_V(0);
        __syncthreads();
        const unsigned call = *lcall;
        if (tid < NR) {
            if (tid != rank && (v >= 64 || tid == (v >> 3)))
                st_sys((unsigned*)p.ws[tid] + (F_UF + (rank * 8 + c) * 16 + (v >= 64 ? v - 64 : 8 + (v & 7))) * 32, call);
        } else if (tid == 64) {
            const unsigned old = __hip_atomic_fetch_add(fl + (F_SCNT + c) * 32, 1u, __ATOMIC_RELAXED, __HIP_MEMORY_SCOPE_AGENT);
            if (old + 1u == call * (c == 0 ? 96u : (unsigned)UNITS_PER_CHUNK)) st_sys(fl + (F_SFLAG + rank * 8 + c) * 32, call);
        }
    }
}

struct Desc { unsigned soff, doff; int kind; int flag; int sig; int half; };
constexpr int NITEM = 1536, NFETCH = NITEM - 64 + 32;
#define ND 4
constexpr int I_E0 = (8 - ND) * 120, I_O0 = I_E0 + ND * 88;
constexpr unsigned INE_ITEMS = 88, INO_ITEMS = 32, INB_ITEMS = 64;
__device__ __forceinline__ Desc puller_desc(int it, int rank, int s) {
    Desc d; d.sig = -1; d.kind = 0; d.half = -1;
    if (it < 960) {
        int c, j;
        if (it < I_E0) { c = it / 120; j = it - c * 120; }
        else if (it < I_O0) { c = (8 - ND) + (it - I_E0) / 88; j = (it - I_E0) - (c - (8 - ND)) * 88; j = j < 32 ? ((j >> 2) << 3) + (j & 3) : j + 32; }
        else { c = (8 - ND) + ((it - I_O0) >> 5); j = (it - I_O0) & 31; j = ((j >> 2) << 3) + 4 + (j & 3); }
        d.flag = it < 64 ? F_XF + s * 32 + (j >> 1) : F_UF + (s * 8 + c) * 16 + (j >> 3); d.sig = ((j < 64 && (j & 4)) ? F_INO : F_INCNT) + s * 8 + c;
        if (j < 64) { d.soff = OFF_A + (unsigned)(8 * s + c) * SLAB + (unsigned)j * 8192u; d.doff = d.soff; }
        else { const unsigned off = (unsigned)(j - 64) * 8192u; d.soff = OFF_W + (unsigned)(rank * 8 + c) * SLAB + off; d.doff = OFF_B + (unsigned)(8 * s + c) * SLAB + off; }
    } else if (it < 1024) {
        const int c = (it - 960) >> 3, j = (it - 960) & 7; const unsigned off = 28u * IMG + (unsigned)j * 8192u;
        d.flag = F_UF + (s * 8 + c) * 16 + 15; d.sig = F_INB + s;
        d.soff = OFF_W + (unsigned)(rank * 8 + c) * SLAB + off; d.doff = OFF_B + (unsigned)(8 * s + c) * SLAB + off;
    } else if (it < 1472) {
        const int idx = (it - 1024) >> 4, j = (it - 1024) & 15, pml = idx / 14, pn = idx - pml * 14;
        d.kind = 1; d.flag = F_OUT + s * 64 + idx; d.half = j >> 3;
        d.soff = OFF_O + (unsigned)((2 * rank + pml) * 16 + pn) * 131072u + (unsigned)j * 8192u;
        d.doff = (unsigned)(pml * 256 + j * 16) * 65536u + (unsigned)(s * 4096 + pn * 256) * 2u;
    } else {
        const int idx = (it - 1472) >> 2, j = (it - 1472) & 3, al = idx >> 2, b = 28 + (idx & 3);
        d.kind = 2; d.flag = F_OUT + s * 64 + 28 + idx;
        d.soff = OFF_O + (unsigned)((2 * rank + (al >> 1)) * 16 + (b >> 1)) * 131072u + (unsigned)((al & 1) * 2 + (b & 1)) * 32768u + (unsigned)j * 8192u;
        d.doff = (unsigned)(al * 128 + j * 32) * 65536u + (unsigned)(s * 4096 + b * 128) * 2u;
    }
    return d;
}
typedef int i32x4 __attribute__((ext_vector_type(4)));
__device__ __forceinline__ i32x4 mk_rsrc(const void* ptr, unsigned bytes) {
    const unsigned long long a = (unsigned long long)ptr; i32x4 r;
    r.x = __builtin_amdgcn_readfirstlane((int)(unsigned)a); r.y = __builtin_amdgcn_readfirstlane((int)(unsigned)(a >> 32) & 0xffff); r.z = (int)bytes; r.w = 0x00020000; return r;
}
#define RING_CLOB "v200","v201","v202","v203","v204","v205","v206","v207","v208","v209","v210","v211","v212","v213","v214","v215"
#define R_LD(r, off_) asm volatile("buffer_load_dwordx4 v[%c2:%c3], %0, %1, 0 offen sc0 sc1" :: "v"(off_), "s"(rsrc), "i"(200 + 4 * (r)), "i"(203 + 4 * (r)) : "memory", RING_CLOB)
#define R_ST(r, off_, rs_, wait, bits) asm volatile(wait "buffer_store_dwordx4 v[%c2:%c3], %0, %1, 0 offen" bits "\n\ts_nop 1" :: "v"(off_), "s"(rs_), "i"(200 + 4 * (r)), "i"(203 + 4 * (r)) : "memory", RING_CLOB)
#define R_NAN(r, wait) asm volatile(wait "v_mov_b32 v%c0, 0x7fc07fc0\n\tv_mov_b32 v%c1, 0x7fc07fc0\n\tv_mov_b32 v%c2, 0x7fc07fc0\n\tv_mov_b32 v%c3, 0x7fc07fc0" \
        :: "i"(200 + 4 * (r)), "i"(201 + 4 * (r)), "i"(202 + 4 * (r)), "i"(203 + 4 * (r)) : "memory", RING_CLOB)
__device__ __forceinline__ void puller(const Params& p, unsigned call, int l, int wl) {
    const int lane = threadIdx.x & 63, rank = p.rank, s = (rank + 1 + l) & 7;
    unsigned* fl = (unsigned*)p.ws[rank]; unsigned* tmo = fl + F_TMO * 32; unsigned* qctr = fl + (F_Q + l) * 32;
    const unsigned qbase = (call - 1u) * (unsigned)NFETCH - 64u;
    const i32x4 rsrc = mk_rsrc(p.ws[s], WS_NEED), rdst = mk_rsrc(p.ws[rank], WS_NEED), rout = mk_rsrc(p.out, OUT_BYTES);
    const unsigned l0 = (unsigned)lane * 16u, l1 = (unsigned)(lane >> 5) * 65536u + (unsigned)(lane & 31) * 16u, l2 = (unsigned)(lane >> 4) * 65536u + (unsigned)(lane & 15) * 16u;
    int cur = wl, nxt = cur + 32;
    bool cur_ready = false, have_prev = false, pbad = false, bad = false; int lastph = -1; (void)lastph;
    unsigned pdoff = 0, plane = 0, pstep = 0; int pkind = 0, psig = -1;
#define P_STORE(r, j, doff_, kind_, lane_, step_, bad_, wait) do { const unsigned o_ = (doff_) + (lane_) + (unsigned)(j) * (step_); \
        if ((kind_) == 0) R_ST(r, o_, rdst, wait, " sc1"); else { if (bad_) R_NAN(r, wait); R_ST(r, o_, rout, wait, ""); } } while (0)
#define P_SIG(sg) do { if ((sg) >= 0 && lane == 0) __hip_atomic_fetch_add(fl + (sg) * 32, 1u, __ATOMIC_RELAXED, __HIP_MEMORY_SCOPE_AGENT); } while (0)
#define P_FLUSH() do { if (have_prev) { WAIT_V(0); _Pragma("unroll") for (int r = 0; r < 4; ++r) P_STORE(r, 4 + r, pdoff, pkind, plane, pstep, pbad, ""); \
            WAIT_V(0); P_SIG(psig); have_prev = false; } } while (0)
    while (cur < NITEM) {
        const Desc dc = puller_desc(cur, rank, s);
        if (!cur_ready) {
            P_FLUSH();
            if (lane == 0) wait_ge(fl + dc.flag * 32, dc.half < 0 ? call : 2u * call - 1u + (unsigned)dc.half, tmo, call);
            bad = __builtin_amdgcn_readfirstlane(__hip_atomic_load(tmo, __ATOMIC_RELAXED, __HIP_MEMORY_SCOPE_AGENT)) == call;
        }
        unsigned ftgt = call;
        if (nxt < NITEM) {
            const Desc dn = puller_desc(nxt, rank, s);
            if (dn.half >= 0) ftgt = 2u * call - 1u + (unsigned)dn.half;
            asm volatile("v_mov_b32 v216, -1\n\tv_mov_b32 v217, -1\n\ts_nop 0\n\tglobal_load_dword v216, %0, off sc0 sc1" :: "v"(fl + dn.flag * 32) : "memory", "v216", "v217");
            if (lane == 0) asm volatile("global_atomic_add v217, %0, %1, off sc0" :: "v"(qctr), "v"(1u) : "memory", "v217");
        }
        const unsigned clane = dc.kind == 0 ? l0 : (dc.kind == 1 ? l1 : l2), cstep = dc.kind == 0 ? 1024u : (dc.kind == 1 ? 131072u : 262144u);
#pragma unroll
        for (int j = 0; j < 8; ++j) {
            const int r = j & 3;
            if (j >= 4) P_STORE(r, j - 4, dc.doff, dc.kind, clane, cstep, bad, "s_waitcnt vmcnt(3)\n\t");
            else if (have_prev) P_STORE(r, 4 + j, pdoff, pkind, plane, pstep, pbad, "s_waitcnt vmcnt(3)\n\t");
            R_LD(r, dc.soff + l0 + (unsigned)j * 1024u);
        }
        if (have_prev) P_SIG(psig);
        unsigned fv, tk;
        asm volatile("v_mov_b32 %0, v216\n\tv_mov_b32 %1, v217" : "=&v"(fv), "=&v"(tk) :: "memory", "v216", "v217");
        if (nxt < NITEM && (__builtin_amdgcn_readfirstlane(fv) == -1 || __builtin_amdgcn_readfirstlane(tk) == -1))
            asm volatile("s_waitcnt vmcnt(0)\n\tv_mov_b32 %0, v216\n\tv_mov_b32 %1, v217" : "=&v"(fv), "=&v"(tk) :: "memory", "v216", "v217");
        have_prev = true; pdoff = dc.doff; pkind = dc.kind; plane = clane; pstep = cstep; psig = dc.sig; pbad = bad;
        cur_ready = nxt < NITEM && (unsigned)__builtin_amdgcn_readfirstlane(fv) >= ftgt;
        const int nn = nxt < NITEM ? (int)(__builtin_amdgcn_readfirstlane(tk) - qbase) : NITEM;
        if ((cur >= I_O0 - 32 && cur < I_O0) || (cur >= 960 - 32 && cur < 960) || (cur >= 1024 - 32 && cur < 1024)) P_FLUSH();
        cur = nxt; nxt = nn;
    }
    P_FLUSH();
}

__device__ __forceinline__ unsigned long long ld_sum(const unsigned* fl, int w, unsigned call) {
    const unsigned long long v = __hip_atomic_load((const unsigned long long*)(fl + F_SUM * 32) + w, __ATOMIC_RELAXED, __HIP_MEMORY_SCOPE_AGENT);
    return (unsigned)(v >> 56) == (call & 0xffu) ? (v & 0x00ffffffffffffffull) : 0ull;
}
__device__ __forceinline__ void monitor(const Params& p, unsigned call) {
    const int lane = threadIdx.x & 63, rank = p.rank;
    unsigned* fl = (unsigned*)p.ws[rank]; unsigned* tmo = fl + F_TMO * 32;
    unsigned long long* sum = (unsigned long long*)(fl + F_SUM * 32);
    const int c = lane / 7, l = lane - c * 7, s = (rank + 1 + l) & 7;
    const unsigned* pe = fl + (F_INCNT + s * 8 + (c & 7)) * 32; const unsigned* po = fl + (F_INO + s * 8 + (c & 7)) * 32; const unsigned* pb = fl + (F_INB + s) * 32;
    const unsigned long long tag = (unsigned long long)(call & 0xffu) << 56, full = 0x00ffffffffffffffull;
    unsigned long long me = 0, mo = 0, mb = 0;
    const unsigned long long t0 = __builtin_amdgcn_s_memrealtime();
    for (;;) {
        const int fe = (int)__builtin_ctzll(~me), fo = (int)__builtin_ctzll(~mo);
        const bool ae = lane < 56 && !((me >> lane) & 1ull) && lane < fe + 14, ao = lane < 56 && !((mo >> lane) & 1ull) && lane < fo + 14 && ((me >> lane) & 1ull);
        const bool ab = lane < 7 && !((mb >> lane) & 1ull) && me == full;
        const bool be = ae && ld_sys(pe) >= INE_ITEMS * call, bo = ao && ld_sys(po) >= INO_ITEMS * call, bb = ab && ld_sys(pb) >= INB_ITEMS * call;
        const unsigned long long ne = me | __ballot(be), no = mo | __ballot(bo), nb = mb | __ballot(bb);
        if (lane == 0) {
            if (ne != me) __hip_atomic_store(sum + 0, ne | tag, __ATOMIC_RELAXED, __HIP_MEMORY_SCOPE_AGENT);
            if (no != mo) __hip_atomic_store(sum + 1, no | tag, __ATOMIC_RELAXED, __HIP_MEMORY_SCOPE_AGENT);
            if (nb != mb) __hip_atomic_store(sum + 2, nb | tag, __ATOMIC_RELAXED, __HIP_MEMORY_SCOPE_AGENT);
        }
        me = ne; mo = no; mb = nb;
        if (me == full && mo == full && mb == 0x7full) break;
        if (__builtin_amdgcn_readfirstlane(__hip_atomic_load(tmo, __ATOMIC_RELAXED, __HIP_MEMORY_SCOPE_AGENT)) == call || __builtin_amdgcn_s_memrealtime() - t0 > DEADLINE) {
            if (lane == 0) __hip_atomic_store(tmo, call, __ATOMIC_RELAXED, __HIP_MEMORY_SCOPE_AGENT); break; }
        __builtin_amdgcn_s_sleep(MON_SLEEP);
    }
}

__device__ __forceinline__ int kt_of(int t, int rank) {
    if (t < 8) return rank * 8 + t;
    const int tt = t - 8, c = tt / 7, l = tt - c * 7; return ((rank + 1 + l) & 7) * 8 + c;
}
__device__ __forceinline__ void store_tile_rows(const f32x4& v0, const f32x4& v1, bool bad, u32x4& w) {
    w.x = gelu2_bf16(v0[0], v0[1]); w.y = gelu2_bf16(v0[2], v0[3]); w.z = gelu2_bf16(v1[0], v1[1]); w.w = gelu2_bf16(v1[2], v1[3]);
    if (bad) w = (u32x4){0x7fc07fc0u, 0x7fc07fc0u, 0x7fc07fc0u, 0x7fc07fc0u};
}

__device__ __forceinline__ void gemm_r1(const Params& p, LAS unsigned char* lds, unsigned call, int pm, int pn) {
    const int tid = threadIdx.x, wid = __builtin_amdgcn_readfirstlane(tid >> 6), lane = tid & 63, wr = wid >> 2, wc = wid & 3, fr = lane & 15, fq = lane >> 4;
    const int rank = p.rank;
    unsigned char* wsl = p.ws[rank]; unsigned* fl = (unsigned*)wsl; unsigned* tmo = fl + F_TMO * 32;
    volatile LAS unsigned* lready = (volatile LAS unsigned*)(lds + STAGE_BYTES);
    const char* gA = (const char*)wsl + OFF_A + (size_t)pm * 2 * IMG + tid * 16;
    const char* gB = (const char*)wsl + OFF_B + (size_t)pn * 2 * IMG + tid * 16;
    const unsigned ldsw = (unsigned)wid * 1024u;
    const int aoff = lds_byte(wr * 64 + fr, fq * 8), boff = lds_byte(wc * 32 + fr, fq * 8);
    const unsigned* myflag = fl + (F_SFLAG + rank * 8 + (lane & 7)) * 32;
    int ready = 0;
#define SA(b, h) (((b) * 2 + (h)) * HTB)
#define SB(b, h) ((4 + (b) * 2 + (h)) * HTB)
#define STAGE(bufoff, gptr) do { _Pragma("unroll") for (int _i = 0; _i < 2; ++_i) \
        __builtin_amdgcn_global_load_lds((const unsigned*)((gptr) + _i * 8192), (LAS unsigned*)(lds + (bufoff) + ldsw + _i * 8192), 16, 0, 0); } while (0)
#define LDA(dst, b, h) do { _Pragma("unroll") for (int m = 0; m < 4; ++m) _Pragma("unroll") for (int k = 0; k < 2; ++k) dst[m][k] = *(const LAS bf16x8*)(lds + SA(b, h) + aoff + m * 2048 + k * 1024); } while (0)
#define LDB(dst, b, h) do { _Pragma("unroll") for (int n = 0; n < 2; ++n) _Pragma("unroll") for (int k = 0; k < 2; ++k) dst[n][k] = *(const LAS bf16x8*)(lds + SB(b, h) + boff + n * 2048 + k * 1024); } while (0)
#define MMA(ai, bj, At, Bt) do { __builtin_amdgcn_s_setprio(1); _Pragma("unroll") for (int m = 0; m < 4; ++m) _Pragma("unroll") for (int n = 0; n < 2; ++n) _Pragma("unroll") for (int k = 0; k < 2; ++k) \
        acc[ai][bj][m][n] = __builtin_amdgcn_mfma_f32_16x16x32_bf16(Bt[n][k], At[m][k], acc[ai][bj][m][n], 0, 0, 0); __builtin_amdgcn_s_setprio(0); } while (0)
    volatile LAS int* lorder = (volatile LAS int*)(lds + STAGE_BYTES + 64);
    unsigned long long sched = 0ull;
    const int mykt = kt_of(lane, rank);
#define NEED(nd) do { if ((nd) > ready) { \
        if (wid == 0) { const unsigned long long t0 = __builtin_amdgcn_s_memrealtime(); int cnt = ready; \
            for (;;) { \
                const unsigned long long sm = ld_sum(fl, pm & 1, call) << 8; \
                bool ok = !((sched >> lane) & 1ull) && (lane < 8 ? ld_sys(myflag) >= call : ((sm >> lane) & 1ull) != 0ull); \
                bool giveup = false; \
                if (cnt + __builtin_popcountll(__ballot(ok)) < (nd)) { \
                    giveup = __builtin_amdgcn_readfirstlane(__hip_atomic_load(tmo, __ATOMIC_RELAXED, __HIP_MEMORY_SCOPE_AGENT)) == call || __builtin_amdgcn_s_memrealtime() - t0 > DEADLINE; \
                    if (giveup) { if (lane == 0) { __hip_atomic_store(tmo, call, __ATOMIC_RELAXED, __HIP_MEMORY_SCOPE_AGENT); lready[1] = 1u; } ok = !((sched >> lane) & 1ull); } } \
                const unsigned long long bal = __ballot(ok); \
                if (ok) lorder[cnt + __builtin_popcountll(bal & ((1ull << lane) - 1ull))] = mykt; \
                sched |= bal; cnt += __builtin_popcountll(bal); \
                if (cnt >= (nd)) break; \
                if ((nd) > 10 && (nd) < ((pm & 1) ? 64 - 7 * ND + ODD_FAST : POLL_FAST_FROM)) __builtin_amdgcn_s_sleep(SLOW_SLEEP); else __builtin_amdgcn_s_sleep(POLL_SLEEP); } \
            ACQ_AGENT; \
            if (lane == 0) { *lready = (unsigned)cnt; \
                if (cnt == 64 && __hip_atomic_load(tmo, __ATOMIC_RELAXED, __HIP_MEMORY_SCOPE_AGENT) == call) lready[1] = 1u; }   \
            asm volatile("s_waitcnt vmcnt(0) lgkmcnt(0)" ::: "memory"); } \
        CFENCE; BAR; CFENCE; \
        ready = __builtin_amdgcn_readfirstlane((int)*lready); } } while (0)
#define KOFF(t) ((size_t)(unsigned)__builtin_amdgcn_readfirstlane(lorder[t]) * SLAB)

    f32x4 acc[2][2][4][2];
#pragma unroll
    for (int a = 0; a < 2; ++a)
#pragma unroll
        for (int b = 0; b < 2; ++b)
#pragma unroll
            for (int m = 0; m < 4; ++m)
#pragma unroll
                for (int n = 0; n < 2; ++n) acc[a][b][m][n] = (f32x4){0.f, 0.f, 0.f, 0.f};
    bf16x8 At[4][2], B0[2][2], B1[2][2];
    if (tid == 0) lready[1] = 0u;
    NEED(4);
    size_t o1, o2, o3;
    {
        const size_t k0 = KOFF(0); o1 = KOFF(1); o2 = KOFF(2); o3 = KOFF(3);
        STAGE(SB(0, 0), gB + k0); STAGE(SA(0, 0), gA + k0); STAGE(SB(0, 1), gB + k0 + IMG); STAGE(SA(0, 1), gA + k0 + IMG);
        if (wr == 1) BAR;
        WAIT_V(4); BAR;
        STAGE(SB(1, 0), gB + o1); STAGE(SA(1, 0), gA + o1); STAGE(SB(1, 1), gB + o1 + IMG);
        WAIT_V(6); BAR;
    }
    STAMP(2);
    for (int t = 0; t < 64; t += 2) {
        LDB(B0, 0, 0); SCHED; LDA(At, 0, 0); STAGE(SA(1, 1), gA + o1 + IMG);
        WAIT_L(8); BAR; WAIT_L(0); MMA(0, 0, At, B0); BAR; SCHED;
        LDB(B1, 0, 1); STAGE(SB(0, 0), gB + o2);
        BAR; WAIT_L(0); MMA(0, 1, At, B1); BAR;
        LDA(At, 0, 1); STAGE(SA(0, 0), gA + o2);
        BAR; WAIT_L(0); MMA(1, 0, At, B0); BAR; SCHED;
        STAGE(SB(0, 1), gB + o2 + IMG);
        WAIT_V(6); BAR; MMA(1, 1, At, B1); BAR;
        LDB(B0, 1, 0); SCHED; LDA(At, 1, 0); STAGE(SA(0, 1), gA + o2 + IMG);
        WAIT_L(8); BAR; WAIT_L(0); MMA(0, 0, At, B0); BAR; SCHED;
        LDB(B1, 1, 1); STAGE(SB(1, 0), gB + o3);
        BAR; WAIT_L(0); MMA(0, 1, At, B1); BAR;
        LDA(At, 1, 1); STAGE(SA(1, 0), gA + o3);
        BAR; WAIT_L(0); MMA(1, 0, At, B0); BAR; SCHED;
        STAGE(SB(1, 1), gB + o3 + IMG);
        WAIT_V(6); BAR; MMA(1, 1, At, B1); BAR;
        o1 = o3;
        if (t + 6 <= 64) { NEED(t + 6); o2 = KOFF(t + 4); o3 = KOFF(t + 5); }
    }
    STAMP(3);
    {
        int tz = threadIdx.x; asm volatile("" : "+v"(tz));
        const int lane2 = tz & 63, fr2 = lane2 & 15, fq2 = lane2 >> 4;
        const bool bad = lready[1] != 0u;
        const int owner = pm >> 1;
        if (wr == 0) BAR;
        if (owner == rank) {
            const auto rout = __builtin_amdgcn_make_buffer_rsrc(p.out, 0, (int)OUT_BYTES, 0x00020000);
            const unsigned base = (unsigned)((pm & 1) * 256 + wr * 64 + fr2) * 65536u + (unsigned)(rank * 4096 + pn * 256 + wc * 32 + fq2 * 8) * 2u;
#pragma unroll
            for (int ai = 0; ai < 2; ++ai)
#pragma unroll
                for (int m = 0; m < 4; ++m)
#pragma unroll
                    for (int bj = 0; bj < 2; ++bj) { u32x4 w; store_tile_rows(acc[ai][bj][m][0], acc[ai][bj][m][1], bad, w);
                        __builtin_amdgcn_raw_buffer_store_b128(w, rout, (int)(base + (unsigned)(ai * 128 + m * 16) * 65536u + (unsigned)bj * 256u), 0, 0); }
            WAIT_V(0); BAR; CFENCE;
        } else {
            const auto rs = __builtin_amdgcn_make_buffer_rsrc(wsl, 0, (int)WS_NEED, 0x00020000);
            const unsigned base = OFF_O + (unsigned)(pm * 16 + pn) * 131072u + (unsigned)(wr * 64 + fr2) * 512u + (unsigned)(wc * 32 + fq2 * 8) * 2u;
            unsigned* oflag = (unsigned*)p.ws[owner] + (F_OUT + rank * 64 + (pm & 1) * 14 + pn) * 32;
#pragma unroll
            for (int ai = 0; ai < 2; ++ai) {
#pragma unroll
                for (int m = 0; m < 4; ++m)
#pragma unroll
                    for (int bj = 0; bj < 2; ++bj) { u32x4 w; store_tile_rows(acc[ai][bj][m][0], acc[ai][bj][m][1], bad, w);
                        __builtin_amdgcn_raw_buffer_store_b128(w, rs, (int)(base + (unsigned)(ai * 128 + m * 16) * 512u + (unsigned)bj * 256u), 0, AUX_SYS); }
                WAIT_V(0); BAR; CFENCE;
                if (tz == 0) {
                    st_sys(oflag, 2u * call - 1u + (unsigned)ai);
                }
            }
        }
    }
#undef NEED
}

__device__ __forceinline__ void gemm_r2(const Params& p, LAS unsigned char* lds, unsigned call, int a, int b) {
    const int tid = threadIdx.x, wid = __builtin_amdgcn_readfirstlane(tid >> 6), lane = tid & 63, wr = wid >> 2, wc = wid & 3, fr = lane & 15, fq = lane >> 4;
    const int rank = p.rank;
    unsigned char* wsl = p.ws[rank]; unsigned* fl = (unsigned*)wsl; unsigned* tmo = fl + F_TMO * 32;
    if (tid < 64) {
        if (lane < 8) wait_ge(fl + (F_SFLAG + rank * 8 + lane) * 32, call, tmo, call);
        const unsigned long long t0 = __builtin_amdgcn_s_memrealtime();
        while (ld_sum(fl, (a >> 1) & 1, call) != 0x00ffffffffffffffull || ld_sum(fl, 2, call) != 0x7full) {
            if (__builtin_amdgcn_readfirstlane(__hip_atomic_load(tmo, __ATOMIC_RELAXED, __HIP_MEMORY_SCOPE_AGENT)) == call || __builtin_amdgcn_s_memrealtime() - t0 > DEADLINE) {
                if (lane == 0) __hip_atomic_store(tmo, call, __ATOMIC_RELAXED, __HIP_MEMORY_SCOPE_AGENT); break; }
            __builtin_amdgcn_s_sleep(16);
        }
        ACQ_AGENT; WAIT_V(0);
    }
    CFENCE; BAR; CFENCE;
    const char* gA = (const char*)wsl + OFF_A + (size_t)a * IMG + tid * 16;
    const char* gB = (const char*)wsl + OFF_B + (size_t)b * IMG + tid * 16;
    const unsigned ldsw = (unsigned)wid * 1024u;
    const int aoff = lds_byte(wr * 64 + fr, fq * 8), boff = lds_byte(wc * 32 + fr, fq * 8);
    f32x4 acc[4][2];
#pragma unroll
    for (int m = 0; m < 4; ++m)
#pragma unroll
        for (int n = 0; n < 2; ++n) acc[m][n] = (f32x4){0.f, 0.f, 0.f, 0.f};
#define STAGE2(slot, kt) do { STAGE((slot) * 32768, gA + (size_t)(kt) * SLAB); STAGE((slot) * 32768 + 16384, gB + (size_t)(kt) * SLAB); } while (0)
#define FRAG2(Af, Bf, slot) do { \
        _Pragma("unroll") for (int n = 0; n < 2; ++n) _Pragma("unroll") for (int k = 0; k < 2; ++k) Bf[n][k] = *(const LAS bf16x8*)(lds + (slot) * 32768 + 16384 + boff + n * 2048 + k * 1024); \
        _Pragma("unroll") for (int m = 0; m < 4; ++m) _Pragma("unroll") for (int k = 0; k < 2; ++k) Af[m][k] = *(const LAS bf16x8*)(lds + (slot) * 32768 + aoff + m * 2048 + k * 1024); } while (0)
#define MMA2(Af, Bf) do { __builtin_amdgcn_s_setprio(1); _Pragma("unroll") for (int k = 0; k < 2; ++k) _Pragma("unroll") for (int m = 0; m < 4; ++m) _Pragma("unroll") for (int n = 0; n < 2; ++n) \
        acc[m][n] = __builtin_amdgcn_mfma_f32_16x16x32_bf16(Bf[n][k], Af[m][k], acc[m][n], 0, 0, 0); __builtin_amdgcn_s_setprio(0); } while (0)
    bf16x8 A0[4][2], Bf0[2][2], A1[4][2], Bf1[2][2];
    STAMP(6);
    STAGE2(0, 0); STAGE2(1, 1); STAGE2(2, 2); STAGE2(3, 3);
    WAIT_V(12); BAR; CFENCE;
    FRAG2(A0, Bf0, 0);
    for (int t = 0; t < 64; t += 4) {
        __builtin_amdgcn_s_waitcnt(0xC07F); WAIT_V(8); BAR; SCHED; STAGE2(0, (t + 4) & 63); FRAG2(A1, Bf1, 1); SCHED; MMA2(A0, Bf0); SCHED;
        __builtin_amdgcn_s_waitcnt(0xC07F); WAIT_V(8); BAR; SCHED; STAGE2(1, (t + 5) & 63); FRAG2(A0, Bf0, 2); SCHED; MMA2(A1, Bf1); SCHED;
        __builtin_amdgcn_s_waitcnt(0xC07F); WAIT_V(8); BAR; SCHED; STAGE2(2, (t + 6) & 63); FRAG2(A1, Bf1, 3); SCHED; MMA2(A0, Bf0); SCHED;
        __builtin_amdgcn_s_waitcnt(0xC07F); WAIT_V(8); BAR; SCHED; STAGE2(3, (t + 7) & 63); FRAG2(A0, Bf0, 0); SCHED; MMA2(A1, Bf1); SCHED;
    }
    WAIT_L(0);
    STAMP(7);
    const bool bad = __hip_atomic_load(tmo, __ATOMIC_RELAXED, __HIP_MEMORY_SCOPE_AGENT) == call;
    const int owner = a >> 2;
    if (owner == rank) {
        const auto rout = __builtin_amdgcn_make_buffer_rsrc(p.out, 0, (int)OUT_BYTES, 0x00020000);
        const unsigned base = (unsigned)((a & 3) * 128 + wr * 64 + fr) * 65536u + (unsigned)(rank * 4096 + b * 128 + wc * 32 + fq * 8) * 2u;
#pragma unroll
        for (int m = 0; m < 4; ++m) { u32x4 w; store_tile_rows(acc[m][0], acc[m][1], bad, w);
            __builtin_amdgcn_raw_buffer_store_b128(w, rout, (int)(base + (unsigned)(m * 16) * 65536u), 0, 0); }
    } else {
        const auto rs = __builtin_amdgcn_make_buffer_rsrc(wsl, 0, (int)WS_NEED, 0x00020000);
        const unsigned base = OFF_O + (unsigned)((a >> 1) * 16 + (b >> 1)) * 131072u + (unsigned)((a & 1) * 2 + (b & 1)) * 32768u + (unsigned)(wr * 64 + fr) * 256u + (unsigned)(wc * 32 + fq * 8) * 2u;
#pragma unroll
        for (int m = 0; m < 4; ++m) { u32x4 w; store_tile_rows(acc[m][0], acc[m][1], bad, w);
            __builtin_amdgcn_raw_buffer_store_b128(w, rs, (int)(base + (unsigned)(m * 16) * 256u), 0, AUX_SYS); }
    }
    WAIT_V(0); BAR; CFENCE;
    if (owner != rank && tid == 0) {
        st_sys((unsigned*)p.ws[owner] + (F_OUT + rank * 64 + 28 + (a & 3) * 4 + (b - 28)) * 32, call);
    }
}

__global__ __launch_bounds__(512, 2) void fused_gemm_a2a(Params p) {
    extern __shared__ __attribute__((aligned(16))) unsigned char shm[];
    LAS unsigned char* lds = (LAS unsigned char*)shm;
    const int tid = threadIdx.x, bid = blockIdx.x;
    unsigned* fl = (unsigned*)p.ws[p.rank];
    volatile LAS unsigned* lcall = (volatile LAS unsigned*)(lds + STAGE_BYTES + 16);
    if (tid == 0) *lcall = __hip_atomic_fetch_add(fl + F_TICKET * 32, 1u, __ATOMIC_RELAXED, __HIP_MEMORY_SCOPE_AGENT) / (unsigned)NWG + 1u;
    STAMP(0);
    const bool is_puller = bid >= NSTAGER;
    if (!is_puller) stage_units(p, lds, lcall, bid);
    __syncthreads();
    const unsigned call = __builtin_amdgcn_readfirstlane(*lcall);
    if (is_puller) {
        const int w = __builtin_amdgcn_readfirstlane(tid >> 6), x = bid & 7, k = x < 4 ? (bid - 232) >> 3 : (bid - 228) >> 3;
        const unsigned cl = 0xB2u, mine = (cl >> p.rank) & 1u;
        unsigned m = (x < 4 ? (mine ? cl : ~cl) : (mine ? ~cl : cl)) & 0xffu & ~(1u << p.rank);
        for (int i = 0; i < k; ++i) m &= m - 1u;
        const int s = __builtin_ctz(m);
        puller(p, call, (s - p.rank - 1) & 7, (x & 3) * 8 + w);
        STAMP(19); return;
    }
    STAMP(1);
    __syncthreads();
    const int xcd = bid & 7, idx = bid >> 3, rg = xcd >> 1, cg = xcd & 1;
    const int n1 = 28, nw = xcd < 4 ? 29 : 28;
    if (idx < n1) {
        gemm_r1(p, lds, call, rg * 4 + (idx & 3), cg * 7 + (idx >> 2));
        STAMP(4);
    } else if (bid == 224 && tid < 64) monitor(p, call);
    volatile LAS unsigned* lq = (volatile LAS unsigned*)(lds + STAGE_BYTES + 32);
    for (;;) {
        __syncthreads();
        if (tid == 0) *lq = __hip_atomic_fetch_add(fl + (F_Q2 + xcd) * 32, 1u, __ATOMIC_RELAXED, __HIP_MEMORY_SCOPE_AGENT) - (call - 1u) * (unsigned)(16 + nw);
        __syncthreads();
        const unsigned r2 = __builtin_amdgcn_readfirstlane(*lq);
        if (r2 >= 16u) break;
        gemm_r2(p, lds, call, rg * 8 + (int)(r2 & 7u), 28 + cg * 2 + (int)(r2 >> 3));
        STAMP(5);
    }
}
}

extern "C" void kernel_launch(int rank, int nranks, ncclComm_t comm,
                              hipStream_t stream,
                              void* const* d_in, const int* in_sizes, int n_in,
                              void* d_out, int out_size,
                              const int* in_dtypes, int out_dtype,
                              void* const* d_out_all,
                              void* const* d_ws_all, size_t ws_bytes) {
    if (nranks != NR || n_in != 2 || ws_bytes < (size_t)WS_NEED || (unsigned)rank >= (unsigned)NR) { fprintf(stderr, "kernel_launch: unexpected configuration\n"); return; }
    Params p;
    p.x = (const unsigned short*)d_in[0]; p.w = (const unsigned short*)d_in[1]; p.out = (unsigned short*)d_out; p.rank = rank;
    for (int i = 0; i < NR; ++i) p.ws[i] = (unsigned char*)d_ws_all[i];
    HIP_CHECK(hipFuncSetAttribute((const void*)fused_gemm_a2a, hipFuncAttributeMaxDynamicSharedMemorySize, LDS_BYTES));
    hipLaunchKernelGGL(fused_gemm_a2a, dim3(NWG), dim3(512), LDS_BYTES, stream, p);
    HIP_CHECK(hipGetLastError());
}
```

```cpp
#include <hip/hip_runtime.h>
#include <rccl/rccl.h>
#include <cstdio>
#include <cstdint>

#define HIP_CHECK(expr) do { hipError_t _e = (expr); if (_e != hipSuccess) { \
    fprintf(stderr, "HIP error %s at %s:%d\n", hipGetErrorString(_e), __FILE__, __LINE__); \
    return; } } while (0)

namespace {
typedef unsigned u32x4 __attribute__((ext_vector_type(4)));
typedef short bf16x8 __attribute__((ext_vector_type(8)));
typedef float f32x4 __attribute__((ext_vector_type(4)));
#define LAS __attribute__((address_space(3)))

constexpr int NR = 8;
constexpr int MM = 4096, KS = 512, NN = 32768, NS = 4096, MS = 512;
constexpr unsigned IMG = 16384;
constexpr unsigned SLAB = 32 * IMG;
constexpr unsigned OFF_A = 1u << 20;
constexpr unsigned OFF_B = OFF_A + 64 * SLAB;
constexpr unsigned OFF_W = OFF_B + 64 * SLAB;
constexpr unsigned OFF_O = OFF_W + 64 * SLAB;
constexpr unsigned WS_NEED = OFF_O + 256u * 131072u;
constexpr unsigned OUT_BYTES = (unsigned)MS * NN * 2u;
constexpr int F_TICKET = 0, F_TMO = 1, F_SCNT = 2, F_SFLAG = 16, F_INCNT = 80, F_INB = 144, F_Q = 152, F_OUT = 160  , F_INO = 672, F_Q2 = 736, F_SUM = 744, F_OFF = 745, F_XF = 768  , F_UF = 1024  ;
constexpr int NPULL = 28, NWG = 256;
constexpr int NSTAGER = NWG - NPULL;
constexpr int UNITS_PER_CHUNK = 72, NUNITS = 8 * UNITS_PER_CHUNK;
constexpr unsigned long long DEADLINE = 50000000ull;
constexpr int HTB = 16384, STAGE_BYTES = 8 * HTB, LDS_BYTES = STAGE_BYTES + 512;
constexpr int AUX_SYS = 17, AUX_AGENT = 16;
#define MON_SLEEP 8
#define ODD_FAST 6
#define SLOW_SLEEP 127
#define POLL_FAST_FROM 56
#define POLL_SLEEP 16

struct Params { const unsigned short* x; const unsigned short* w; unsigned char* ws[NR]; unsigned short* out; int rank; };

__device__ __forceinline__ int lds_byte(int r, int c) { const int st = (r >> 4) * 2 + (c >> 5), rr = r & 15, cc = c & 31, ob = rr * 64 + cc * 2; return st * 1024 + (ob ^ (((ob >> 9) & 1) << 5)); }
__device__ __forceinline__ void stage_rc(int b, int& R, int& C) { const int st = b / 1024, sb = b % 1024, swz = sb ^ (((sb >> 9) & 1) << 5); R = (st >> 1) * 16 + swz / 64; C = (st & 1) * 32 + (swz % 64) / 2; }

__device__ __forceinline__ unsigned ld_sys(const unsigned* p) { return __hip_atomic_load(p, __ATOMIC_RELAXED, __HIP_MEMORY_SCOPE_SYSTEM); }
__device__ __forceinline__ void st_sys(unsigned* p, unsigned v) { __hip_atomic_store(p, v, __ATOMIC_RELAXED, __HIP_MEMORY_SCOPE_SYSTEM); }
__device__ __forceinline__ void wait_ge(const unsigned* flag, unsigned val, unsigned* tmo, unsigned call) {
    const unsigned long long t0 = __builtin_amdgcn_s_memrealtime();
    while (ld_sys(flag) < val) {
        if (__hip_atomic_load(tmo, __ATOMIC_RELAXED, __HIP_MEMORY_SCOPE_AGENT) == call || __builtin_amdgcn_s_memrealtime() - t0 > DEADLINE) {
            __hip_atomic_store(tmo, call, __ATOMIC_RELAXED, __HIP_MEMORY_SCOPE_AGENT); break; }
        __builtin_amdgcn_s_sleep(2);
    }
}
__device__ __forceinline__ unsigned cvt_pk_bf16(float lo, float hi) { unsigned r; asm volatile("v_cvt_pk_bf16_f32 %0, %1, %2" : "=v"(r) : "v"(lo), "v"(hi)); return r; }
typedef float f32x2 __attribute__((ext_vector_type(2)));
__device__ __forceinline__ unsigned gelu2_bf16(float y0, float y1) {
    const f32x2 y = {y0, y1};
    const f32x2 t = (y * y) * -0.1029432395800235f + -2.302208198144325f;
    const f32x2 a = y * t;
    f32x2 d; d.x = __builtin_amdgcn_exp2f(a.x); d.y = __builtin_amdgcn_exp2f(a.y);
    d = d + 1.0f;
    f32x2 r; r.x = __builtin_amdgcn_rcpf(d.x); r.y = __builtin_amdgcn_rcpf(d.y);
    r = r * y;
    return cvt_pk_bf16(r.x, r.y);
}
#define STAMP(i) do { } while (0)
#define WAIT_V(n) asm volatile("s_waitcnt vmcnt(" #n ")" ::: "memory")
#define WAIT_L(n) asm volatile("s_waitcnt lgkmcnt(" #n ")" ::: "memory")
#define BAR __builtin_amdgcn_s_barrier()
#define SCHED __builtin_amdgcn_sched_barrier(0)
#define CFENCE asm volatile("" ::: "memory")
#define ACQ_AGENT CFENCE

__device__ __forceinline__ void stage_units(const Params& p, LAS unsigned char* lds, volatile LAS unsigned* lcall, int first) {
    const int tid = threadIdx.x, rank = p.rank;
    unsigned char* wsl = p.ws[rank]; unsigned* fl = (unsigned*)wsl;
    const auto rs = __builtin_amdgcn_make_buffer_rsrc(wsl, 0, (int)WS_NEED, 0x00020000);
    if (first < 32) {
        const unsigned dbase = OFF_A + (unsigned)(rank * 8) * SLAB + (unsigned)first * IMG;
        u32x4 t[2];
#pragma unroll
        for (int i = 0; i < 2; ++i) { int R, C; stage_rc((tid + 512 * i) * 16, R, C); t[i] = *(const u32x4*)(p.x + (size_t)(first * 128 + R) * KS + C); }
#pragma unroll
        for (int i = 0; i < 2; ++i) __builtin_amdgcn_raw_buffer_store_b128(t[i], rs, (int)(dbase + (unsigned)(tid + 512 * i) * 16u), 0, AUX_SYS);
        WAIT_V(0);
        __syncthreads();
        const unsigned call = *lcall;
        if (tid < NR) { if (tid != rank) st_sys((unsigned*)p.ws[tid] + (F_XF + rank * 32 + first) * 32, call); }
        else if (tid == 64) {
            const unsigned old = __hip_atomic_fetch_add(fl + F_SCNT * 32, 1u, __ATOMIC_RELAXED, __HIP_MEMORY_SCOPE_AGENT);
            if (old + 1u == call * 96u) st_sys(fl + (F_SFLAG + rank * 8) * 32, call);
        }
    }
    for (int u = 8 + (first + NSTAGER - 32) % NSTAGER; u < NUNITS; u += NSTAGER) {
        const int c = u / UNITS_PER_CHUNK, v0 = u % UNITS_PER_CHUNK, v = v0 < 8 ? 64 + v0 : v0 - 8;
        if (v >= 64) {
            const int vv = v - 64;
            const unsigned dbase = OFF_A + (unsigned)(rank * 8 + c) * SLAB + (unsigned)vv * 4u * IMG;
            u32x4 t[8];
#pragma unroll
            for (int i = 0; i < 8; ++i) { const int pc = tid + 512 * i, img = pc >> 10; int R, C; stage_rc((pc & 1023) * 16, R, C);
                t[i] = *(const u32x4*)(p.x + (size_t)(vv * 512 + img * 128 + R) * KS + c * 64 + C); }
#pragma unroll
            for (int i = 0; i < 8; ++i) __builtin_amdgcn_raw_buffer_store_b128(t[i], rs, (int)(dbase + (unsigned)(tid + 512 * i) * 16u), 0, AUX_SYS);
        } else {
            const int d = v >> 3, kb = tid >> 6, nb = tid & 63;
            const unsigned dbase = (d == rank ? OFF_B + (unsigned)(rank * 8 + c) * SLAB : OFF_W + (unsigned)(d * 8 + c) * SLAB) + (unsigned)(v & 7) * 4u * IMG;
            u32x4 r[8];
#pragma unroll
            for (int i = 0; i < 8; ++i) r[i] = *(const u32x4*)(p.w + (size_t)(c * 64 + kb * 8 + i) * NN + v * 512 + nb * 8);
#pragma unroll
            for (int j = 0; j < 8; ++j) {
                u32x4 o;
#pragma unroll
                for (int e = 0; e < 4; ++e) o[e] = __builtin_amdgcn_perm(r[2 * e + 1][j >> 1], r[2 * e][j >> 1], (j & 1) ? 0x07060302u : 0x05040100u);
                const int nl = nb * 8 + j, img = nl >> 7, rowi = nl & 127, cc = rowi & 31;
                const int rho = (rowi & ~31) + 16 * ((cc >> 2) & 1) + 4 * (cc >> 3) + (cc & 3);
                *(LAS u32x4*)(lds + img * IMG + lds_byte(rho, kb * 8)) = o;
            }
            __syncthreads();
#pragma unroll
            for (int i = 0; i < 8; ++i) { const unsigned b = (unsigned)(tid + 512 * i) * 16u; const u32x4 t = *(const LAS u32x4*)(lds + b);
                __builtin_amdgcn_raw_buffer_store_b128(t, rs, (int)(dbase + b), 0, AUX_SYS); }
        }
        WAIT_V(0);
        __syncthreads();
        const unsigned call = *lcall;
        if (tid < NR) {
            if (tid != rank && (v >= 64 || tid == (v >> 3)))
                st_sys((unsigned*)p.ws[tid] + (F_UF + (rank * 8 + c) * 16 + (v >= 64 ? v - 64 : 8 + (v & 7))) * 32, call);
        } else if (tid == 64) {
            const unsigned old = __hip_atomic_fetch_add(fl + (F_SCNT + c) * 32, 1u, __ATOMIC_RELAXED, __HIP_MEMORY_SCOPE_AGENT);
            if (old + 1u == call * (c == 0 ? 96u : (unsigned)UNITS_PER_CHUNK)) st_sys(fl + (F_SFLAG + rank * 8 + c) * 32, call);
        }
    }
}

struct Desc { unsigned soff, doff; int kind; int flag; int sig; int half; };
constexpr int NITEM = 1536, NFETCH = NITEM - 64 + 32;
#define ND 4
constexpr int I_E0 = (8 - ND) * 120, I_O0 = I_E0 + ND * 88;
constexpr unsigned INE_ITEMS = 88, INO_ITEMS = 32, INB_ITEMS = 64;
__device__ __forceinline__ Desc puller_desc(int it, int rank, int s) {
    Desc d; d.sig = -1; d.kind = 0; d.half = -1;
    if (it < 960) {
        int c, j;
        if (it < I_E0) { c = it / 120; j = it - c * 120; }
        else if (it < I_O0) { c = (8 - ND) + (it - I_E0) / 88; j = (it - I_E0) - (c - (8 - ND)) * 88; j = j < 32 ? ((j >> 2) << 3) + (j & 3) : j + 32; }
        else { c = (8 - ND) + ((it - I_O0) >> 5); j = (it - I_O0) & 31; j = ((j >> 2) << 3) + 4 + (j & 3); }
        d.flag = it < 64 ? F_XF + s * 32 + (j >> 1) : F_UF + (s * 8 + c) * 16 + (j >> 3); d.sig = ((j < 64 && (j & 4)) ? F_INO : F_INCNT) + s * 8 + c;
        if (j < 64) { d.soff = OFF_A + (unsigned)(8 * s + c) * SLAB + (unsigned)j * 8192u; d.doff = d.soff; }
        else { const unsigned off = (unsigned)(j - 64) * 8192u; d.soff = OFF_W + (unsigned)(rank * 8 + c) * SLAB + off; d.doff = OFF_B + (unsigned)(8 * s + c) * SLAB + off; }
    } else if (it < 1024) {
        const int c = (it - 960) >> 3, j = (it - 960) & 7; const unsigned off = 28u * IMG + (unsigned)j * 8192u;
        d.flag = F_UF + (s * 8 + c) * 16 + 15; d.sig = F_INB + s;
        d.soff = OFF_W + (unsigned)(rank * 8 + c) * SLAB + off; d.doff = OFF_B + (unsigned)(8 * s + c) * SLAB + off;
    } else if (it < 1472) {
        const int idx = (it - 1024) >> 4, j = (it - 1024) & 15, pml = idx / 14, pn = idx - pml * 14;
        d.kind = 1; d.flag = F_OUT + s * 64 + idx; d.half = j >> 3;
        d.soff = OFF_O + (unsigned)((2 * rank + pml) * 16 + pn) * 131072u + (unsigned)j * 8192u;
        d.doff = (unsigned)(pml * 256 + j * 16) * 65536u + (unsigned)(s * 4096 + pn * 256) * 2u;
    } else {
        const int idx = (it - 1472) >> 2, j = (it - 1472) & 3, al = idx >> 2, b = 28 + (idx & 3);
        d.kind = 2; d.flag = F_OUT + s * 64 + 28 + idx;
        d.soff = OFF_O + (unsigned)((2 * rank + (al >> 1)) * 16 + (b >> 1)) * 131072u + (unsigned)((al & 1) * 2 + (b & 1)) * 32768u + (unsigned)j * 8192u;
        d.doff = (unsigned)(al * 128 + j * 32) * 65536u + (unsigned)(s * 4096 + b * 128) * 2u;
    }
    return d;
}
typedef int i32x4 __attribute__((ext_vector_type(4)));
__device__ __forceinline__ i32x4 mk_rsrc(const void* ptr, unsigned bytes) {
    const unsigned long long a = (unsigned long long)ptr; i32x4 r;
    r.x = __builtin_amdgcn_readfirstlane((int)(unsigned)a); r.y = __builtin_amdgcn_readfirstlane((int)(unsigned)(a >> 32) & 0xffff); r.z = (int)bytes; r.w = 0x00020000; return r;
}
#define RING_CLOB "v200","v201","v202","v203","v204","v205","v206","v207","v208","v209","v210","v211","v212","v213","v214","v215"
#define R_LD(r, off_) asm volatile("buffer_load_dwordx4 v[%c2:%c3], %0, %1, 0 offen sc0 sc1" :: "v"(off_), "s"(rsrc), "i"(200 + 4 * (r)), "i"(203 + 4 * (r)) : "memory", RING_CLOB)
#define R_ST(r, off_, rs_, wait, bits) asm volatile(wait "buffer_store_dwordx4 v[%c2:%c3], %0, %1, 0 offen" bits "\n\ts_nop 1" :: "v"(off_), "s"(rs_), "i"(200 + 4 * (r)), "i"(203 + 4 * (r)) : "memory", RING_CLOB)
#define R_NAN(r, wait) asm volatile(wait "v_mov_b32 v%c0, 0x7fc07fc0\n\tv_mov_b32 v%c1, 0x7fc07fc0\n\tv_mov_b32 v%c2, 0x7fc07fc0\n\tv_mov_b32 v%c3, 0x7fc07fc0" \
        :: "i"(200 + 4 * (r)), "i"(201 + 4 * (r)), "i"(202 + 4 * (r)), "i"(203 + 4 * (r)) : "memory", RING_CLOB)
__device__ __forceinline__ void puller(const Params& p, unsigned call, int l, int wl) {
    const int lane = threadIdx.x & 63, rank = p.rank, s = (rank + 1 + l) & 7;
    unsigned* fl = (unsigned*)p.ws[rank]; unsigned* tmo = fl + F_TMO * 32; unsigned* qctr = fl + (F_Q + l) * 32;
    const unsigned qbase = (call - 1u) * (unsigned)NFETCH - 64u;
    const i32x4 rsrc = mk_rsrc(p.ws[s], WS_NEED), rdst = mk_rsrc(p.ws[rank], WS_NEED), rout = mk_rsrc(p.out, OUT_BYTES);
    const unsigned l0 = (unsigned)lane * 16u, l1 = (unsigned)(lane >> 5) * 65536u + (unsigned)(lane & 31) * 16u, l2 = (unsigned)(lane >> 4) * 65536u + (unsigned)(lane & 15) * 16u;
    int cur = wl, nxt = cur + 32;
    bool cur_ready = false, have_prev = false, pbad = false, bad = false; int lastph = -1; (void)lastph;
    unsigned pdoff = 0, plane = 0, pstep = 0; int pkind = 0, psig = -1;
#define P_STORE(r, j, doff_, kind_, lane_, step_, bad_, wait) do { const unsigned o_ = (doff_) + (lane_) + (unsigned)(j) * (step_); \
        if ((kind_) == 0) R_ST(r, o_, rdst, wait, " sc1"); else { if (bad_) R_NAN(r, wait); R_ST(r, o_, rout, wait, ""); } } while (0)
#define P_SIG(sg) do { if ((sg) >= 0 && lane == 0) __hip_atomic_fetch_add(fl + (sg) * 32, 1u, __ATOMIC_RELAXED, __HIP_MEMORY_SCOPE_AGENT); } while (0)
#define P_FLUSH() do { if (have_prev) { WAIT_V(0); _Pragma("unroll") for (int r = 0; r < 4; ++r) P_STORE(r, 4 + r, pdoff, pkind, plane, pstep, pbad, ""); \
            WAIT_V(0); P_SIG(psig); have_prev = false; } } while (0)
    while (cur < NITEM) {
        const Desc dc = puller_desc(cur, rank, s);
        if (pkind == 0 && dc.kind != 0) P_FLUSH();
        if (!cur_ready) {
            P_FLUSH();
            if (lane == 0) wait_ge(fl + dc.flag * 32, dc.half < 0 ? call : 2u * call - 1u + (unsigned)dc.half, tmo, call);
            bad = __builtin_amdgcn_readfirstlane(__hip_atomic_load(tmo, __ATOMIC_RELAXED, __HIP_MEMORY_SCOPE_AGENT)) == call;
        }
        unsigned ftgt = call;
        if (nxt < NITEM) {
            const Desc dn = puller_desc(nxt, rank, s);
            if (dn.half >= 0) ftgt = 2u * call - 1u + (unsigned)dn.half;
            asm volatile("v_mov_b32 v216, -1\n\tv_mov_b32 v217, -1\n\ts_nop 0\n\tglobal_load_dword v216, %0, off sc0 sc1" :: "v"(fl + dn.flag * 32) : "memory", "v216", "v217");
            if (lane == 0) asm volatile("global_atomic_add v217, %0, %1, off sc0" :: "v"(qctr), "v"(1u) : "memory", "v217");
        }
        const unsigned clane = dc.kind == 0 ? l0 : (dc.kind == 1 ? l1 : l2), cstep = dc.kind == 0 ? 1024u : (dc.kind == 1 ? 131072u : 262144u);
#pragma unroll
        for (int j = 0; j < 8; ++j) {
            const int r = j & 3;
            if (j >= 4) P_STORE(r, j - 4, dc.doff, dc.kind, clane, cstep, bad, "s_waitcnt vmcnt(3)\n\t");
            else if (have_prev) P_STORE(r, 4 + j, pdoff, pkind, plane, pstep, pbad, "s_waitcnt vmcnt(3)\n\t");
            R_LD(r, dc.soff + l0 + (unsigned)j * 1024u);
        }
        if (have_prev) P_SIG(psig);
        unsigned fv, tk;
        asm volatile("v_mov_b32 %0, v216\n\tv_mov_b32 %1, v217" : "=&v"(fv), "=&v"(tk) :: "memory", "v216", "v217");
        if (nxt < NITEM && (__builtin_amdgcn_readfirstlane(fv) == -1 || __builtin_amdgcn_readfirstlane(tk) == -1))
            asm volatile("s_waitcnt vmcnt(0)\n\tv_mov_b32 %0, v216\n\tv_mov_b32 %1, v217" : "=&v"(fv), "=&v"(tk) :: "memory", "v216", "v217");
        have_prev = true; pdoff = dc.doff; pkind = dc.kind; plane = clane; pstep = cstep; psig = dc.sig; pbad = bad;
        cur_ready = nxt < NITEM && (unsigned)__builtin_amdgcn_readfirstlane(fv) >= ftgt;
        const unsigned tn = (unsigned)__builtin_amdgcn_readfirstlane(tk) - qbase;
        const int nn = (nxt < NITEM && tn < (unsigned)NITEM) ? (int)tn : NITEM;
        if ((cur >= I_O0 - 32 && cur < I_O0) || (cur >= 960 - 32 && cur < 960) || (cur >= 1024 - 32 && cur < 1024)) P_FLUSH();
        cur = nxt; nxt = nn;
    }
    P_FLUSH();
}

__device__ __forceinline__ unsigned long long ld_sum(const unsigned* fl, int w, unsigned call) {
    const unsigned long long v = __hip_atomic_load((const unsigned long long*)(fl + F_SUM * 32) + w, __ATOMIC_RELAXED, __HIP_MEMORY_SCOPE_AGENT);
    return (unsigned)(v >> 56) == (call & 0xffu) ? (v & 0x00ffffffffffffffull) : 0ull;
}
__device__ __forceinline__ void monitor(const Params& p, unsigned call) {
    const int lane = threadIdx.x & 63, rank = p.rank;
    unsigned* fl = (unsigned*)p.ws[rank]; unsigned* tmo = fl + F_TMO * 32;
    unsigned long long* sum = (unsigned long long*)(fl + F_SUM * 32);
    const int c = lane / 7, l = lane - c * 7, s = (rank + 1 + l) & 7;
    const unsigned* pe = fl + (F_INCNT + s * 8 + (c & 7)) * 32; const unsigned* po = fl + (F_INO + s * 8 + (c & 7)) * 32; const unsigned* pb = fl + (F_INB + s) * 32;
    const unsigned long long tag = (unsigned long long)(call & 0xffu) << 56, full = 0x00ffffffffffffffull;
    unsigned long long me = 0, mo = 0, mb = 0;
    const unsigned long long t0 = __builtin_amdgcn_s_memrealtime();
    for (;;) {
        const int fe = (int)__builtin_ctzll(~me), fo = (int)__builtin_ctzll(~mo);
        const bool ae = lane < 56 && !((me >> lane) & 1ull) && lane < fe + 14, ao = lane < 56 && !((mo >> lane) & 1ull) && lane < fo + 14 && ((me >> lane) & 1ull);
        const bool ab = lane < 7 && !((mb >> lane) & 1ull) && me == full;
        const bool be = ae && ld_sys(pe) >= INE_ITEMS * call, bo = ao && ld_sys(po) >= INO_ITEMS * call, bb = ab && ld_sys(pb) >= INB_ITEMS * call;
        const unsigned long long ne = me | __ballot(be), no = mo | __ballot(bo), nb = mb | __ballot(bb);
        if (lane == 0) {
            if (ne != me) __hip_atomic_store(sum + 0, ne | tag, __ATOMIC_RELAXED, __HIP_MEMORY_SCOPE_AGENT);
            if (no != mo) __hip_atomic_store(sum + 1, no | tag, __ATOMIC_RELAXED, __HIP_MEMORY_SCOPE_AGENT);
            if (nb != mb) __hip_atomic_store(sum + 2, nb | tag, __ATOMIC_RELAXED, __HIP_MEMORY_SCOPE_AGENT);
        }
        me = ne; mo = no; mb = nb;
        if (me == full && mo == full && mb == 0x7full) break;
        if (__builtin_amdgcn_readfirstlane(__hip_atomic_load(tmo, __ATOMIC_RELAXED, __HIP_MEMORY_SCOPE_AGENT)) == call || __builtin_amdgcn_s_memrealtime() - t0 > DEADLINE) {
            if (lane == 0) __hip_atomic_store(tmo, call, __ATOMIC_RELAXED, __HIP_MEMORY_SCOPE_AGENT); break; }
        __builtin_amdgcn_s_sleep(MON_SLEEP);
    }
}

__device__ __forceinline__ int kt_of(int t, int rank) {
    if (t < 8) return rank * 8 + t;
    const int tt = t - 8, c = tt / 7, l = tt - c * 7; return ((rank + 1 + l) & 7) * 8 + c;
}
__device__ __forceinline__ void store_tile_rows(const f32x4& v0, const f32x4& v1, bool bad, u32x4& w) {
    w.x = gelu2_bf16(v0[0], v0[1]); w.y = gelu2_bf16(v0[2], v0[3]); w.z = gelu2_bf16(v1[0], v1[1]); w.w = gelu2_bf16(v1[2], v1[3]);
    if (bad) w = (u32x4){0x7fc07fc0u, 0x7fc07fc0u, 0x7fc07fc0u, 0x7fc07fc0u};
}

__device__ __forceinline__ void gemm_r1(const Params& p, LAS unsigned char* lds, unsigned call, int pm, int pn) {
    const int tid = threadIdx.x, wid = __builtin_amdgcn_readfirstlane(tid >> 6), lane = tid & 63, wr = wid >> 2, wc = wid & 3, fr = lane & 15, fq = lane >> 4;
    const int rank = p.rank;
    unsigned char* wsl = p.ws[rank]; unsigned* fl = (unsigned*)wsl; unsigned* tmo = fl + F_TMO * 32;
    volatile LAS unsigned* lready = (volatile LAS unsigned*)(lds + STAGE_BYTES);
    const char* gA = (const char*)wsl + OFF_A + (size_t)pm * 2 * IMG + tid * 16;
    const char* gB = (const char*)wsl + OFF_B + (size_t)pn * 2 * IMG + tid * 16;
    const unsigned ldsw = (unsigned)wid * 1024u;
    const int aoff = lds_byte(wr * 64 + fr, fq * 8), boff = lds_byte(wc * 32 + fr, fq * 8);
    const unsigned* myflag = fl + (F_SFLAG + rank * 8 + (lane & 7)) * 32;
    int ready = 0;
#define SA(b, h) (((b) * 2 + (h)) * HTB)
#define SB(b, h) ((4 + (b) * 2 + (h)) * HTB)
#define STAGE(bufoff, gptr) do { _Pragma("unroll") for (int _i = 0; _i < 2; ++_i) \
        __builtin_amdgcn_global_load_lds((const unsigned*)((gptr) + _i * 8192), (LAS unsigned*)(lds + (bufoff) + ldsw + _i * 8192), 16, 0, 0); } while (0)
#define LDA(dst, b, h) do { _Pragma("unroll") for (int m = 0; m < 4; ++m) _Pragma("unroll") for (int k = 0; k < 2; ++k) dst[m][k] = *(const LAS bf16x8*)(lds + SA(b, h) + aoff + m * 2048 + k * 1024); } while (0)
#define LDB(dst, b, h) do { _Pragma("unroll") for (int n = 0; n < 2; ++n) _Pragma("unroll") for (int k = 0; k < 2; ++k) dst[n][k] = *(const LAS bf16x8*)(lds + SB(b, h) + boff + n * 2048 + k * 1024); } while (0)
#define MMA(ai, bj, At, Bt) do { __builtin_amdgcn_s_setprio(1); _Pragma("unroll") for (int m = 0; m < 4; ++m) _Pragma("unroll") for (int n = 0; n < 2; ++n) _Pragma("unroll") for (int k = 0; k < 2; ++k) \
        acc[ai][bj][m][n] = __builtin_amdgcn_mfma_f32_16x16x32_bf16(Bt[n][k], At[m][k], acc[ai][bj][m][n], 0, 0, 0); __builtin_amdgcn_s_setprio(0); } while (0)
    volatile LAS int* lorder = (volatile LAS int*)(lds + STAGE_BYTES + 64);
    unsigned long long sched = 0ull;
    const int mykt = kt_of(lane, rank);
#define NEED(nd) do { if ((nd) > ready) { \
        if (wid == 0) { const unsigned long long t0 = __builtin_amdgcn_s_memrealtime(); int cnt = ready; \
            for (;;) { \
                const unsigned long long sm = ld_sum(fl, pm & 1, call) << 8; \
                bool ok = !((sched >> lane) & 1ull) && (lane < 8 ? ld_sys(myflag) >= call : ((sm >> lane) & 1ull) != 0ull); \
                bool giveup = false; \
                if (cnt + __builtin_popcountll(__ballot(ok)) < (nd)) { \
                    giveup = __builtin_amdgcn_readfirstlane(__hip_atomic_load(tmo, __ATOMIC_RELAXED, __HIP_MEMORY_SCOPE_AGENT)) == call || __builtin_amdgcn_s_memrealtime() - t0 > DEADLINE; \
                    if (giveup) { if (lane == 0) { __hip_atomic_store(tmo, call, __ATOMIC_RELAXED, __HIP_MEMORY_SCOPE_AGENT); lready[1] = 1u; } ok = !((sched >> lane) & 1ull); } } \
                const unsigned long long bal = __ballot(ok); \
                if (ok) lorder[cnt + __builtin_popcountll(bal & ((1ull << lane) - 1ull))] = mykt; \
                sched |= bal; cnt += __builtin_popcountll(bal); \
                if (cnt >= (nd)) break; \
                if ((nd) > 10 && (nd) < ((pm & 1) ? 64 - 7 * ND + ODD_FAST : POLL_FAST_FROM)) __builtin_amdgcn_s_sleep(SLOW_SLEEP); else __builtin_amdgcn_s_sleep(POLL_SLEEP); } \
            ACQ_AGENT; \
            if (lane == 0) { *lready = (unsigned)cnt; \
                if (cnt == 64 && __hip_atomic_load(tmo, __ATOMIC_RELAXED, __HIP_MEMORY_SCOPE_AGENT) == call) lready[1] = 1u; }   \
            asm volatile("s_waitcnt vmcnt(0) lgkmcnt(0)" ::: "memory"); } \
        CFENCE; BAR; CFENCE; \
        ready = __builtin_amdgcn_readfirstlane((int)*lready); } } while (0)
#define KOFF(t) ((size_t)(unsigned)__builtin_amdgcn_readfirstlane(lorder[t]) * SLAB)

    f32x4 acc[2][2][4][2];
#pragma unroll
    for (int a = 0; a < 2; ++a)
#pragma unroll
        for (int b = 0; b < 2; ++b)
#pragma unroll
            for (int m = 0; m < 4; ++m)
#pragma unroll
                for (int n = 0; n < 2; ++n) acc[a][b][m][n] = (f32x4){0.f, 0.f, 0.f, 0.f};
    bf16x8 At[4][2], B0[2][2], B1[2][2];
    if (tid == 0) lready[1] = 0u;
    NEED(4);
    size_t o1, o2, o3;
    {
        const size_t k0 = KOFF(0); o1 = KOFF(1); o2 = KOFF(2); o3 = KOFF(3);
        STAGE(SB(0, 0), gB + k0); STAGE(SA(0, 0), gA + k0); STAGE(SB(0, 1), gB + k0 + IMG); STAGE(SA(0, 1), gA + k0 + IMG);
        if (wr == 1) BAR;
        WAIT_V(4); BAR;
        STAGE(SB(1, 0), gB + o1); STAGE(SA(1, 0), gA + o1); STAGE(SB(1, 1), gB + o1 + IMG);
        WAIT_V(6); BAR;
    }
    STAMP(2);
    for (int t = 0; t < 64; t += 2) {
        LDB(B0, 0, 0); SCHED; LDA(At, 0, 0); STAGE(SA(1, 1), gA + o1 + IMG);
        WAIT_L(8); BAR; WAIT_L(0); MMA(0, 0, At, B0); BAR; SCHED;
        LDB(B1, 0, 1); STAGE(SB(0, 0), gB + o2);
        BAR; WAIT_L(0); MMA(0, 1, At, B1); BAR;
        LDA(At, 0, 1); STAGE(SA(0, 0), gA + o2);
        BAR; WAIT_L(0); MMA(1, 0, At, B0); BAR; SCHED;
        STAGE(SB(0, 1), gB + o2 + IMG);
        WAIT_V(6); BAR; MMA(1, 1, At, B1); BAR;
        LDB(B0, 1, 0); SCHED; LDA(At, 1, 0); STAGE(SA(0, 1), gA + o2 + IMG);
        WAIT_L(8); BAR; WAIT_L(0); MMA(0, 0, At, B0); BAR; SCHED;
        LDB(B1, 1, 1); STAGE(SB(1, 0), gB + o3);
        BAR; WAIT_L(0); MMA(0, 1, At, B1); BAR;
        LDA(At, 1, 1); STAGE(SA(1, 0), gA + o3);
        BAR; WAIT_L(0); MMA(1, 0, At, B0); BAR; SCHED;
        STAGE(SB(1, 1), gB + o3 + IMG);
        WAIT_V(6); BAR; MMA(1, 1, At, B1); BAR;
        o1 = o3;
        if (t + 6 <= 64) { NEED(t + 6); o2 = KOFF(t + 4); o3 = KOFF(t + 5); }
    }
    STAMP(3);
    {
        int tz = threadIdx.x; asm volatile("" : "+v"(tz));
        const int lane2 = tz & 63, fr2 = lane2 & 15, fq2 = lane2 >> 4;
        const bool bad = lready[1] != 0u;
        const int owner = pm >> 1;
        if (wr == 0) BAR;
        if (owner == rank) {
            const auto rout = __builtin_amdgcn_make_buffer_rsrc(p.out, 0, (int)OUT_BYTES, 0x00020000);
            const unsigned base = (unsigned)((pm & 1) * 256 + wr * 64 + fr2) * 65536u + (unsigned)(rank * 4096 + pn * 256 + wc * 32 + fq2 * 8) * 2u;
#pragma unroll
            for (int ai = 0; ai < 2; ++ai)
#pragma unroll
                for (int m = 0; m < 4; ++m)
#pragma unroll
                    for (int bj = 0; bj < 2; ++bj) { u32x4 w; store_tile_rows(acc[ai][bj][m][0], acc[ai][bj][m][1], bad, w);
                        __builtin_amdgcn_raw_buffer_store_b128(w, rout, (int)(base + (unsigned)(ai * 128 + m * 16) * 65536u + (unsigned)bj * 256u), 0, 0); }
            WAIT_V(0); BAR; CFENCE;
        } else {
            const auto rs = __builtin_amdgcn_make_buffer_rsrc(wsl, 0, (int)WS_NEED, 0x00020000);
            const unsigned base = OFF_O + (unsigned)(pm * 16 + pn) * 131072u + (unsigned)(wr * 64 + fr2) * 512u + (unsigned)(wc * 32 + fq2 * 8) * 2u;
            unsigned* oflag = (unsigned*)p.ws[owner] + (F_OUT + rank * 64 + (pm & 1) * 14 + pn) * 32;
#pragma unroll
            for (int ai = 0; ai < 2; ++ai) {
#pragma unroll
                for (int m = 0; m < 4; ++m)
#pragma unroll
                    for (int bj = 0; bj < 2; ++bj) { u32x4 w; store_tile_rows(acc[ai][bj][m][0], acc[ai][bj][m][1], bad, w);
                        __builtin_amdgcn_raw_buffer_store_b128(w, rs, (int)(base + (unsigned)(ai * 128 + m * 16) * 512u + (unsigned)bj * 256u), 0, AUX_SYS); }
                WAIT_V(0); BAR; CFENCE;
                if (tz == 0) {
                    st_sys(oflag, 2u * call - 1u + (unsigned)ai);
                }
            }
        }
    }
#undef NEED
}

__device__ __forceinline__ void gemm_r2(const Params& p, LAS unsigned char* lds, unsigned call, int a, int b) {
    const int tid = threadIdx.x, wid = __builtin_amdgcn_readfirstlane(tid >> 6), lane = tid & 63, wr = wid >> 2, wc = wid & 3, fr = lane & 15, fq = lane >> 4;
    const int rank = p.rank;
    unsigned char* wsl = p.ws[rank]; unsigned* fl = (unsigned*)wsl; unsigned* tmo = fl + F_TMO * 32;
    if (tid < 64) {
        if (lane < 8) wait_ge(fl + (F_SFLAG + rank * 8 + lane) * 32, call, tmo, call);
        const unsigned long long t0 = __builtin_amdgcn_s_memrealtime();
        while (ld_sum(fl, (a >> 1) & 1, call) != 0x00ffffffffffffffull || ld_sum(fl, 2, call) != 0x7full) {
            if (__builtin_amdgcn_readfirstlane(__hip_atomic_load(tmo, __ATOMIC_RELAXED, __HIP_MEMORY_SCOPE_AGENT)) == call || __builtin_amdgcn_s_memrealtime() - t0 > DEADLINE) {
                if (lane == 0) __hip_atomic_store(tmo, call, __ATOMIC_RELAXED, __HIP_MEMORY_SCOPE_AGENT); break; }
            __builtin_amdgcn_s_sleep(16);
        }
        ACQ_AGENT; WAIT_V(0);
    }
    CFENCE; BAR; CFENCE;
    const char* gA = (const char*)wsl + OFF_A + (size_t)a * IMG + tid * 16;
    const char* gB = (const char*)wsl + OFF_B + (size_t)b * IMG + tid * 16;
    const unsigned ldsw = (unsigned)wid * 1024u;
    const int aoff = lds_byte(wr * 64 + fr, fq * 8), boff = lds_byte(wc * 32 + fr, fq * 8);
    f32x4 acc[4][2];
#pragma unroll
    for (int m = 0; m < 4; ++m)
#pragma unroll
        for (int n = 0; n < 2; ++n) acc[m][n] = (f32x4){0.f, 0.f, 0.f, 0.f};
#define STAGE2(slot, kt) do { STAGE((slot) * 32768, gA + (size_t)(kt) * SLAB); STAGE((slot) * 32768 + 16384, gB + (size_t)(kt) * SLAB); } while (0)
#define FRAG2(Af, Bf, slot) do { \
        _Pragma("unroll") for (int n = 0; n < 2; ++n) _Pragma("unroll") for (int k = 0; k < 2; ++k) Bf[n][k] = *(const LAS bf16x8*)(lds + (slot) * 32768 + 16384 + boff + n * 2048 + k * 1024); \
        _Pragma("unroll") for (int m = 0; m < 4; ++m) _Pragma("unroll") for (int k = 0; k < 2; ++k) Af[m][k] = *(const LAS bf16x8*)(lds + (slot) * 32768 + aoff + m * 2048 + k * 1024); } while (0)
#define MMA2(Af, Bf) do { __builtin_amdgcn_s_setprio(1); _Pragma("unroll") for (int k = 0; k < 2; ++k) _Pragma("unroll") for (int m = 0; m < 4; ++m) _Pragma("unroll") for (int n = 0; n < 2; ++n) \
        acc[m][n] = __builtin_amdgcn_mfma_f32_16x16x32_bf16(Bf[n][k], Af[m][k], acc[m][n], 0, 0, 0); __builtin_amdgcn_s_setprio(0); } while (0)
    bf16x8 A0[4][2], Bf0[2][2], A1[4][2], Bf1[2][2];
    STAMP(6);
    STAGE2(0, 0); STAGE2(1, 1); STAGE2(2, 2); STAGE2(3, 3);
    WAIT_V(12); BAR; CFENCE;
    FRAG2(A0, Bf0, 0);
    for (int t = 0; t < 64; t += 4) {
        __builtin_amdgcn_s_waitcnt(0xC07F); WAIT_V(8); BAR; SCHED; STAGE2(0, (t + 4) & 63); FRAG2(A1, Bf1, 1); SCHED; MMA2(A0, Bf0); SCHED;
        __builtin_amdgcn_s_waitcnt(0xC07F); WAIT_V(8); BAR; SCHED; STAGE2(1, (t + 5) & 63); FRAG2(A0, Bf0, 2); SCHED; MMA2(A1, Bf1); SCHED;
        __builtin_amdgcn_s_waitcnt(0xC07F); WAIT_V(8); BAR; SCHED; STAGE2(2, (t + 6) & 63); FRAG2(A1, Bf1, 3); SCHED; MMA2(A0, Bf0); SCHED;
        __builtin_amdgcn_s_waitcnt(0xC07F); WAIT_V(8); BAR; SCHED; STAGE2(3, (t + 7) & 63); FRAG2(A0, Bf0, 0); SCHED; MMA2(A1, Bf1); SCHED;
    }
    WAIT_L(0);
    STAMP(7);
    const bool bad = __hip_atomic_load(tmo, __ATOMIC_RELAXED, __HIP_MEMORY_SCOPE_AGENT) == call;
    const int owner = a >> 2;
    if (owner == rank) {
        const auto rout = __builtin_amdgcn_make_buffer_rsrc(p.out, 0, (int)OUT_BYTES, 0x00020000);
        const unsigned base = (unsigned)((a & 3) * 128 + wr * 64 + fr) * 65536u + (unsigned)(rank * 4096 + b * 128 + wc * 32 + fq * 8) * 2u;
#pragma unroll
        for (int m = 0; m < 4; ++m) { u32x4 w; store_tile_rows(acc[m][0], acc[m][1], bad, w);
            __builtin_amdgcn_raw_buffer_store_b128(w, rout, (int)(base + (unsigned)(m * 16) * 65536u), 0, 0); }
    } else {
        const auto rs = __builtin_amdgcn_make_buffer_rsrc(wsl, 0, (int)WS_NEED, 0x00020000);
        const unsigned base = OFF_O + (unsigned)((a >> 1) * 16 + (b >> 1)) * 131072u + (unsigned)((a & 1) * 2 + (b & 1)) * 32768u + (unsigned)(wr * 64 + fr) * 256u + (unsigned)(wc * 32 + fq * 8) * 2u;
#pragma unroll
        for (int m = 0; m < 4; ++m) { u32x4 w; store_tile_rows(acc[m][0], acc[m][1], bad, w);
            __builtin_amdgcn_raw_buffer_store_b128(w, rs, (int)(base + (unsigned)(m * 16) * 256u), 0, AUX_SYS); }
    }
    WAIT_V(0); BAR; CFENCE;
    if (owner != rank && tid == 0) {
        st_sys((unsigned*)p.ws[owner] + (F_OUT + rank * 64 + 28 + (a & 3) * 4 + (b - 28)) * 32, call);
    }
}

__global__ __launch_bounds__(512, 2) void fused_gemm_a2a(Params p) {
    extern __shared__ __attribute__((aligned(16))) unsigned char shm[];
    LAS unsigned char* lds = (LAS unsigned char*)shm;
    const int tid = threadIdx.x, bid = blockIdx.x;
    unsigned* fl = (unsigned*)p.ws[p.rank];
    volatile LAS unsigned* lcall = (volatile LAS unsigned*)(lds + STAGE_BYTES + 16);
    if (tid == 0) {
        const unsigned cn = __hip_atomic_fetch_add(fl + F_TICKET * 32, 1u, __ATOMIC_RELAXED, __HIP_MEMORY_SCOPE_AGENT) / (unsigned)NWG + 1u;
        *lcall = cn;
        if (bid == 0) {
            unsigned xcc; asm volatile("s_getreg_b32 %0, hwreg(HW_REG_XCC_ID)" : "=s"(xcc));
            __hip_atomic_store(fl + F_OFF * 32, cn * 8u + (xcc & 7u), __ATOMIC_RELAXED, __HIP_MEMORY_SCOPE_AGENT);
        }
    }
    STAMP(0);
    const bool is_puller = bid >= NSTAGER;
    if (is_puller && tid == 64) lcall[1] = __hip_atomic_load(fl + F_OFF * 32, __ATOMIC_RELAXED, __HIP_MEMORY_SCOPE_AGENT);
    if (!is_puller) stage_units(p, lds, lcall, bid);
    __syncthreads();
    const unsigned call = __builtin_amdgcn_readfirstlane(*lcall);
    if (is_puller && (unsigned)__builtin_amdgcn_readfirstlane((int)lcall[1]) >> 3 != call) {
        __syncthreads();
        if (tid == 0) { wait_ge(fl + F_OFF * 32, call * 8u, fl + F_TMO * 32, call); lcall[1] = __hip_atomic_load(fl + F_OFF * 32, __ATOMIC_RELAXED, __HIP_MEMORY_SCOPE_AGENT); }
        __syncthreads();
    }
    if (is_puller) {
        const int w = __builtin_amdgcn_readfirstlane(tid >> 6);
        const int x = bid < 232 ? bid - 224 : (bid + __builtin_amdgcn_readfirstlane((int)lcall[1])) & 7, k = bid < 232 ? 3 : (bid - 232) >> 3;
        const unsigned cl = 0xB2u, mine = (cl >> p.rank) & 1u;
        unsigned m = (x < 4 ? (mine ? cl : ~cl) : (mine ? ~cl : cl)) & 0xffu & ~(1u << p.rank);
        for (int i = 0; i < k; ++i) m &= m - 1u;
        const int s = __builtin_ctz(m);
        puller(p, call, (s - p.rank - 1) & 7, (x & 3) * 8 + w);
        STAMP(19); return;
    }
    STAMP(1);
    __syncthreads();
    const int xcd = bid & 7, idx = bid >> 3, rg = xcd >> 1, cg = xcd & 1;
    const int n1 = 28, nw = xcd < 4 ? 29 : 28;
    if (idx < n1) {
        gemm_r1(p, lds, call, rg * 4 + (idx & 3), cg * 7 + (idx >> 2));
        STAMP(4);
    } else if (bid == 224 && tid < 64) monitor(p, call);
    volatile LAS unsigned* lq = (volatile LAS unsigned*)(lds + STAGE_BYTES + 32);
    for (;;) {
        __syncthreads();
        if (tid == 0) *lq = __hip_atomic_fetch_add(fl + (F_Q2 + xcd) * 32, 1u, __ATOMIC_RELAXED, __HIP_MEMORY_SCOPE_AGENT) - (call - 1u) * (unsigned)(16 + nw);
        __syncthreads();
        const unsigned r2 = __builtin_amdgcn_readfirstlane(*lq);
        if (r2 >= 16u) break;
        gemm_r2(p, lds, call, rg * 8 + (int)(r2 & 7u), 28 + cg * 2 + (int)(r2 >> 3));
        STAMP(5);
    }
}
}

extern "C" void kernel_launch(int rank, int nranks, ncclComm_t comm,
                              hipStream_t stream,
                              void* const* d_in, const int* in_sizes, int n_in,
                              void* d_out, int out_size,
                              const int* in_dtypes, int out_dtype,
                              void* const* d_out_all,
                              void* const* d_ws_all, size_t ws_bytes) {
    if (nranks != NR || n_in != 2 || ws_bytes < (size_t)WS_NEED || (unsigned)rank >= (unsigned)NR) { fprintf(stderr, "kernel_launch: unexpected configuration\n"); return; }
    Params p;
    p.x = (const unsigned short*)d_in[0]; p.w = (const unsigned short*)d_in[1]; p.out = (unsigned short*)d_out; p.rank = rank;
    for (int i = 0; i < NR; ++i) p.ws[i] = (unsigned char*)d_ws_all[i];
    HIP_CHECK(hipFuncSetAttribute((const void*)fused_gemm_a2a, hipFuncAttributeMaxDynamicSharedMemorySize, LDS_BYTES));
    hipLaunchKernelGGL(fused_gemm_a2a, dim3(NWG), dim3(512), LDS_BYTES, stream, p);
    HIP_CHECK(hipGetLastError());
}
```
